```python
import numpy as np

kernel_name = "gpu_a2a_gemm_m4096_k4096_n4096_bf16_reluq8"


ABSMAX_RTOL = 0.02

COMPUTE_DTYPE = "bfloat16"
EPILOGUE = "relu_quant_int8"
IN_QUANT = None
OUT_QUANT = "int8"
IN_DTYPES = {"x": "bfloat16", "w_mat": "bfloat16"}
WORLD_SIZE = 8

def _to_bf16(x):
    u = np.ascontiguousarray(x, dtype=np.float32).view(np.uint32)
    return ((u + 0x7FFF + ((u >> 16) & 1)) & 0xFFFF0000).view(np.float32)

def setup_inputs():

    x = _to_bf16(np.random.randn(4096, 4096).astype(np.float32))
    w_mat = _to_bf16((np.random.randn(4096, 4096) / np.sqrt(4096)).astype(np.float32))
    return {"x": x, "w_mat": w_mat}

def _quant_dequant(y, amax):
    scale = amax / 127.0
    q = np.clip(np.round(y / scale), -127.0, 127.0)
    return (q * scale).astype(np.float32)
def _epilogue(y):
    y = np.maximum(y, 0.0)
    return _quant_dequant(y, np.abs(y).max())

def reference(x, w_mat):
    y = (x @ w_mat).astype(np.float32)
    return _epilogue(y)
```

```cpp
#include <hip/hip_runtime.h>
#include <rccl/rccl.h>
#include <cstdio>

#define HIP_CHECK(expr) do { hipError_t _e = (expr); if (_e != hipSuccess) { \
    fprintf(stderr, "HIP error %s at %s:%d\n", hipGetErrorString(_e), __FILE__, __LINE__); \
    return; } } while (0)

#define LAS __attribute__((address_space(3)))
#define GAS __attribute__((address_space(1)))
typedef unsigned u32;
typedef unsigned long long u64;
typedef unsigned u32x4 __attribute__((ext_vector_type(4)));
typedef unsigned u32x2 __attribute__((ext_vector_type(2)));
typedef short s16x4 __attribute__((ext_vector_type(4)));
typedef short s16x8 __attribute__((ext_vector_type(8)));
typedef __bf16 bf16x8 __attribute__((ext_vector_type(8)));
typedef float f32x16 __attribute__((ext_vector_type(16)));

constexpr int NR = 8;
constexpr int NWG = 256;
constexpr int NTHREADS = 448;
constexpr int NSTEP = 64;
constexpr size_t WS_CNT = 0;
constexpr size_t WS_TMO = 2048;
constexpr size_t WS_FLAG = 4096;
constexpr size_t WS_AMAX = 8192;
constexpr size_t WS_DIAG = 65536;
constexpr size_t WS_STAGE = 1u << 20;
constexpr size_t STAGE_BYTES = 4u << 20;
#define NBUF 2
constexpr size_t WS_NEED = WS_STAGE + NBUF * STAGE_BYTES;
#define RW 6
#define RA 6
#define LAGW 2
#define LAGA 2
#define PUSH_LA 2
constexpr int LDS_W = 0;
constexpr int LDS_A = RW * 16384;
constexpr int LDS_CTL = LDS_A + RA * 8192;
constexpr int LDS_BYTES = LDS_CTL + 128;
constexpr int C_CNTW = 0, C_CNTA = 1, C_DONE = 4, C_MAX = 8, C_GMAX = 12;
constexpr u64 DEADLINE = 40000000ull;

struct Params {
    const char* x;
    const char* w;
    char* out;
    char* ws[NR];
    int rank;
    int pad;
};

#define WAITV(n) asm volatile("s_waitcnt vmcnt(%0)" ::"n"(n) : "memory")
#define WAITL(n) asm volatile("s_waitcnt lgkmcnt(%0)" ::"n"(n) : "memory")

__device__ __forceinline__ u64 rt() { return __builtin_amdgcn_s_memrealtime(); }
__device__ __forceinline__ u32 rfl(u32 v) { return __builtin_amdgcn_readfirstlane(v); }

#define GLDS8(SUF)                                                                       \
    asm volatile("s_mov_b32 m0, %9\n\ts_nop 0\n\t"                                      \
                 "global_load_lds_dwordx4 %0, %8 offset:0" SUF "\n\t"                    \
                 "global_load_lds_dwordx4 %1, %8 offset:1024" SUF "\n\t"                 \
                 "global_load_lds_dwordx4 %2, %8 offset:2048" SUF "\n\t"                 \
                 "global_load_lds_dwordx4 %3, %8 offset:3072" SUF "\n\t"                 \
                 "s_mov_b32 m0, %10\n\ts_nop 0\n\t"                                     \
                 "global_load_lds_dwordx4 %4, %8 offset:0" SUF "\n\t"                    \
                 "global_load_lds_dwordx4 %5, %8 offset:1024" SUF "\n\t"                 \
                 "global_load_lds_dwordx4 %6, %8 offset:2048" SUF "\n\t"                 \
                 "global_load_lds_dwordx4 %7, %8 offset:3072" SUF                        \
                 ::"v"(v[0]), "v"(v[1]), "v"(v[2]), "v"(v[3]), "v"(v[4]), "v"(v[5]), "v"(v[6]), "v"(v[7]), \
                   "s"(base), "s"(lds), "s"(lds + 4096) : "memory")
template <bool SC>
__device__ __forceinline__ void glds8(const u32 (&v)[8], const char* base, u32 lds) {
    if constexpr (SC) GLDS8(" sc0 sc1"); else GLDS8("");
}

__device__ __forceinline__ void lds_store(u32 addr, u32 val) {
    asm volatile("s_mov_b64 exec, 1\n\tds_write_b32 %0, %1\n\ts_mov_b64 exec, -1" ::"v"(addr), "v"(val) : "memory");
}
__device__ __forceinline__ u32 lds_load_min4(u32 addr) {
    u32x4 d;
    asm volatile("ds_read_b128 %0, %1\n\ts_waitcnt lgkmcnt(0)" : "=&v"(d) : "v"(addr) : "memory");
    return rfl(min(min(d[0], d[1]), min(d[2], d[3])));
}
__device__ __forceinline__ u32 lds_load_min2(u32 addr) {
    u32x2 d;
    asm volatile("ds_read_b64 %0, %1\n\ts_waitcnt lgkmcnt(0)" : "=&v"(d) : "v"(addr) : "memory");
    return rfl(min(d[0], d[1]));
}

#define STAMP(slot) do {} while (0)

struct KOrder {
    int rank, t, i, j;
    __device__ __forceinline__ void init(int r) { rank = r; t = 0; i = 0; j = 0; }
    __device__ __forceinline__ int src() const { return t < 8 ? rank : ((rank + 1 + i) & 7); }
    __device__ __forceinline__ int stage() const { return t < 8 ? t : j; }
    __device__ __forceinline__ void next() { if (t >= 8) { if (++i == 7) { i = 0; ++j; } } ++t; }
};

__global__ void __launch_bounds__(NTHREADS) fused_a2a_gemm(Params P) {
    extern __shared__ __attribute__((aligned(1024))) char smem[];
    const u32 lds0 = (u32)(size_t)(LAS char*)smem;
    const u32 ctl = lds0 + LDS_CTL;
    const int lane = threadIdx.x & 63;
    const int wid = rfl(threadIdx.x >> 6);
    const int rank = P.rank & 7;
    const int xcd = blockIdx.x & 7, idx = blockIdx.x >> 3;
    const int tm = idx & 7, tn = idx >> 3;
    const int col0 = xcd * 512 + tn * 128;
    char* const myws = P.ws[rank];
    GAS u32* const cntp = (GAS u32*)(myws + WS_CNT) + blockIdx.x;
    GAS u32* const tmo = (GAS u32*)(myws + WS_TMO);

    if (wid == 6) {
        const int dest = idx & 7, sub = (idx >> 3) * 8 + xcd;
        STAMP(0);
        if (dest == rank) { __builtin_amdgcn_s_barrier(); return; }
        const char* src = P.x + (size_t)(dest * 512 + sub * 16) * 1024;
        const u32 vs = (lane >> 3) * 1024 + (lane & 7) * 16;
        const u32 vs2 = vs + 8192;
        u32x4 d[16];
        u32 cnt;
        asm volatile("global_load_dword %0, %1, off sc1" : "=&v"(cnt) : "v"(cntp) : "memory");
#pragma unroll
        for (int j = 0; j < 8; ++j) {
            asm volatile("global_load_dwordx4 %0, %1, %2 offset:%3" : "=&v"(d[2 * j]) : "v"(vs), "s"(src), "n"(j * 128) : "memory");
            asm volatile("global_load_dwordx4 %0, %1, %2 offset:%3" : "=&v"(d[2 * j + 1]) : "v"(vs2), "s"(src), "n"(j * 128) : "memory");
        }
        __builtin_amdgcn_s_barrier();
        asm volatile("s_waitcnt vmcnt(0)"
                     : "+v"(cnt), "+v"(d[0]), "+v"(d[1]), "+v"(d[2]), "+v"(d[3]), "+v"(d[4]), "+v"(d[5]), "+v"(d[6]), "+v"(d[7]),
                       "+v"(d[8]), "+v"(d[9]), "+v"(d[10]), "+v"(d[11]), "+v"(d[12]), "+v"(d[13]), "+v"(d[14]), "+v"(d[15])
                     :: "memory");
        const u32 call = rfl(cnt) + 1;
        char* const pws = P.ws[dest];
        char* dst = pws + WS_STAGE + (size_t)(call % NBUF) * STAGE_BYTES + (size_t)(rank * 8) * 65536 + (size_t)sub * 2048;
        char* const flag = pws + WS_FLAG + (sub >> 2) * 128 + (rank * 4 + (sub & 3)) * 4;
        const u32 vd = lane * 16, vz = 0;
        auto put = [&](int j) {
            char* p = dst + (size_t)j * 65536;
            asm volatile("global_store_dwordx4 %0, %1, %3 sc0 sc1\n\t"
                         "global_store_dwordx4 %0, %2, %3 offset:1024 sc0 sc1\n\ts_nop 1"
                         ::"v"(vd), "v"(d[2 * j]), "v"(d[2 * j + 1]), "s"(p) : "memory");
        };
        auto sig = [&](int j) {
            const u32 val = call * 16 + j + 1;
            asm volatile("s_mov_b64 exec, 1\n\tglobal_store_dword %0, %1, %2 sc0 sc1\n\ts_mov_b64 exec, -1"
                         ::"v"(vz), "v"(val), "s"(flag) : "memory");
        };
#pragma unroll
        for (int j = 0; j < PUSH_LA; ++j) put(j);
#pragma unroll
        for (int j = 0; j < 8; ++j) {
            constexpr int LA = PUSH_LA;
            const int later = (j + LA - 1 < 7 ? j + LA - 1 : 7) - j;
            const int flags = (j < LA ? j : LA - 1);
            switch (2 * later + flags) {
#define CASE(n) case n: WAITV(n); break;
                CASE(0) CASE(1) CASE(2) CASE(3) CASE(4) CASE(5) CASE(6) CASE(7) CASE(8) CASE(9) CASE(10) CASE(11)
                CASE(12) CASE(13) CASE(14) CASE(15) CASE(16) CASE(17) CASE(18) CASE(19) CASE(20) CASE(21)
#undef CASE
                default: WAITV(0);
            }
            sig(j);
            if (j + LA < 8) put(j + LA);
        }
        return;
    }

    if (wid == 0) {
        if (lane < 32) ((LAS u32*)(smem + LDS_CTL))[lane] = 0;
        WAITL(0);
    }

    if (wid == 4) {
        __builtin_amdgcn_s_setprio(2);
        u32 v0[8], v1[8];
#pragma unroll
        for (int i = 0; i < 8; ++i) {
            const u32 q = lane >> 4, ch = (lane & 15) ^ (q << 2);
            v0[i] = (4 * i + q) * 8192 + col0 * 2 + ch * 16 - (i & 3) * 1024;
            v1[i] = v0[i] + 32 * 8192;
        }
        __builtin_amdgcn_s_barrier();
        KOrder ko; ko.init(rank);
        u32 freed = 0; int slot = 0, pub = 0;
        for (int t = 0; t < NSTEP; ++t) {
            if (t >= RW && freed < (u32)(t - RW + 1)) {
                freed = lds_load_min4(ctl + C_DONE * 4);
                if (freed < (u32)(t - RW + 1)) {
                    WAITV(0); pub = t; lds_store(ctl + C_CNTW * 4, t);
                    do { freed = lds_load_min4(ctl + C_DONE * 4); } while (freed < (u32)(t - RW + 1));
                }
            }
            const char* base = P.w + (size_t)(ko.src() * 8 + ko.stage()) * (64 * 8192);
            const u32 l = lds0 + LDS_W + slot * 16384;
            glds8<false>(v0, base, l);
            glds8<false>(v1, base, l + 8192);
            WAITV(16 * LAGW);
            if (t - LAGW + 1 > pub) { pub = t - LAGW + 1; lds_store(ctl + C_CNTW * 4, pub); }
            ko.next(); if (++slot == RW) slot = 0;
        }
#pragma unroll
        for (int e = LAGW - 1; e >= 0; --e) {
            if (e == 2) WAITV(32); else if (e == 1) WAITV(16); else WAITV(0);
            lds_store(ctl + C_CNTW * 4, NSTEP - e);
        }
        STAMP(2);
        return;
    }

    if (wid == 5) {
        __builtin_amdgcn_s_setprio(2);
        u32 cnt;
        asm volatile("global_load_dword %0, %1, off sc1" : "=&v"(cnt) : "v"(cntp) : "memory");
        u32 vo[8], vr[8];
#pragma unroll
        for (int i = 0; i < 8; ++i) {
            const u32 row = 8 * i + (lane >> 3), ch = (lane & 7) ^ ((row >> 1) & 7);
            vo[i] = (rank * 512 + tm * 64 + row) * 1024 + ch * 16 - (i & 3) * 1024;
            vr[i] = (tm * 64 + row) * 128 + ch * 16 - (i & 3) * 1024;
        }
        __builtin_amdgcn_s_barrier();
        u32 freed = 0; int slot = 0, pub = 0;
        for (int t = 0; t < 8; ++t) {
            if (t >= RA && freed < (u32)(t - RA + 1)) {
                WAITV(0); pub = t; lds_store(ctl + C_CNTA * 4, t);
                do { freed = lds_load_min4(ctl + C_DONE * 4); } while (freed < (u32)(t - RA + 1));
            }
            glds8<false>(vo, P.x + t * 128, lds0 + LDS_A + slot * 8192);
            WAITV(8 * LAGA);
            if (t - LAGA + 1 > pub) { pub = t - LAGA + 1; lds_store(ctl + C_CNTA * 4, pub); }
            if (++slot == RA) slot = 0;
        }
        asm volatile("" : "+v"(cnt));
        const u32 call = rfl(cnt) + 1;
        const char* stg = myws + WS_STAGE + (size_t)(call % NBUF) * STAGE_BYTES;
        const char* flagp = myws + WS_FLAG + tm * 128;
        const u32 vf = (lane & 31) * 4;
        u32 fl = 0;
        bool dead = false;
        int i = 0, j = 0;
        for (int t = 8; t < NSTEP; ++t) {
            const int src = (rank + 1 + i) & 7;
            const u32 thr = call * 16 + j + 1;
            bool drained = false;
            if (((u32)(__ballot(fl >= thr) >> (src * 4)) & 15u) != 15u) {
                WAITV(0); pub = t; lds_store(ctl + C_CNTA * 4, t); drained = true;
                const u64 t0 = rt();
                for (;;) {
                    asm volatile("global_load_dword %0, %1, %2 sc0 sc1\n\ts_waitcnt vmcnt(0)" : "=&v"(fl) : "v"(vf), "s"(flagp) : "memory");
                    if (((u32)(__ballot(fl >= thr) >> (src * 4)) & 15u) == 15u) break;
                    if (__hip_atomic_load(tmo, __ATOMIC_RELAXED, __HIP_MEMORY_SCOPE_AGENT) == call || rt() - t0 > DEADLINE) {
                        __hip_atomic_store(tmo, call, __ATOMIC_RELAXED, __HIP_MEMORY_SCOPE_AGENT);
                        dead = true; break;
                    }
                }
                if (dead) break;
            }
            if (freed < (u32)(t - RA + 1)) {
                freed = lds_load_min4(ctl + C_DONE * 4);
                if (freed < (u32)(t - RA + 1)) {
                    if (!drained) { WAITV(0); pub = t; lds_store(ctl + C_CNTA * 4, t); }
                    do { freed = lds_load_min4(ctl + C_DONE * 4); } while (freed < (u32)(t - RA + 1));
                }
            }
            glds8<true>(vr, stg + (size_t)(src * 8 + j) * 65536, lds0 + LDS_A + slot * 8192);
            WAITV(8 * LAGA);
            if (t - LAGA + 1 > pub) { pub = t - LAGA + 1; lds_store(ctl + C_CNTA * 4, pub); }
            if (++slot == RA) slot = 0;
            if (++i == 7) { i = 0; ++j; }
        }
#pragma unroll
        for (int e = LAGA - 1; e >= 0; --e) {
            if (e == 3) WAITV(24); else if (e == 2) WAITV(16); else if (e == 1) WAITV(8); else WAITV(0);
            lds_store(ctl + C_CNTA * 4, NSTEP - e);
        }
        STAMP(3);
        return;
    }

    const u32 cnt0 = __hip_atomic_load(cntp, __ATOMIC_RELAXED, __HIP_MEMORY_SCOPE_AGENT);
    const int wm = wid & 1, wn = wid >> 1;
    u32 aoff[4];
    {
        const u32 row = wm * 32 + (lane & 31), h = lane >> 5;
#pragma unroll
        for (int ks = 0; ks < 4; ++ks) aoff[ks] = LDS_A + row * 128 + (((2 * ks + h) ^ ((row >> 1) & 7)) * 16);
    }
    u32 woff[2];
    {
        const u32 h = lane >> 5, blk = (lane >> 4) & 1, q = (lane >> 2) & 3, p = lane & 3;
#pragma unroll
        for (int nb = 0; nb < 2; ++nb) {
            const u32 colb = (wn * 64 + nb * 32 + 16 * (p & 1) + 8 * blk + 4 * (p >> 1)) * 2;
            const u32 ch = (colb >> 4) ^ (q << 2);
            woff[nb] = LDS_W + (8 * h + q) * 256 + ch * 16 + (colb & 8);
        }
    }
    f32x16 acc[2] = {};
    __builtin_amdgcn_s_barrier();
    STAMP(4);

    struct Frag { bf16x8 a[4]; s16x4 w[4][2][2]; };
    auto readf = [&](Frag& f, int slotw, int slota) {
        LAS char* pa = (LAS char*)smem + slota * 8192;
        LAS char* pw = (LAS char*)smem + slotw * 16384;
#pragma unroll
        for (int ks = 0; ks < 4; ++ks) {
            f.a[ks] = *(LAS bf16x8*)(pa + aoff[ks]);
#pragma unroll
            for (int nb = 0; nb < 2; ++nb)
#pragma unroll
                for (int e = 0; e < 2; ++e)
                    f.w[ks][nb][e] = __builtin_amdgcn_ds_read_tr16_b64_v4i16((LAS s16x4*)(pw + woff[nb] + (16 * ks + 4 * e) * 256));
        }
    };
    auto mma = [&](const Frag& f) {
#pragma unroll
        for (int ks = 0; ks < 4; ++ks)
#pragma unroll
            for (int nb = 0; nb < 2; ++nb) {
                const s16x8 wv = __builtin_shufflevector(f.w[ks][nb][0], f.w[ks][nb][1], 0, 1, 2, 3, 4, 5, 6, 7);
                acc[nb] = __builtin_amdgcn_mfma_f32_32x32x16_bf16(__builtin_bit_cast(bf16x8, wv), f.a[ks], acc[nb], 0, 0, 0);
            }
    };
    u32 have = 0;
    auto need = [&](int t) {
        while (have <= (u32)t) have = lds_load_min2(ctl + C_CNTW * 4);
    };
    const u32 donea = ctl + (C_DONE + wid) * 4;
    Frag f0, f1;
    int sw = 0, sa = 0;
    auto adv = [&]() { if (++sw == RW) sw = 0; if (++sa == RA) sa = 0; };
    need(0); readf(f0, sw, sa); lds_store(donea, 1); adv();
    for (int t = 0; t < NSTEP - 2; t += 2) {
        need(t + 1); readf(f1, sw, sa); lds_store(donea, t + 2); adv();
        mma(f0);
        need(t + 2); readf(f0, sw, sa); lds_store(donea, t + 3); adv();
        mma(f1);
    }
    need(NSTEP - 1); readf(f1, sw, sa);
    mma(f0);
    mma(f1);
    STAMP(5);

    const u32 call = rfl(cnt0) + 1;
    float m = 0.f;
#pragma unroll
    for (int nb = 0; nb < 2; ++nb)
#pragma unroll
        for (int r = 0; r < 16; ++r) { acc[nb][r] = fmaxf(acc[nb][r], 0.f); m = fmaxf(m, acc[nb][r]); }
#pragma unroll
    for (int o = 32; o >= 1; o >>= 1) m = fmaxf(m, __shfl_xor(m, o));
    volatile LAS u32* ctlp = (volatile LAS u32*)(smem + LDS_CTL);
    if (lane == 0) ctlp[C_MAX + wid] = __float_as_uint(m);
    WAITL(0);
    __builtin_amdgcn_s_barrier();
    if (wid == 0) {
        const u32 mb = max(max(ctlp[C_MAX], ctlp[C_MAX + 1]), max(ctlp[C_MAX + 2], ctlp[C_MAX + 3]));
        char* pw = P.ws[0];
#pragma unroll
        for (int p = 1; p < NR; ++p) pw = (lane == p) ? P.ws[p] : pw;
        if (lane < NR)
            __hip_atomic_store((GAS u64*)(pw + WS_AMAX) + rank * NWG + blockIdx.x, ((u64)call << 32) | mb,
                               __ATOMIC_RELAXED, __HIP_MEMORY_SCOPE_SYSTEM);
        if (lane == 0) __hip_atomic_store(cntp, call, __ATOMIC_RELAXED, __HIP_MEMORY_SCOPE_AGENT);
    }
    u32 gm = 0;
    {
        GAS u64* g = (GAS u64*)(myws + WS_AMAX) + threadIdx.x;
        const u64 t0 = rt();
        for (;;) {
            u64 v[8];
#pragma unroll
            for (int p = 0; p < 8; ++p) v[p] = __hip_atomic_load(g + p * NWG, __ATOMIC_RELAXED, __HIP_MEMORY_SCOPE_SYSTEM);
            bool ok = true; gm = 0;
#pragma unroll
            for (int p = 0; p < 8; ++p) { ok &= (u32)(v[p] >> 32) == call; gm = max(gm, (u32)v[p]); }
            if (__all(ok)) break;
            if (__hip_atomic_load(tmo, __ATOMIC_RELAXED, __HIP_MEMORY_SCOPE_AGENT) == call || rt() - t0 > DEADLINE) {
                __hip_atomic_store(tmo, call, __ATOMIC_RELAXED, __HIP_MEMORY_SCOPE_AGENT);
                break;
            }
        }
    }
#pragma unroll
    for (int o = 32; o >= 1; o >>= 1) gm = max(gm, (u32)__shfl_xor((int)gm, o));
    if (lane == 0) ctlp[C_GMAX + wid] = gm;
    WAITV(0);
    WAITL(0);
    __builtin_amdgcn_s_barrier();
    STAMP(6);
    const bool bad = __hip_atomic_load(tmo, __ATOMIC_RELAXED, __HIP_MEMORY_SCOPE_AGENT) == call;
    const float amax = __uint_as_float(max(max(ctlp[C_GMAX], ctlp[C_GMAX + 1]), max(ctlp[C_GMAX + 2], ctlp[C_GMAX + 3])));
    const float scale = amax / 127.0f;
    const float inv = amax > 0.f ? 1.0f / scale : 0.f;
    char* orow = P.out + (size_t)(tm * 64 + wm * 32 + (lane & 31)) * 4096 + col0 + wn * 64 + (lane >> 5) * 16;
#pragma unroll
    for (int nb = 0; nb < 2; ++nb) {
        u32x4 o;
#pragma unroll
        for (int g = 0; g < 4; ++g) {
            u32 wd = 0;
#pragma unroll
            for (int e = 0; e < 4; ++e) {
                const float qf = fminf(__builtin_rintf(acc[nb][4 * g + e] * inv), 127.f);
                wd |= (u32)qf << (8 * e);
            }
            o[g] = bad ? 0x80808080u : wd;
        }
        *(GAS u32x4*)(orow + nb * 32) = o;
    }
    STAMP(7);
}

extern "C" void kernel_launch(int rank, int nranks, ncclComm_t comm,
                              hipStream_t stream,
                              void* const* d_in, const int* in_sizes, int n_in,
                              void* d_out, int out_size,
                              const int* in_dtypes, int out_dtype,
                              void* const* d_out_all,
                              void* const* d_ws_all, size_t ws_bytes) {
    if (nranks != NR || (unsigned)rank >= (unsigned)NR || ws_bytes < WS_NEED || n_in < 2 ||
        in_sizes[0] != 4096 * 512 || in_sizes[1] != 4096 * 4096 || out_size != 512 * 4096) {
        fprintf(stderr, "kernel_launch: unexpected problem shape\n");
        return;
    }
    HIP_CHECK(hipFuncSetAttribute((const void*)fused_a2a_gemm, hipFuncAttributeMaxDynamicSharedMemorySize, LDS_BYTES));
    Params P;
    P.x = (const char*)d_in[0];
    P.w = (const char*)d_in[1];
    P.out = (char*)d_out;
    for (int p = 0; p < NR; ++p) P.ws[p] = (char*)d_ws_all[p];
    P.rank = rank;
    P.pad = 0;
    hipLaunchKernelGGL(fused_a2a_gemm, dim3(NWG), dim3(NTHREADS), LDS_BYTES, stream, P);
    HIP_CHECK(hipGetLastError());
}
```

```cpp
#include <hip/hip_runtime.h>
#include <rccl/rccl.h>
#include <cstdio>

#define HIP_CHECK(expr) do { hipError_t _e = (expr); if (_e != hipSuccess) { \
    fprintf(stderr, "HIP error %s at %s:%d\n", hipGetErrorString(_e), __FILE__, __LINE__); \
    return; } } while (0)

#define LAS __attribute__((address_space(3)))
#define GAS __attribute__((address_space(1)))
typedef unsigned u32;
typedef unsigned long long u64;
typedef unsigned u32x4 __attribute__((ext_vector_type(4)));
typedef unsigned u32x2 __attribute__((ext_vector_type(2)));
typedef short s16x4 __attribute__((ext_vector_type(4)));
typedef short s16x8 __attribute__((ext_vector_type(8)));
typedef __bf16 bf16x8 __attribute__((ext_vector_type(8)));
typedef float f32x4 __attribute__((ext_vector_type(4)));
typedef float f32x16 __attribute__((ext_vector_type(16)));

constexpr int NR = 8;
constexpr int NWG = 256;
constexpr int NTM = 6;
constexpr int NGEMM = NTM * 32;
constexpr int NTHREADS = 640;
constexpr int NSTEP = 64;
#define M16 1
constexpr int MT = 3;
constexpr int TROWS = 86;
constexpr int APC = 11;
constexpr size_t WS_CNT = 0;
constexpr size_t WS_TMO = 2048;
constexpr size_t WS_FLAG = 4096;
constexpr size_t WS_GMAX = 8192;
constexpr size_t WS_LMAX = 16384;
constexpr size_t WS_DIAG = 65536;
constexpr size_t WS_STAGE = 1u << 20;
constexpr size_t STAGE_BYTES = 4u << 20;
#define NBUF 2
constexpr size_t WS_NEED = WS_STAGE + NBUF * STAGE_BYTES;
#define RW 7
#define RA 4
#define WD0 3
#define AD 3
#define PUSH_LA 2
#define LD_LA 3
#define T0 8
#define TA0 15
#define PEARLY 3
#define KMASK 63
constexpr int ASLOT = APC * 1024;
constexpr int LDS_W = 0;
constexpr int LDS_A = RW * 16384;
constexpr int LDS_CTL = LDS_A + RA * ASLOT + (MT * 4096 - ASLOT);
constexpr int LDS_BYTES = LDS_CTL + 128;
static_assert(LDS_BYTES <= 160 * 1024 && LDS_BYTES > 80 * 1024, "one workgroup per CU");
constexpr int C_MAX = 0  , C_GMAX = 8;
constexpr u32 BADMAX = 0xffffffffu;
constexpr u64 DEADLINE = 40000000ull;

struct Params {
    const char* x;
    const char* w;
    char* out;
    char* ws[NR];
    int rank;
    int pad;
};

#define WAITV(n) asm volatile("s_waitcnt vmcnt(%0)" ::"n"(n) : "memory")
#define WAITL(n) asm volatile("s_waitcnt lgkmcnt(%0)" ::"n"(n) : "memory")
#define FENCE() asm volatile("" ::: "memory")
#define SB() __builtin_amdgcn_sched_barrier(0)
#define BAR() __builtin_amdgcn_s_barrier()

__device__ __forceinline__ u64 rt() { return __builtin_amdgcn_s_memrealtime(); }
__device__ __forceinline__ u32 rfl(u32 v) { return __builtin_amdgcn_readfirstlane(v); }
__device__ __forceinline__ u32 row16_max(u32 c) {
    c = max(c, (u32)__builtin_amdgcn_update_dpp((int)c, (int)c, 0x128, 0xf, 0xf, false));
    c = max(c, (u32)__builtin_amdgcn_update_dpp((int)c, (int)c, 0x124, 0xf, 0xf, false));
    c = max(c, (u32)__builtin_amdgcn_update_dpp((int)c, (int)c, 0x122, 0xf, 0xf, false));
    c = max(c, (u32)__builtin_amdgcn_update_dpp((int)c, (int)c, 0x121, 0xf, 0xf, false));
    return c;
}
__device__ __forceinline__ u32 wave_max(u32 v) {
    v = row16_max(v);
    return max(max((u32)__builtin_amdgcn_readlane(v, 0), (u32)__builtin_amdgcn_readlane(v, 16)),
               max((u32)__builtin_amdgcn_readlane(v, 32), (u32)__builtin_amdgcn_readlane(v, 48)));
}
__device__ __forceinline__ int row16_min(int c) {
    c = min(c, __builtin_amdgcn_update_dpp(c, c, 0x128, 0xf, 0xf, false));
    c = min(c, __builtin_amdgcn_update_dpp(c, c, 0x124, 0xf, 0xf, false));
    c = min(c, __builtin_amdgcn_update_dpp(c, c, 0x122, 0xf, 0xf, false));
    c = min(c, __builtin_amdgcn_update_dpp(c, c, 0x121, 0xf, 0xf, false));
    return c;
}
__device__ __forceinline__ bool expired(GAS u32* tmo, u32 call, u64 t0, u32& spins) {
    if ((++spins & 63u) != 0) return false;
    if (rfl(__hip_atomic_load(tmo, __ATOMIC_RELAXED, __HIP_MEMORY_SCOPE_AGENT)) == call || rt() - t0 > DEADLINE) {
        __hip_atomic_store(tmo, call, __ATOMIC_RELAXED, __HIP_MEMORY_SCOPE_AGENT);
        return true;
    }
    return false;
}

#define STAMP(slot) do {} while (0)

struct Sched { int wl[8], ws[8]; };
constexpr Sched mk_sched() {
    Sched s{}; int posL[16] = {}, posS[16] = {}, n = 0;
    for (int j = 0; j < LD_LA; ++j) { n += 2; posL[j] = n; }
    for (int j = 0; j < 8 + PUSH_LA; ++j) {
        if (j >= PUSH_LA) s.ws[j - PUSH_LA] = n - posS[j - PUSH_LA];
        if (j < 8) { s.wl[j] = n - posL[j]; n += 2; posS[j] = n; if (j + LD_LA < 8) { n += 2; posL[j + LD_LA] = n; } }
    }
    return s;
}
constexpr Sched SCHED = mk_sched();

template <int J>
__device__ __forceinline__ void push_stage(u32x4 (&d)[8][2], u32 vs0, u32 vs1, const char* src, char* dst, char* flag,
                                           u32 call, int wid, u32 vd) {
    if constexpr (J >= PUSH_LA) {
        WAITV(SCHED.ws[J - PUSH_LA]);
        BAR();
        if (wid == 0) {
            const u32 val = call * 16 + (J - PUSH_LA) + 1, vz = 0;
            asm volatile("s_mov_b64 exec, 1\n\tglobal_store_dword %0, %1, %2 sc0 sc1\n\ts_mov_b64 exec, -1"
                         ::"v"(vz), "v"(val), "s"(flag) : "memory");
        }
    }
    if constexpr (J < 8) {
        WAITV(SCHED.wl[J]);
        char* p = dst + (size_t)J * 65536;
        asm volatile("global_store_dwordx4 %2, %0, %3 sc0 sc1\n\t"
                     "global_store_dwordx4 %2, %1, %3 offset:1024 sc0 sc1\n\ts_nop 1"
                     : "+v"(d[J][0]), "+v"(d[J][1]) : "v"(vd), "s"(p) : "memory");
        if constexpr (J + LD_LA < 8)
            asm volatile("global_load_dwordx4 %0, %2, %4 offset:%5\n\tglobal_load_dwordx4 %1, %3, %4 offset:%5"
                         : "=&v"(d[J + LD_LA][0]), "=&v"(d[J + LD_LA][1]) : "v"(vs0), "v"(vs1), "s"(src), "n"((J + LD_LA) * 128) : "memory");
    }
    if constexpr (J + 1 < 8 + PUSH_LA) push_stage<J + 1>(d, vs0, vs1, src, dst, flag, call, wid, vd);
}

__global__ void __launch_bounds__(NTHREADS) fused_a2a_gemm(Params P) {
    extern __shared__ __attribute__((aligned(1024))) char smem[];
    const u32 lds0 = (u32)(size_t)(LAS char*)smem;
    const int lane = threadIdx.x & 63;
    const int wid = rfl(threadIdx.x >> 6);
    const int rank = P.rank & 7;
    const int xcd = blockIdx.x & 7, idx = 31 - (blockIdx.x >> 3);
    char* const myws = P.ws[rank];
    GAS u32* const cntp = (GAS u32*)(myws + WS_CNT) + blockIdx.x;
    GAS u32* const tmo = (GAS u32*)(myws + WS_TMO);
    if (wid == 0) STAMP(0);

    if (idx >= 28) {
        const int g = blockIdx.x;
        if (g < 28 && wid < 8) {
            const int dest = (rank + 1 + g % 7) & 7, q = g / 7;
            const int row0 = q * 128 + wid * 16;
            const char* src = P.x + (size_t)(dest * 512 + row0) * 1024;
            const u32 vs0 = (lane >> 3) * 1024 + (lane & 7) * 16, vs1 = vs0 + 8192;
            u32x4 d[8][2];
            u32 cnt;
            asm volatile("global_load_dword %0, %1, off sc1" : "=&v"(cnt) : "v"(cntp) : "memory");
#pragma unroll
            for (int j = 0; j < LD_LA; ++j)
                asm volatile("global_load_dwordx4 %0, %2, %4 offset:%5\n\tglobal_load_dwordx4 %1, %3, %4 offset:%5"
                             : "=&v"(d[j][0]), "=&v"(d[j][1]) : "v"(vs0), "v"(vs1), "s"(src), "n"(j * 128) : "memory");
            asm volatile("s_waitcnt vmcnt(%1)" : "+v"(cnt) : "n"(2 * LD_LA) : "memory");
            const u32 call = rfl(cnt) + 1;
            char* const pws = P.ws[dest];
            char* dst = pws + WS_STAGE + (size_t)(call % NBUF) * STAGE_BYTES + (size_t)(rank * 8) * 65536 + (size_t)row0 * 128;
            char* const flag = pws + WS_FLAG + (rank * 4 + q) * 128;
            push_stage<0>(d, vs0, vs1, src, dst, flag, call, wid, lane * 16);
            if (threadIdx.x == 0) __hip_atomic_store(cntp, call, __ATOMIC_RELAXED, __HIP_MEMORY_SCOPE_AGENT);
            if (wid == 0) STAMP(1);
        } else if (g == 28 && wid == 0) {
            const u32 call = rfl(__hip_atomic_load(cntp, __ATOMIC_RELAXED, __HIP_MEMORY_SCOPE_AGENT)) + 1;
            char* pw = P.ws[0];
#pragma unroll
            for (int p = 1; p < NR; ++p) pw = ((lane & 7) == p) ? P.ws[p] : pw;
            GAS u64* dstp = (GAS u64*)(pw + WS_GMAX + ((lane >> 3) * 8 + rank) * 128);
            GAS u64* g0 = (GAS u64*)(myws + WS_LMAX) + lane * 16;
            static_assert(NGEMM == 192, "");
            const u64 t0 = rt();
            u32 spins = 0, m;
            for (;;) {
                u64 v[3];
#pragma unroll
                for (int e = 0; e < 3; ++e) v[e] = __hip_atomic_load(g0 + e * 1024, __ATOMIC_RELAXED, __HIP_MEMORY_SCOPE_AGENT);
                bool ok = true; m = 0;
#pragma unroll
                for (int e = 0; e < 3; ++e) { ok &= (u32)(v[e] >> 32) == call; m = max(m, (u32)v[e]); }
                if (__all(ok)) break;
                if (expired(tmo, call, t0, spins)) { m = BADMAX; break; }
            }
            m = wave_max(m);
            __hip_atomic_store(dstp, ((u64)call << 32) | m, __ATOMIC_RELAXED, __HIP_MEMORY_SCOPE_SYSTEM);
            if (lane == 0) __hip_atomic_store(cntp, call, __ATOMIC_RELAXED, __HIP_MEMORY_SCOPE_AGENT);
            STAMP(17);
        }
        return;
    }

    int ktab;
    {
        const int tt = lane - 8, j = (tt * 37) >> 8, i = tt - 7 * j;
        ktab = (lane < 8 ? rank * 8 + lane : ((rank + 1 + i) & 7) * 8 + j) & KMASK;
    }

    if (idx >= 24) {
        const int c = idx - 24;
        if (wid < 2) {
            const u32 vo = (c * 16 + wid * 8 + (lane >> 3)) * 8192 + xcd * 1024 + (lane & 7) * 128;
            const u32 gtab = (u32)ktab << 19;
#pragma unroll 1
            for (int u = T0; u < NSTEP; ++u) {
                asm volatile("s_mov_b32 m0, %2\n\ts_nop 0\n\tglobal_load_lds_dword %0, %1\n\ts_waitcnt vmcnt(2)"
                             ::"v"(vo), "s"(P.w + (u32)__builtin_amdgcn_readlane(gtab, u)), "s"(lds0 + wid * 256) : "memory");
            }
            WAITV(0);
            if (wid == 0) STAMP(19);
        } else if (wid == 2) {
            const u32 call = rfl(__hip_atomic_load(cntp, __ATOMIC_RELAXED, __HIP_MEMORY_SCOPE_AGENT)) + 1;
            if (lane == 0) __hip_atomic_store(cntp, call, __ATOMIC_RELAXED, __HIP_MEMORY_SCOPE_AGENT);
            const int r0 = (c * 8 + xcd) * 16;
            const char* stg = myws + WS_STAGE + (size_t)(call % NBUF) * STAGE_BYTES + r0 * 128;
            const char* flagp = myws + WS_FLAG;
            const int pi = (lane & 7) % 7;
            const u32 vf = (((rank + 1 + pi) & 7) * 4 + (r0 >> 7)) * 128;
            const u64 t0 = rt();
            u32 spins = 0;
            int a = TA0;
            while (a < NSTEP) {
                u32 fl;
                asm volatile("s_mov_b64 exec, 0xffff\n\tglobal_load_dword %0, %1, %2 sc0 sc1\n\ts_mov_b64 exec, -1\n\ts_waitcnt vmcnt(0)"
                             : "=&v"(fl) : "v"(vf), "s"(flagp) : "memory");
                const int rdy = min((int)rfl(row16_min(8 + pi + 7 * max((int)(fl - call * 16), 0))), NSTEP);
                if (rdy > a) {
#pragma unroll 1
                    for (; a < rdy; a += 4) {
                        const u32 vo = ((u32)__shfl(ktab, min(a + (lane >> 4), rdy - 1)) << 16) + (lane & 15) * 128;
                        asm volatile("s_mov_b32 m0, %2\n\ts_nop 0\n\tglobal_load_lds_dword %0, %1 sc0 sc1"
                                     ::"v"(vo), "s"(stg), "s"(lds0 + 512) : "memory");
                    }
                    a = rdy;
                } else if (expired(tmo, call, t0, spins)) break;
            }
            WAITV(0);
            STAMP(21);
        }
        return;
    }

    const int tm = idx % NTM, tn = idx / NTM;
    const int row0 = tm * TROWS;
    const int nrows = tm == NTM - 1 ? 512 - (NTM - 1) * TROWS : TROWS;
    const int col0 = xcd * 512 + tn * 128;

    if (wid == 8) {
        __builtin_amdgcn_s_setprio(2);
        u32 v[8];
#pragma unroll
        for (int i = 0; i < 8; ++i) {
            const u32 q = lane >> 4, ch = (lane & 15) ^ ((q << 2) | (M16 ? ((i >> 1) & 1) << 1 : 0));
            v[i] = (4 * i + q) * 8192 + col0 * 2 + ch * 16 - (i & 3) * 1024;
        }
        const u32 gtab = (u32)ktab << 19, ltab = lds0 + LDS_W + (lane % RW) * 16384;
        auto issue = [&](int u) {
            const char* base = P.w + (u32)__builtin_amdgcn_readlane(gtab, u);
            const u32 l = __builtin_amdgcn_readlane(ltab, u);
            asm volatile("s_mov_b32 m0, %10\n\ts_nop 0\n\t"
                         "global_load_lds_dwordx4 %0, %8 offset:0\n\tglobal_load_lds_dwordx4 %1, %8 offset:1024\n\t"
                         "global_load_lds_dwordx4 %2, %8 offset:2048\n\tglobal_load_lds_dwordx4 %3, %8 offset:3072\n\t"
                         "s_add_u32 m0, m0, 0x1000\n\ts_nop 0\n\t"
                         "global_load_lds_dwordx4 %4, %8 offset:0\n\tglobal_load_lds_dwordx4 %5, %8 offset:1024\n\t"
                         "global_load_lds_dwordx4 %6, %8 offset:2048\n\tglobal_load_lds_dwordx4 %7, %8 offset:3072\n\t"
                         "s_add_u32 m0, m0, 0x1000\n\ts_nop 0\n\t"
                         "global_load_lds_dwordx4 %0, %9 offset:0\n\tglobal_load_lds_dwordx4 %1, %9 offset:1024\n\t"
                         "global_load_lds_dwordx4 %2, %9 offset:2048\n\tglobal_load_lds_dwordx4 %3, %9 offset:3072\n\t"
                         "s_add_u32 m0, m0, 0x1000\n\ts_nop 0\n\t"
                         "global_load_lds_dwordx4 %4, %9 offset:0\n\tglobal_load_lds_dwordx4 %5, %9 offset:1024\n\t"
                         "global_load_lds_dwordx4 %6, %9 offset:2048\n\tglobal_load_lds_dwordx4 %7, %9 offset:3072"
                         ::"v"(v[0]), "v"(v[1]), "v"(v[2]), "v"(v[3]), "v"(v[4]), "v"(v[5]), "v"(v[6]), "v"(v[7]),
                           "s"(base), "s"(base + 32 * 8192), "s"(l) : "memory", "scc");
        };
        int u = 0;
#pragma unroll 1
        for (; u < WD0; ++u) issue(u);
        WAITV(16 * (WD0 - 1));
        BAR();
        int t = 0;
#pragma unroll 1
        for (; t < RW - 1 - WD0; ++t) {
            issue(u); issue(u + 1); u += 2;
            WAITV(32);
            BAR();
        }
#pragma unroll 1
        for (; u < NSTEP; ++u, ++t) {
            issue(u);
            WAITV(32);
            BAR();
        }
        STAMP(2);
        for (; t < NSTEP - 3; ++t) BAR();
        WAITV(16);
        BAR();
        WAITV(0);
        BAR();
        return;
    }

    if (wid == 9) {
        __builtin_amdgcn_s_setprio(2);
        u32 cnt;
        asm volatile("global_load_dword %0, %1, off sc1" : "=&v"(cnt) : "v"(cntp) : "memory");
        u32 vo[APC], vr[APC];
#pragma unroll
        for (int i = 0; i < APC; ++i) {
            const int row = 8 * i + (lane >> 3), ch = (lane & 7) ^ ((row >> 1) & 7);
            const int grow = row0 + min(row, nrows - 1);
            vo[i] = (rank * 512 + grow) * 1024 + ch * 16 - (i & 3) * 1024;
            vr[i] = grow * 128 + ch * 16 - (i & 3) * 1024;
        }
        static_assert(APC == 11 && RA >= 4 && RW >= 5 && AD < RA && WD0 >= 2 && WD0 < RW, "");
        const u32 gtab = lane < 8 ? lane * 128 : (u32)ktab << 16, ltab = lds0 + LDS_A + (lane % RA) * ASLOT;
        auto issue = [&](const u32 (&v)[APC], const char* b0, int u) {
            const char* base = b0 + (u32)__builtin_amdgcn_readlane(gtab, u);
            const u32 l = __builtin_amdgcn_readlane(ltab, u);
            asm volatile("s_mov_b32 m0, %12\n\ts_nop 0\n\t"
                         "global_load_lds_dwordx4 %0, %11 offset:0 sc0 sc1\n\tglobal_load_lds_dwordx4 %1, %11 offset:1024 sc0 sc1\n\t"
                         "global_load_lds_dwordx4 %2, %11 offset:2048 sc0 sc1\n\tglobal_load_lds_dwordx4 %3, %11 offset:3072 sc0 sc1\n\t"
                         "s_add_u32 m0, m0, 0x1000\n\ts_nop 0\n\t"
                         "global_load_lds_dwordx4 %4, %11 offset:0 sc0 sc1\n\tglobal_load_lds_dwordx4 %5, %11 offset:1024 sc0 sc1\n\t"
                         "global_load_lds_dwordx4 %6, %11 offset:2048 sc0 sc1\n\tglobal_load_lds_dwordx4 %7, %11 offset:3072 sc0 sc1\n\t"
                         "s_add_u32 m0, m0, 0x1000\n\ts_nop 0\n\t"
                         "global_load_lds_dwordx4 %8, %11 offset:0 sc0 sc1\n\tglobal_load_lds_dwordx4 %9, %11 offset:1024 sc0 sc1\n\t"
                         "global_load_lds_dwordx4 %10, %11 offset:2048 sc0 sc1"
                         ::"v"(v[0]), "v"(v[1]), "v"(v[2]), "v"(v[3]), "v"(v[4]), "v"(v[5]), "v"(v[6]), "v"(v[7]), "v"(v[8]), "v"(v[9]), "v"(v[10]),
                           "s"(base), "s"(l) : "memory", "scc");
        };
        int u = 0, t = 0;
#pragma unroll 1
        for (; u < AD; ++u) issue(vo, P.x, u);
        const char* flagp = myws + WS_FLAG;
        int pi = ((lane & 15) >> 1) % 7;
        u32 vf = (((rank + 1 + pi) & 7) * 4 + ((lane & 1) ? (row0 + nrows - 1) >> 7 : row0 >> 7)) * 128;
        asm volatile("" : "+v"(pi), "+v"(vf), "+v"(vr[0]), "+v"(vr[1]), "+v"(vr[2]), "+v"(vr[3]), "+v"(vr[4]), "+v"(vr[5]),
                          "+v"(vr[6]), "+v"(vr[7]), "+v"(vr[8]), "+v"(vr[9]), "+v"(vr[10]));
        WAITV(APC * (AD - 1));
        BAR();
#pragma unroll 1
        for (; u < 8; ++u, ++t) {
            issue(vo, P.x, u);
            WAITV(APC * (AD - 1));
            BAR();
        }
        asm volatile("" : "+v"(cnt));
        const u32 call = rfl(cnt) + 1;
        const char* stg = myws + WS_STAGE + (size_t)(call % NBUF) * STAGE_BYTES;
        auto decode = [&](u32 fl) -> int {
            return min((int)rfl(row16_min(8 + pi + 7 * max((int)(fl - call * 16), 0))), NSTEP);
        };
        auto poll = [&]() -> int {
            u32 fl;
            asm volatile("s_mov_b64 exec, 0xffff\n\tglobal_load_dword %0, %1, %2 sc0 sc1\n\ts_mov_b64 exec, -1\n\ts_waitcnt vmcnt(0)"
                         : "=&v"(fl) : "v"(vf), "s"(flagp) : "memory");
            return decode(fl);
        };
        const u32 mbox = lds0 + LDS_CTL + 64;
        auto poll_issue = [&]() {
            asm volatile("s_mov_b32 m0, %2\n\ts_mov_b64 exec, 0xffff\n\tglobal_load_lds_dword %0, %1 sc0 sc1\n\ts_mov_b64 exec, -1"
                         ::"v"(vf), "s"(flagp), "s"(mbox) : "memory");
        };
        auto poll_read = [&]() -> int {
            u32 fl;
            asm volatile("ds_read_b32 %0, %1\n\ts_waitcnt lgkmcnt(0)" : "=&v"(fl) : "v"(mbox + (lane & 15) * 4) : "memory");
            return decode(fl);
        };
        int rdy = 8;
        auto steady = [&](int end) {
            t += end - u;
#pragma unroll 1
            do {
                issue(vr, stg, u);
                WAITV(2 * APC);
                BAR();
            } while (++u < end);
        };
        int landed = t + 1;
        bool pend = false;
        while (u < NSTEP) {
            if (pend) {
                u32 vc;
                if (u == t + 1) WAITV(0);
                asm volatile("s_getreg_b32 %0, hwreg(HW_REG_IB_STS)" : "=s"(vc) :: "memory");
                if ((vc & 0xc0000fu) == 0) {
                    WAITV(0);
                    rdy = poll_read(); pend = false; landed = u;
                }
            }
            if (u == t + 1 && u >= rdy) {
                const u64 t0 = rt();
                u32 spins = 0;
                for (;;) {
                    rdy = poll();
                    if (u < rdy) break;
                    if (expired(tmo, call, t0, spins)) { u = NSTEP; break; }
                }
            }
            if (u == t + RA - 1 && u < rdy) {
                if (rdy == NSTEP) { steady(NSTEP); break; }
                landed = 0;
                if (rdy - u > PEARLY) { steady(rdy - PEARLY); continue; }
                if (rdy - u >= 3) {
                    poll_issue();
                    steady(u + 2);
                    issue(vr, stg, u);
                    rdy = poll_read();
                    ++u; ++t;
                    WAITV(2 * APC);
                    BAR();
                    continue;
                }
            }
            if (!pend) {
                const int lim = min(min(rdy, t + RA), u + (u == t + 1 ? 3 : 2));
#pragma unroll 1
                for (; u < lim; ++u) issue(vr, stg, u);
                if (u >= rdy && u < NSTEP) {
                    poll_issue();
                    pend = true;
                }
            }
            if (landed <= t + 1) {
                const int ahead = u - (t + 2);
                if (pend) { if (ahead >= 2) WAITV(2 * APC + 1); else if (ahead == 1) WAITV(APC + 1); else WAITV(1); }
                else      { if (ahead >= 2) WAITV(2 * APC);     else if (ahead == 1) WAITV(APC);     else WAITV(0); }
            }
            BAR();
            ++t;
        }
        WAITV(2 * APC);
#pragma unroll 1
        for (; t < NSTEP - 3; ++t) BAR();
        if (t < NSTEP - 2) { WAITV(APC); BAR(); ++t; }
        if (t < NSTEP - 1) { WAITV(0); BAR(); }
        STAMP(3);
        return;
    }

    const u32 cnt0 = __hip_atomic_load(cntp, __ATOMIC_RELAXED, __HIP_MEMORY_SCOPE_AGENT);
    const int wn = wid & 3, grp = wid >> 2;
    constexpr int NPA = 2, NPW = 2, NKS = 2, NM = 24;
    u32 pa[NPA], pw[NPW];
#pragma unroll
    for (int ks = 0; ks < 2; ++ks)
        pa[ks] = lds0 + LDS_A + grp * ASLOT + (lane & 15) * 128 + (((4 * ks + (lane >> 4)) ^ ((lane >> 1) & 7)) * 16);
    {
        const u32 g = lane >> 4, q = (lane >> 2) & 3, p = lane & 3;
#pragma unroll
        for (int nb = 0; nb < 2; ++nb) {
            const u32 ch = (wn * 4 + nb * 2 + (p >> 1)) ^ ((q << 2) | ((g & 1) << 1));
            pw[nb] = lds0 + LDS_W + grp * 16384 + (8 * g + q) * 256 + ch * 16 + (p & 1) * 8;
        }
    }
    f32x4 acc[12] = {};
    bf16x8 fa[2][6];
    s16x4 fw[2][2][2];
    auto readf = [&](int ks) {
#pragma unroll
        for (int nb = 0; nb < 2; ++nb)
#pragma unroll
            for (int e = 0; e < 2; ++e)
                fw[ks][nb][e] = __builtin_amdgcn_ds_read_tr16_b64_v4i16((LAS s16x4*)((LAS char*)(size_t)pw[nb] + (32 * ks + 4 * e) * 256));
#pragma unroll
        for (int mt = 0; mt < 6; ++mt) fa[ks][mt] = *(LAS bf16x8*)((LAS char*)(size_t)pa[ks] + mt * 2048);
    };
    auto mma = [&]() {
#pragma unroll
        for (int n = 0; n < NM; ++n) {
            const int ks = n / 12, mt = (n % 12) >> 1, nb = n & 1;
            const s16x8 wv = __builtin_shufflevector(fw[ks][nb][0], fw[ks][nb][1], 0, 1, 2, 3, 4, 5, 6, 7);
            acc[mt * 2 + nb] = __builtin_amdgcn_mfma_f32_16x16x32_bf16(__builtin_bit_cast(bf16x8, wv), fa[ks][mt], acc[mt * 2 + nb], 0, 0, 0);
        }
    };
    if (wid == 0) STAMP(4);
    auto readall = [&]() {
#pragma unroll
        for (int ks = 0; ks < NKS; ++ks) { readf(ks); SB(); }
    };
    int sw = grp, sa = grp;
    auto advance = [&]() {
        sw += 2; sa += 2;
        const int dw = sw >= RW ? (2 - RW) * 16384 : 2 * 16384, da = sa >= RA ? (2 - RA) * ASLOT : 2 * ASLOT;
        if (sw >= RW) sw -= RW;
        if (sa >= RA) sa -= RA;
#pragma unroll
        for (int e = 0; e < NPA; ++e) pa[e] += da;
#pragma unroll
        for (int e = 0; e < NPW; ++e) pw[e] += dw;
    };
    if (grp) BAR();
    BAR(); FENCE(); SB();
    readall();
#pragma unroll 1
    for (int t = grp; t < NSTEP - 2; t += 2) {
        SB(); __builtin_amdgcn_s_waitcnt(0xC07F); FENCE(); BAR(); SB();
        advance();
        mma();
        __builtin_amdgcn_sched_group_barrier(0x008, 1, 0);
#pragma unroll
        for (int e = 0; e < (M16 ? 16 : 5); ++e) {
            __builtin_amdgcn_sched_group_barrier(0x006, M16 ? 1 : 4, 0);
            __builtin_amdgcn_sched_group_barrier(0x008, 1, 0);
        }
        SB(); BAR(); FENCE(); SB();
        readall();
    }
    SB();
    if (!grp) { __builtin_amdgcn_s_waitcnt(0xC07F); FENCE(); BAR(); }
    SB();
    mma();
    if (wid == 0) STAMP(5);
    if (wid == 7) STAMP(20);

    constexpr int XOFF = LDS_W + ((NSTEP - 1) % RW == 0 ? 16384 : ((NSTEP - 1) % RW >= 3 ? 0 : 3 * 16384));
    static_assert(4 * MT * 4096 <= 3 * 16384 && RW >= 6, "");
    LAS char* xp = (LAS char*)smem + XOFF + wn * (MT * 4096) + lane * 16;
    if (!grp) {
#pragma unroll
        for (int n = 0; n < 12; ++n) *(LAS f32x4*)(xp + n * 1024) = acc[n];
        WAITL(0);
        BAR();
        return;
    }
    BAR(); FENCE();
    const u32 call = rfl(cnt0) + 1;
    volatile LAS u32* ctlp = (volatile LAS u32*)(smem + LDS_CTL);
    float m = 0.f;
    {
        f32x4 xv[12];
#pragma unroll
        for (int n = 0; n < 12; ++n) xv[n] = *(LAS f32x4*)(xp + n * 1024);
        SB();
        constexpr int NL = M16 ? 10 : 8;
        float ml = 0.f;
#pragma unroll
        for (int n = 0; n < 12; ++n) {
            acc[n] += xv[n];
            const float mm = fmaxf(fmaxf(acc[n][0], acc[n][1]), fmaxf(acc[n][2], acc[n][3]));
            if (n < NL) m = fmaxf(m, mm); else ml = fmaxf(ml, mm);
        }
        if ((M16 ? 80 + (lane & 15) : 64 + (lane & 31)) < nrows) m = fmaxf(m, ml);
    }
    const u32 wm = wave_max(__float_as_uint(m));
    if (lane == 0) ctlp[C_MAX + wn] = wm;
    WAITL(0);
    BAR();
    if (wn == 0) {
        const u32x4 mv = *(volatile LAS u32x4*)(smem + LDS_CTL + C_MAX * 4);
        const u32 mb = max(max(mv[0], mv[1]), max(mv[2], mv[3]));
        if (lane == 0) {
            __hip_atomic_store((GAS u64*)(myws + WS_LMAX) + (blockIdx.x - 64) * 16, ((u64)call << 32) | mb, __ATOMIC_RELAXED, __HIP_MEMORY_SCOPE_AGENT);
            __hip_atomic_store(cntp, call, __ATOMIC_RELAXED, __HIP_MEMORY_SCOPE_AGENT);
        }
        STAMP(18);
    }
    {
        LAS char* tp = (LAS char*)smem + LDS_W + 4 * 16384 + wn * (96 * 144);
        static_assert(LDS_W + 4 * 16384 + 4 * 96 * 144 <= LDS_CTL && XOFF + 4 * MT * 4096 <= LDS_W + 4 * 16384, "");
#pragma unroll
        for (int n = 0; n < 12; ++n)
            *(LAS f32x4*)(tp + ((n >> 1) * 16 + (lane & 15)) * 144 + (n & 1) * 64 + (lane >> 4) * 16) = acc[n];
        WAITL(0);
#pragma unroll
        for (int n = 0; n < 12; ++n)
            acc[n] = *(LAS f32x4*)(tp + ((n >> 2) * 32 + (lane & 31)) * 144 + (lane >> 5) * 64 + (n & 3) * 16);
    }
    u32 gm;
    {
        GAS u64* g = (lane & 8) ? (GAS u64*)(myws + WS_TMO) : (GAS u64*)(myws + WS_GMAX + (xcd * 8 + (lane & 7)) * 128);
        const u64 t0 = rt();
        u32 spins = 0;
        for (;;) {
            u64 v = 0;
            if (lane < 16) v = __hip_atomic_load(g, __ATOMIC_RELAXED, __HIP_MEMORY_SCOPE_SYSTEM);
            gm = (lane & 8) ? ((u32)v == call ? BADMAX : 0u) : (u32)v;
            if (((u32)__ballot((u32)(v >> 32) == call) & 255u) == 255u) break;
            if (expired(tmo, call, t0, spins)) { gm = BADMAX; break; }
        }
        gm = rfl(row16_max(gm));
    }
    if (wn == 0) STAMP(6);
    const bool bad = gm == BADMAX;
    const float amax = __uint_as_float(gm);
    const float scale = amax / 127.0f;
    const float inv = amax > 0.f ? 1.0f / scale : 0.f;
    char* op = P.out + (size_t)(row0 + (lane & 31)) * 4096 + col0 + wn * 32 + (lane >> 5) * 16;
#pragma unroll
    for (int mt = 0; mt < MT; ++mt) {
        u32x4 o;
#pragma unroll
        for (int g = 0; g < 4; ++g) {
            u32 wd = 0;
#pragma unroll
            for (int r = 0; r < 4; ++r)
                wd = __builtin_amdgcn_cvt_pk_u8_f32(acc[mt * 4 + g][r] * inv, r, wd);
            o[g] = bad ? 0x80808080u : wd;
        }
        if (mt * 32 + (lane & 31) < nrows) *(GAS u32x4*)(op + (size_t)mt * 32 * 4096) = o;
    }
    if (wn == 0) STAMP(7);
}

extern "C" void kernel_launch(int rank, int nranks, ncclComm_t comm,
                              hipStream_t stream,
                              void* const* d_in, const int* in_sizes, int n_in,
                              void* d_out, int out_size,
                              const int* in_dtypes, int out_dtype,
                              void* const* d_out_all,
                              void* const* d_ws_all, size_t ws_bytes) {
    if (nranks != NR || (unsigned)rank >= (unsigned)NR || ws_bytes < WS_NEED || n_in < 2 ||
        in_sizes[0] != 4096 * 512 || in_sizes[1] != 4096 * 4096 || out_size != 512 * 4096) {
        fprintf(stderr, "kernel_launch: unexpected problem shape\n");
        return;
    }
    HIP_CHECK(hipFuncSetAttribute((const void*)fused_a2a_gemm, hipFuncAttributeMaxDynamicSharedMemorySize, LDS_BYTES));
    Params P;
    P.x = (const char*)d_in[0];
    P.w = (const char*)d_in[1];
    P.out = (char*)d_out;
    for (int p = 0; p < NR; ++p) P.ws[p] = (char*)d_ws_all[p];
    P.rank = rank;
    P.pad = 0;
    hipLaunchKernelGGL(fused_a2a_gemm, dim3(NWG), dim3(NTHREADS), LDS_BYTES, stream, P);
    HIP_CHECK(hipGetLastError());
}
```

```cpp
#include <hip/hip_runtime.h>
#include <rccl/rccl.h>
#include <cstdio>

#define HIP_CHECK(expr) do { hipError_t _e = (expr); if (_e != hipSuccess) { \
    fprintf(stderr, "HIP error %s at %s:%d\n", hipGetErrorString(_e), __FILE__, __LINE__); \
    return; } } while (0)

#define LAS __attribute__((address_space(3)))
#define GAS __attribute__((address_space(1)))
typedef unsigned u32;
typedef unsigned long long u64;
typedef unsigned u32x4 __attribute__((ext_vector_type(4)));
typedef unsigned u32x2 __attribute__((ext_vector_type(2)));
typedef short s16x4 __attribute__((ext_vector_type(4)));
typedef short s16x8 __attribute__((ext_vector_type(8)));
typedef __bf16 bf16x8 __attribute__((ext_vector_type(8)));
typedef float f32x4 __attribute__((ext_vector_type(4)));
typedef float f32x16 __attribute__((ext_vector_type(16)));
typedef unsigned u32x16 __attribute__((ext_vector_type(16)));

constexpr int NR = 8;
constexpr int NWG = 256;
constexpr int NGEMM = 192;
constexpr int NTHREADS = 704;
constexpr int NG = 64;
#define NTAIL 8
#define XD 2
constexpr int NSH = (NG - NTAIL) / 2;
constexpr int NS = NSH + NTAIL;
constexpr int NBAR = NS;
static_assert(NTAIL % 2 == 0 && NTAIL >= 4 && XD >= 2 && XD <= 3, "");
constexpr size_t WS_CNT = 0;
constexpr size_t WS_TMO = 2048;
constexpr size_t WS_FLAG = 4096;
constexpr size_t WS_GMAX = 8192;
constexpr size_t WS_LMAX = 16384;
constexpr size_t WS_DIAG = 65536;
constexpr size_t WS_XCH = 9u << 20;
constexpr size_t WS_STAGE = 1u << 20;
constexpr size_t STAGE_BYTES = 4u << 20;
#define NBUF 2
static_assert(WS_STAGE + NBUF * STAGE_BYTES <= WS_XCH, "");
#define XSTR 9216
constexpr size_t WS_NEED = WS_XCH + (size_t)NGEMM * 4 * XSTR;
#define PUSH_LA 2
#define LD_LA 3
#define T0 3
#define PSPIN 5
#define ASPIN 2
#define KB 4
#define KBU 4
#define LATE0 24
#define S1DELAY 0
#define SP0 2
#define SPI 2
#define KMASK 63
constexpr int RING = 4;
constexpr int ASLOT = 22 * 1024;
constexpr int LDS_W = 0;
constexpr int LDS_A = RING * 16384;
constexpr int LDS_CTL = LDS_A + RING * ASLOT;
constexpr int LDS_BYTES = LDS_CTL + 128 + 256;
static_assert(LDS_BYTES <= 160 * 1024 && LDS_BYTES > 80 * 1024, "one workgroup per CU");
constexpr int C_MAX = 0  , C_LIM = 8;
#define PMARGIN 2
constexpr u32 BADMAX = 0xffffffffu;
constexpr u64 DEADLINE = 40000000ull;

struct Params {
    const char* x;
    const char* w;
    char* out;
    char* ws[NR];
    int rank;
    int pad;
};

#define WAITV(n) asm volatile("s_waitcnt vmcnt(%0)" ::"n"(n) : "memory")
#define WAITL(n) asm volatile("s_waitcnt lgkmcnt(%0)" ::"n"(n) : "memory")
#define FENCE() asm volatile("" ::: "memory")
#define SB() __builtin_amdgcn_sched_barrier(0)
#define BAR() __builtin_amdgcn_s_barrier()

__device__ __forceinline__ u64 rt() { return __builtin_amdgcn_s_memrealtime(); }
__device__ __forceinline__ u32 rfl(u32 v) { return __builtin_amdgcn_readfirstlane(v); }
__device__ __forceinline__ u32 row16_max(u32 c) {
    c = max(c, (u32)__builtin_amdgcn_update_dpp((int)c, (int)c, 0x128, 0xf, 0xf, false));
    c = max(c, (u32)__builtin_amdgcn_update_dpp((int)c, (int)c, 0x124, 0xf, 0xf, false));
    c = max(c, (u32)__builtin_amdgcn_update_dpp((int)c, (int)c, 0x122, 0xf, 0xf, false));
    c = max(c, (u32)__builtin_amdgcn_update_dpp((int)c, (int)c, 0x121, 0xf, 0xf, false));
    return c;
}
__device__ __forceinline__ u32 wave_max(u32 v) {
    v = row16_max(v);
    return max(max((u32)__builtin_amdgcn_readlane(v, 0), (u32)__builtin_amdgcn_readlane(v, 16)),
               max((u32)__builtin_amdgcn_readlane(v, 32), (u32)__builtin_amdgcn_readlane(v, 48)));
}
__device__ __forceinline__ int row16_min(int c) {
    c = min(c, __builtin_amdgcn_update_dpp(c, c, 0x128, 0xf, 0xf, false));
    c = min(c, __builtin_amdgcn_update_dpp(c, c, 0x124, 0xf, 0xf, false));
    c = min(c, __builtin_amdgcn_update_dpp(c, c, 0x122, 0xf, 0xf, false));
    c = min(c, __builtin_amdgcn_update_dpp(c, c, 0x121, 0xf, 0xf, false));
    return c;
}
__device__ __forceinline__ u32 pk_bf16(float lo, float hi) {
    typedef __bf16 bf16x2 __attribute__((ext_vector_type(2)));
    const bf16x2 v = {(__bf16)lo, (__bf16)hi};
    return __builtin_bit_cast(u32, v);
}
__device__ __forceinline__ bool expired(GAS u32* tmo, u32 call, u64 t0, u32& spins) {
    if ((++spins & 63u) != 0) return false;
    if (rfl(__hip_atomic_load(tmo, __ATOMIC_RELAXED, __HIP_MEMORY_SCOPE_AGENT)) == call || rt() - t0 > DEADLINE) {
        __hip_atomic_store(tmo, call, __ATOMIC_RELAXED, __HIP_MEMORY_SCOPE_AGENT);
        return true;
    }
    return false;
}

#define STAMP(slot) do {} while (0)

struct Sched { int wl[8], ws[8]; };
constexpr Sched mk_sched() {
    Sched s{}; int posL[16] = {}, posS[16] = {}, n = 0;
    for (int j = 0; j < LD_LA; ++j) { n += 2; posL[j] = n; }
    for (int j = 0; j < 8 + PUSH_LA; ++j) {
        if (j >= PUSH_LA) s.ws[j - PUSH_LA] = n - posS[j - PUSH_LA];
        if (j < 8) { s.wl[j] = n - posL[j]; n += 2; posS[j] = n; if (j + LD_LA < 8) { n += 2; posL[j + LD_LA] = n; } }
    }
    return s;
}
constexpr Sched SCHED = mk_sched();

template <int J>
__device__ __forceinline__ void push_stage(u32x4 (&d)[8][2], u32 vs0, u32 vs1, const char* src, char* dst, char* flag,
                                           u32 call, int wid, u32 vd) {
    if constexpr (J >= PUSH_LA) {
        WAITV(SCHED.ws[J - PUSH_LA]);
        BAR();
        if (wid == 0) {
            const u32 val = call * 16 + (J - PUSH_LA) + 1, vz = 0;
            asm volatile("s_mov_b64 exec, 1\n\tglobal_store_dword %0, %1, %2 sc0 sc1\n\ts_mov_b64 exec, -1"
                         ::"v"(vz), "v"(val), "s"(flag) : "memory");
        }
    }
    if constexpr (J < 8) {
        if constexpr (J == 1 && S1DELAY > 0) __builtin_amdgcn_s_sleep(S1DELAY);
        WAITV(SCHED.wl[J]);
        char* p = dst + (size_t)J * 65536;
        asm volatile("global_store_dwordx4 %2, %0, %3 sc0 sc1\n\t"
                     "global_store_dwordx4 %2, %1, %3 offset:1024 sc0 sc1\n\ts_nop 1"
                     : "+v"(d[J][0]), "+v"(d[J][1]) : "v"(vd), "s"(p) : "memory");
        if constexpr (J + LD_LA < 8)
            asm volatile("global_load_dwordx4 %0, %2, %4 offset:%5\n\tglobal_load_dwordx4 %1, %3, %4 offset:%5"
                         : "=&v"(d[J + LD_LA][0]), "=&v"(d[J + LD_LA][1]) : "v"(vs0), "v"(vs1), "s"(src), "n"((J + LD_LA) * 128) : "memory");
    }
    if constexpr (J + 1 < 8 + PUSH_LA) push_stage<J + 1>(d, vs0, vs1, src, dst, flag, call, wid, vd);
}

#define ML_MACROS                                                                                                   \
    ".macro RD k, ks, slot\n"                                                                                       \
    ".if (\\k) < 2\n"                                                                                               \
    "ds_read_b64_tr_b16 v[88+32*(\\ks)+2*(\\k):89+32*(\\ks)+2*(\\k)], %[pw0] offset:16384*(\\slot)+8192*(\\ks)+1024*(\\k)\n" \
    ".elseif (\\k) < 4\n"                                                                                           \
    "ds_read_b64_tr_b16 v[88+32*(\\ks)+2*(\\k):89+32*(\\ks)+2*(\\k)], %[pw1] offset:16384*(\\slot)+8192*(\\ks)+1024*((\\k)-2)\n" \
    ".elseif (\\ks) == 0 && (\\slot) < 2\n"                                                                         \
    "ds_read_b128 v[80+4*(\\k):83+4*(\\k)], %[pa00] offset:%c[aslot]*(\\slot)+2048*((\\k)-4)\n"                      \
    ".elseif (\\ks) == 0\n"                                                                                         \
    "ds_read_b128 v[80+4*(\\k):83+4*(\\k)], %[pa01] offset:%c[aslot]*((\\slot)-2)+2048*((\\k)-4)\n"                  \
    ".elseif (\\slot) < 2\n"                                                                                        \
    "ds_read_b128 v[112+4*(\\k):115+4*(\\k)], %[pa10] offset:%c[aslot]*(\\slot)+2048*((\\k)-4)\n"                    \
    ".else\n"                                                                                                       \
    "ds_read_b128 v[112+4*(\\k):115+4*(\\k)], %[pa11] offset:%c[aslot]*((\\slot)-2)+2048*((\\k)-4)\n"                \
    ".endif\n"                                                                                                      \
    ".endm\n"                                                                                                       \
    ".macro PH ks, slot, rd, cnt, xl=8\n"                                                                                 \
    "s_waitcnt lgkmcnt(%c[mt]-2)\n"                                                                                 \
    ".set ml_n, 0\n"                                                                                                \
    ".rept 2*%c[mt]\n"                                                                                              \
    ".if ml_n == 4\n"                                                                                               \
    "s_waitcnt lgkmcnt(6*(\\rd))\n"                                                                                 \
    ".endif\n"                                                                                                      \
    "v_mfma_f32_16x16x32_bf16 v[40+4*ml_n:43+4*ml_n], v[88+32*(\\ks)+4*(ml_n&1):91+32*(\\ks)+4*(ml_n&1)], "         \
    "v[96+32*(\\ks)+4*(ml_n>>1):99+32*(\\ks)+4*(ml_n>>1)], v[40+4*ml_n:43+4*ml_n]\n"                                \
    ".if (\\rd) && ml_n < 2\n"                                                                                      \
    "RD 2*ml_n, 1-(\\ks), \\slot\n"                                                                                 \
    "RD 2*ml_n+1, 1-(\\ks), \\slot\n"                                                                               \
    ".elseif (\\rd) && ml_n < %c[mt]+2\n"                                                                           \
    "RD ml_n+2, 1-(\\ks), \\slot\n"                                                                                 \
    ".elseif (\\cnt) && ml_n == %c[mt]+2\n"                                                                         \
    "s_sub_u32 %[cnt], %[cnt], 1\n"                                                                                 \
    ".elseif (\\cnt) && ml_n == %c[mt]+3\n"                                                                         \
    "s_cmp_eq_u32 %[cnt], 0\n"                                                                                      \
    ".endif\n"                                                                                                      \
    ".if (ml_n == 2 || ml_n == 5 || ml_n == 8) && (\\xl)+(ml_n-2)/3 < 8\n"                                          \
    "XL (\\xl)+(ml_n-2)/3\n"                                                                                       \
    ".endif\n"                                                                                                      \
    ".set ml_n, ml_n+1\n"                                                                                           \
    ".endr\n"                                                                                                       \
    ".endm\n"                                                                                                       \
    ".macro STEP s, cnt\n"                                                                                          \
    "PH 0, (\\s)&3, 1, 0\n"                                                                                         \
    "s_barrier\n"                                                                                                   \
    "PH 1, ((\\s)+1)&3, 1, \\cnt\n"                                                                                 \
    ".endm\n"
#define ML_LOOP                                                                                                     \
    ".set ml_n, 40\n.rept 8*%c[mt]\nv_mov_b32 v[ml_n], 0\n.set ml_n, ml_n+1\n.endr\n"                               \
    "s_mov_b32 %[cnt], (%c[nloop]+3)/4\n"                                                                           \
    "s_waitcnt lgkmcnt(0)\n"                                                                                        \
    "s_barrier\n"                                                                                                   \
    ".set ml_k, 0\n.rept %c[mt]+4\nRD ml_k, 0, 0\n.set ml_k, ml_k+1\n.endr\n"                                       \
    "1:\n"                                                                                                          \
    ".set ml_s, 0\n"                                                                                                \
    ".rept 4\n"                                                                                                     \
    ".if ml_s == ((%c[nloop]-1)&3)\n"                                                                               \
    "STEP ml_s, 1\n"                                                                                                \
    "s_cbranch_scc1 2f\n"                                                                                           \
    ".else\n"                                                                                                       \
    "STEP ml_s, 0\n"                                                                                                \
    ".endif\n"                                                                                                      \
    ".set ml_s, ml_s+1\n"                                                                                           \
    ".endr\n"                                                                                                       \
    "s_branch 1b\n"                                                                                                 \
    "2:\n"
#define ML_END "s_nop 7\ns_nop 1\n.purgem RD\n.purgem PH\n.purgem STEP"
#define ML_FRAGS                                                                                                    \
    "v88", "v89", "v90", "v91", "v92", "v93", "v94", "v95", "v96", "v97", "v98", "v99", "v100", "v101", "v102", "v103",       \
    "v104", "v105", "v106", "v107", "v108", "v109", "v110", "v111", "v112", "v113", "v114", "v115", "v116", "v117", "v118", "v119", \
    "v120", "v121", "v122", "v123", "v124", "v125", "v126", "v127", "v128", "v129", "v130", "v131", "v132", "v133", "v134", "v135", \
    "v136", "v137", "v138", "v139", "v140", "v141", "v142", "v143", "v144", "v145", "v146", "v147", "v148", "v149", "v150", "v151"

template <int MT, int lag>
__device__ __forceinline__ void consume(const Params& P, char* smem, u32 lds0, int lane, int wid, char* myws, GAS u32* cntp, GAS u32* tmo,
                                        u32 cnt0, int row0, int rh, int col0, int skip, int xcd) {
    const int wn = wid & 3;
    u32 pa[2], pw[2];
#pragma unroll
    for (int ks = 0; ks < 2; ++ks)
        pa[ks] = lds0 + LDS_A + (rh + (lane & 15)) * 128 + (((4 * ks + (lane >> 4)) ^ ((lane >> 1) & 7)) * 16);
    {
        const u32 g = lane >> 4, q = (lane >> 2) & 3, p = lane & 3;
#pragma unroll
        for (int nb = 0; nb < 2; ++nb) {
            const u32 ch = (wn * 4 + nb * 2 + (p >> 1)) ^ ((q << 2) | ((g & 1) << 1));
            pw[nb] = lds0 + LDS_W + (8 * g + q) * 256 + ch * 16 + (p & 1) * 8;
        }
    }
    static_assert(RING == 4 && LDS_W == 0, "");
    f32x16 a0, a1, a2;
    u32 lc;
    const u32 call = rfl(cnt0) + 1;
    constexpr int NV = 8 * MT, NQ = (NV + 5) / 6;
    u32 tag = (call % 65535u + 1u) << 16;
    u32 vl = lane * 16, vl2 = vl + 4096;
    char* xp = myws + WS_XCH + (size_t)(((blockIdx.x ^ lag) - 64) * 4 + wn) * XSTR;
    asm volatile("" : "+s"(tag), "+s"(xp), "+v"(vl), "+v"(vl2));
#define ACC(n) ((n) < 16 ? a0[(n) & 15] : (n) < 32 ? a1[(n) & 15] : a2[(n) & 15])
    if constexpr (!lag) {
        asm volatile(ML_MACROS ML_LOOP
                     "PH 0, %c[nloop]&3, 1, 0\n"
                     "s_barrier\n"
                     "PH 1, 0, 0, 0\n"
                     ML_END
                     : "=&{v[40:55]}"(a0), "=&{v[56:71]}"(a1), "=&{v[72:87]}"(a2), [cnt] "=&s"(lc)
                     : [pw0] "v"(pw[0]), [pw1] "v"(pw[1]), [pa00] "v"(pa[0]), [pa01] "v"(pa[0] + 2 * ASLOT),
                       [pa10] "v"(pa[1]), [pa11] "v"(pa[1] + 2 * ASLOT), [mt] "n"(MT), [aslot] "n"(ASLOT), [nloop] "n"(NSH - 1)
                     : "memory", "scc", ML_FRAGS);
        u32x4 o[NQ];
#pragma unroll
        for (int q = 0; q < NQ; ++q)
#pragma unroll
            for (int hf = 0; hf < 2; ++hf) {
                float f[3];
#pragma unroll
                for (int e = 0; e < 3; ++e) { const int n = q * 6 + hf * 3 + e; f[e] = n < NV ? ACC(n) : 0.f; }
                o[q][2 * hf] = pk_bf16(f[0], f[1]);
                o[q][2 * hf + 1] = (pk_bf16(f[2], 0.f) & 0xffffu) | tag;
            }
#pragma unroll
        for (int q = 0; q < 8; ++q) {
            if (q >= SP0 && (q - SP0) % SPI == 0) { SB(); BAR(); SB(); }
            if (q < NQ)
                asm volatile("global_store_dwordx4 %0, %1, %2 offset:%3 sc1\n\ts_nop 1"
                             ::"v"(q < 4 ? vl : vl2), "v"(o[q < NQ ? q : 0]), "s"(xp), "n"((q & 3) * 1024) : "memory");
        }
        if (wid == 0) STAMP(5);
        static_assert(NSH + 1 + (8 - SP0 + SPI - 1) / SPI <= NBAR, "");
#pragma unroll 1
        for (int t = NSH + 1 + (8 - SP0 + SPI - 1) / SPI; t < NBAR; ++t) {
            BAR();
            u32 vc;
            asm volatile("s_getreg_b32 %0, hwreg(HW_REG_IB_STS, 0, 4)" : "=s"(vc) :: "memory");
            if (vc == 0) break;
        }
        return;
    }
    u32x16 x0, x1;
    asm volatile(ML_MACROS ML_LOOP
                 ".macro XL k\n"
                 ".if (\\k) < 4\n"
                 "global_load_dwordx4 v[8+4*(\\k):11+4*(\\k)], %[vl], %[xp] offset:1024*(\\k) sc1\n"
                 ".elseif (\\k) < 7\n"
                 "global_load_dwordx4 v[8+4*(\\k):11+4*(\\k)], %[vl2], %[xp] offset:1024*((\\k)-4) sc1\n"
                 ".else\n"
                 "global_load_dwordx4 v[36:39], %[vl2], %[xp] offset:%c[xlast] sc1\n"
                 ".endif\n"
                 ".endm\n"
                 ".set ml_s, %c[nloop]\n.set ml_x, 0\n"
                 ".rept %c[xd]-1\n"
                 "PH 0, ml_s&3, 1, 0, ml_x\ns_barrier\nPH 1, (ml_s+1)&3, 1, 0, ml_x+3\n"
                 ".set ml_s, ml_s+1\n.set ml_x, ml_x+6\n.endr\n"
                 "PH 0, ml_s&3, 1, 0, ml_x\n"
                 "PH 1, 0, 0, 0\n"
                 ".purgem XL\n"
                 "s_waitcnt vmcnt(0)\n"
                 ML_END
                 : "=&{v[40:55]}"(a0), "=&{v[56:71]}"(a1), "=&{v[72:87]}"(a2), "=&{v[8:23]}"(x0), "=&{v[24:39]}"(x1), [cnt] "=&s"(lc)
                 : [pw0] "v"(pw[0]), [pw1] "v"(pw[1]), [pa00] "v"(pa[0]), [pa01] "v"(pa[0] + 2 * ASLOT),
                   [pa10] "v"(pa[1]), [pa11] "v"(pa[1] + 2 * ASLOT), [mt] "n"(MT), [aslot] "n"(ASLOT), [nloop] "n"(NS - XD),
                   [vl] "v"(vl), [vl2] "v"(vl2), [xp] "s"(xp), [xlast] "n"((NQ - 5) * 1024), [xd] "n"(XD)
                 : "memory", "scc", ML_FRAGS);
    u32x4 in[8];
#pragma unroll
    for (int q = 0; q < 8; ++q)
#pragma unroll
        for (int e = 0; e < 4; ++e) in[q][e] = q < 4 ? x0[4 * q + e] : x1[4 * (q - 4) + e];
    f32x4 acc[2 * MT];
#pragma unroll
    for (int n = 0; n < NV; ++n) acc[n >> 2][n & 3] = ACC(n);
#define XLOADS(TAIL)                                                                                                             \
    asm volatile("global_load_dwordx4 %0, %8, %10 sc1\n\tglobal_load_dwordx4 %1, %8, %10 offset:1024 sc1\n\t"                    \
                 "global_load_dwordx4 %2, %8, %10 offset:2048 sc1\n\tglobal_load_dwordx4 %3, %8, %10 offset:3072 sc1\n\t"        \
                 "global_load_dwordx4 %4, %9, %10 sc1\n\tglobal_load_dwordx4 %5, %9, %10 offset:1024 sc1\n\t"                    \
                 "global_load_dwordx4 %6, %9, %10 offset:2048 sc1\n\tglobal_load_dwordx4 %7, %9, %10 offset:%11 sc1" TAIL        \
                 : "=&v"(in[0]), "=&v"(in[1]), "=&v"(in[2]), "=&v"(in[3]), "=&v"(in[4]), "=&v"(in[5]), "=&v"(in[6]), "=&v"(in[7]) \
                 : "v"(vl), "v"(vl2), "s"(xp), "n"((NQ - 5) * 1024) : "memory")
    if (wid == 7) STAMP(20);
    {
        u32 bad = 0;
#pragma unroll
        for (int q = 0; q < 8; ++q) bad |= (in[q][1] ^ tag) | (in[q][3] ^ tag);
        if (!__all(bad < 0x10000u)) {
            const u64 t0 = rt();
            u32 spins = 0;
            for (;;) {
                XLOADS("\n\ts_waitcnt vmcnt(0)");
                bad = 0;
#pragma unroll
                for (int q = 0; q < 8; ++q) bad |= (in[q][1] ^ tag) | (in[q][3] ^ tag);
                if (__all(bad < 0x10000u)) break;
                if (expired(tmo, call, t0, spins)) break;
            }
        }
    }
    if (wn == 0) STAMP(21);
    volatile LAS u32* ctlp = (volatile LAS u32*)(smem + LDS_CTL);
    float m = 0.f;
    {
        float m0 = 0.f;
#pragma unroll
        for (int n = 0; n < NV; ++n) {
            const int q = n / 6, hf = (n % 6) / 3, e = n % 3;
            const u32 w = in[q][2 * hf + (e >> 1)];
            acc[n >> 2][n & 3] += __uint_as_float(e == 1 ? w & 0xffff0000u : w << 16);
            if (n < 8) m0 = fmaxf(m0, acc[n >> 2][n & 3]); else m = fmaxf(m, acc[n >> 2][n & 3]);
        }
        if (!skip) m = fmaxf(m, m0);
    }
    const u32 wm = wave_max(__float_as_uint(m));
    if (lane == 0) ctlp[C_MAX + wn] = wm;
    WAITL(0);
    BAR();
    if (wn == 0) {
        const u32x4 mv = *(volatile LAS u32x4*)(smem + LDS_CTL + C_MAX * 4);
        const u32 mb = max(max(mv[0], mv[1]), max(mv[2], mv[3]));
        if (lane == 0) {
            __hip_atomic_store((GAS u64*)(myws + WS_LMAX) + (blockIdx.x - 64) * 16, ((u64)call << 32) | mb, __ATOMIC_RELAXED, __HIP_MEMORY_SCOPE_AGENT);
            __hip_atomic_store(cntp, call, __ATOMIC_RELAXED, __HIP_MEMORY_SCOPE_AGENT);
        }
        STAMP(18);
    }
    f32x4 y[12];
    {
        LAS char* tp = (LAS char*)smem + wn * (96 * 144);
#pragma unroll
        for (int n = 0; n < 2 * MT; ++n)
            *(LAS f32x4*)(tp + ((n >> 1) * 16 + (lane & 15)) * 144 + (n & 1) * 64 + (lane >> 4) * 16) = acc[n];
        WAITL(0);
#pragma unroll
        for (int n = 0; n < 12; ++n)
            y[n] = *(LAS f32x4*)(tp + ((n >> 2) * 32 + (lane & 31)) * 144 + (lane >> 5) * 64 + (n & 3) * 16);
    }
    u32 gm;
    {
        GAS u64* g = (lane & 8) ? (GAS u64*)(myws + WS_TMO) : (GAS u64*)(myws + WS_GMAX + (xcd * 8 + (lane & 7)) * 128);
        const u64 t0 = rt();
        u32 spins = 0;
        for (;;) {
            u64 v = 0;
            if (lane < 16) v = __hip_atomic_load(g, __ATOMIC_RELAXED, __HIP_MEMORY_SCOPE_SYSTEM);
            gm = (lane & 8) ? ((u32)v == call ? BADMAX : 0u) : (u32)v;
            if (((u32)__ballot((u32)(v >> 32) == call) & 255u) == 255u) break;
            if (expired(tmo, call, t0, spins)) { gm = BADMAX; break; }
        }
        gm = rfl(row16_max(gm));
    }
    if (wn == 0) STAMP(6);
    const bool bad = gm == BADMAX;
    const float amax = __uint_as_float(gm);
    const float scale = amax / 127.0f;
    const float inv = amax > 0.f ? 1.0f / scale : 0.f;
    char* op = P.out + (size_t)(row0 + rh + (lane & 31)) * 4096 + col0 + wn * 32 + (lane >> 5) * 16;
#pragma unroll
    for (int mt = 0; mt < 3; ++mt) {
        u32x4 o;
#pragma unroll
        for (int g = 0; g < 4; ++g) {
            u32 wd = 0;
#pragma unroll
            for (int r = 0; r < 4; ++r)
                wd = __builtin_amdgcn_cvt_pk_u8_f32(y[mt * 4 + g][r] * inv, r, wd);
            o[g] = bad ? 0x80808080u : wd;
        }
        const int r = mt * 32 + (lane & 31);
        if (r >= skip && r < 16 * MT) *(GAS u32x4*)(op + (size_t)mt * 32 * 4096) = o;
    }
    if (wn == 0) STAMP(7);
}

__global__ void __launch_bounds__(NTHREADS) fused_a2a_gemm(Params P) {
    extern __shared__ __attribute__((aligned(1024))) char smem[];
    const u32 lds0 = (u32)(size_t)(LAS char*)smem;
    const int lane = threadIdx.x & 63;
    const int wid = rfl(threadIdx.x >> 6);
    const int rank = P.rank & 7;
    const int xcd = blockIdx.x & 7, idx = 31 - (blockIdx.x >> 3);
    char* const myws = P.ws[rank];
    GAS u32* const cntp = (GAS u32*)(myws + WS_CNT) + blockIdx.x;
    GAS u32* const tmo = (GAS u32*)(myws + WS_TMO);
    if (wid == 0) STAMP(0);

    const int kh = xcd & 1;
    int ktab;
    {
        const int g = min(lane < NSH ? 2 * lane + kh : lane + NSH, NG - 1);
        const int tt = g - 8, j = (tt * 37) >> 8, i = tt - 7 * j;
        ktab = (g < 8 ? rank * 8 + g : ((rank + 1 + i) & 7) * 8 + j) & KMASK;
    }

    if (idx >= 28) {
        const int g = blockIdx.x;
        if (g < 28 && wid < 8) {
            const int dest = (rank + 1 + g % 7) & 7, q = g / 7;
            const int row0 = q * 128 + wid * 16;
            const char* src = P.x + (size_t)(dest * 512 + row0) * 1024;
            const u32 vs0 = (lane >> 3) * 1024 + (lane & 7) * 16, vs1 = vs0 + 8192;
            u32x4 d[8][2];
            u32 cnt;
            asm volatile("global_load_dword %0, %1, off sc1" : "=&v"(cnt) : "v"(cntp) : "memory");
#pragma unroll
            for (int j = 0; j < LD_LA; ++j)
                asm volatile("global_load_dwordx4 %0, %2, %4 offset:%5\n\tglobal_load_dwordx4 %1, %3, %4 offset:%5"
                             : "=&v"(d[j][0]), "=&v"(d[j][1]) : "v"(vs0), "v"(vs1), "s"(src), "n"(j * 128) : "memory");
            asm volatile("s_waitcnt vmcnt(%1)" : "+v"(cnt) : "n"(2 * LD_LA) : "memory");
            const u32 call = rfl(cnt) + 1;
            char* const pws = P.ws[dest];
            char* dst = pws + WS_STAGE + (size_t)(call % NBUF) * STAGE_BYTES + (size_t)(rank * 8) * 65536 + (size_t)row0 * 128;
            char* const flag = pws + WS_FLAG + (rank * 4 + q) * 128;
            push_stage<0>(d, vs0, vs1, src, dst, flag, call, wid, lane * 16);
            if (threadIdx.x == 0) __hip_atomic_store(cntp, call, __ATOMIC_RELAXED, __HIP_MEMORY_SCOPE_AGENT);
            if (wid == 0) STAMP(1);
        } else if (g == 28 && wid == 9) {
            const u32 call = rfl(__hip_atomic_load(cntp, __ATOMIC_RELAXED, __HIP_MEMORY_SCOPE_AGENT)) + 1;
            char* pw = P.ws[0];
#pragma unroll
            for (int p = 1; p < NR; ++p) pw = ((lane & 7) == p) ? P.ws[p] : pw;
            GAS u64* dstp = (GAS u64*)(pw + WS_GMAX + ((lane >> 3) * 8 + rank) * 128);
            GAS u64* g0 = (GAS u64*)(myws + WS_LMAX) + lane * 16;
            static_assert(NGEMM == 192, "");
            const u64 t0 = rt();
            u32 spins = 0, m;
            for (;;) {
                u64 v[3];
#pragma unroll
                for (int e = 0; e < 3; ++e) v[e] = __hip_atomic_load(g0 + e * 1024, __ATOMIC_RELAXED, __HIP_MEMORY_SCOPE_AGENT);
                bool ok = true; m = 0;
#pragma unroll
                for (int e = 0; e < 3; ++e) { ok &= (u32)(v[e] >> 32) == call; m = max(m, (u32)v[e]); }
                if (__all(ok)) break;
                if (expired(tmo, call, t0, spins)) { m = BADMAX; break; }
            }
            m = wave_max(m);
            __hip_atomic_store(dstp, ((u64)call << 32) | m, __ATOMIC_RELAXED, __HIP_MEMORY_SCOPE_SYSTEM);
            if (lane == 0) __hip_atomic_store(cntp, call, __ATOMIC_RELAXED, __HIP_MEMORY_SCOPE_AGENT);
            STAMP(17);
        }
        if (wid < 8) {
            const u32 call = rfl(__hip_atomic_load(cntp + 256, __ATOMIC_RELAXED, __HIP_MEMORY_SCOPE_AGENT)) + 1;
            const u32 vf = (lane & 31) * 128;
            const u64 t0 = rt();
            u32 spins = 0;
            bool ok = false;
            for (;;) {
                u32 fl;
                asm volatile("global_load_dword %0, %1, %2 sc0 sc1\n\ts_waitcnt vmcnt(0)" : "=&v"(fl) : "v"(vf), "s"(myws + WS_FLAG) : "memory");
                if (__all(fl >= call * 16 + 8 || ((lane & 31) >> 2) == rank)) { ok = true; break; }
                if (expired(tmo, call, t0, spins)) break;
                __builtin_amdgcn_s_sleep(8);
            }
            if (ok) {
                const char* stg = myws + WS_STAGE + (size_t)(call % NBUF) * STAGE_BYTES;
                const u32 vo = (wid * 64 + lane) * 128;
#pragma unroll 1
                for (int u = LATE0 + (g >> 3); u < NS; u += 4)
                    asm volatile("s_mov_b32 m0, %2\n\ts_nop 0\n\tglobal_load_lds_dword %0, %1 sc0 sc1"
                                 ::"v"(vo), "s"(stg + ((u32)__builtin_amdgcn_readlane(ktab, u) << 16)), "s"(lds0 + wid * 256) : "memory");
            }
            WAITV(0);
            BAR();
            if (threadIdx.x == 0) __hip_atomic_store(cntp + 256, call, __ATOMIC_RELAXED, __HIP_MEMORY_SCOPE_AGENT);
            if (wid == 0) STAMP(23);
        }
        return;
    }

    if (idx >= 24) {
        if (wid < 4) {
            const u32 vo = ((idx - 24) * 16 + wid * 4 + (lane >> 4)) * 8192 + (xcd >> 1) * 2048 + (lane & 15) * 128;
            const u32 gtab = (u32)ktab << 19;
#pragma unroll 1
            for (int u = T0; u < NS - KB * 24 / 16; ++u) {
                asm volatile("s_mov_b32 m0, %2\n\ts_nop 0\n\tglobal_load_lds_dword %0, %1\n\ts_waitcnt vmcnt(1)"
                             ::"v"(vo), "s"(P.w + (u32)__builtin_amdgcn_readlane(gtab, u)), "s"(lds0 + wid * 256) : "memory");
            }
            WAITV(0);
            if (wid == 0) STAMP(19);
        }
        return;
    }

    const int tm = idx % 3, tn = idx / 3;
    const int row0 = tm == 2 ? 336 : tm * 176;
    const int col0 = (xcd >> 1) * 1024 + tn * 128;

    if (wid == 8) {
        __builtin_amdgcn_s_setprio(2);
        u32 v[8];
#pragma unroll
        for (int i = 0; i < 8; ++i) {
            const u32 q = lane >> 4, ch = (lane & 15) ^ ((q << 2) | (((i >> 1) & 1) << 1));
            v[i] = (4 * i + q) * 8192 + col0 * 2 + ch * 16 - (i & 3) * 1024;
        }
        const u32 gtab = (u32)ktab << 19;
        auto issue = [&](int u) {
            const char* base = P.w + (u32)__builtin_amdgcn_readlane(gtab, u);
            const u32 l = lds0 + LDS_W + (u & (RING - 1)) * 16384;
            asm volatile("s_mov_b32 m0, %10\n\ts_nop 0\n\t"
                         "global_load_lds_dwordx4 %0, %8 offset:0\n\tglobal_load_lds_dwordx4 %1, %8 offset:1024\n\t"
                         "global_load_lds_dwordx4 %2, %8 offset:2048\n\tglobal_load_lds_dwordx4 %3, %8 offset:3072\n\t"
                         "s_add_u32 m0, m0, 0x1000\n\ts_nop 0\n\t"
                         "global_load_lds_dwordx4 %4, %8 offset:0\n\tglobal_load_lds_dwordx4 %5, %8 offset:1024\n\t"
                         "global_load_lds_dwordx4 %6, %8 offset:2048\n\tglobal_load_lds_dwordx4 %7, %8 offset:3072\n\t"
                         "s_add_u32 m0, m0, 0x1000\n\ts_nop 0\n\t"
                         "global_load_lds_dwordx4 %0, %9 offset:0\n\tglobal_load_lds_dwordx4 %1, %9 offset:1024\n\t"
                         "global_load_lds_dwordx4 %2, %9 offset:2048\n\tglobal_load_lds_dwordx4 %3, %9 offset:3072\n\t"
                         "s_add_u32 m0, m0, 0x1000\n\ts_nop 0\n\t"
                         "global_load_lds_dwordx4 %4, %9 offset:0\n\tglobal_load_lds_dwordx4 %5, %9 offset:1024\n\t"
                         "global_load_lds_dwordx4 %6, %9 offset:2048\n\tglobal_load_lds_dwordx4 %7, %9 offset:3072"
                         ::"v"(v[0]), "v"(v[1]), "v"(v[2]), "v"(v[3]), "v"(v[4]), "v"(v[5]), "v"(v[6]), "v"(v[7]),
                           "s"(base), "s"(base + 32 * 8192), "s"(l) : "memory", "scc");
        };
        issue(0);
#pragma unroll 1
        for (int u = 1; u < NS; ++u) {
            issue(u);
            WAITV(16);
            if (u == KBU) {
#pragma unroll 1
                for (int k = 0; k < KB; ++k) {
                    const int n = idx + 24 * k;
                    const u32 vo = ((n & 15) * 4 + (lane >> 4)) * 8192 + (xcd >> 1) * 2048 + (lane & 15) * 128;
                    asm volatile("s_mov_b32 m0, %2\n\ts_nop 0\n\tglobal_load_lds_dword %0, %1"
                                 ::"v"(vo), "s"(P.w + (u32)__builtin_amdgcn_readlane(gtab, NS - 1 - (n >> 4))), "s"(lds0 + LDS_CTL + 128) : "memory");
                }
            }
            BAR();
        }
        WAITV(0);
        BAR();
        STAMP(2);
        return;
    }

    const char* const flagp = myws + WS_FLAG;
    const int pi8 = 8 + ((lane & 15) >> 1) % 7;
    const u32 vf = (((rank + 1 + pi8) & 7) * 4 + ((lane & 1) ? (row0 + 175) >> 7 : row0 >> 7)) * 128;
    const u32 lima = lds0 + LDS_CTL + C_LIM * 4;

    if (wid == 10) {
        const u32 call = rfl(__hip_atomic_load(cntp, __ATOMIC_RELAXED, __HIP_MEMORY_SCOPE_AGENT)) + 1;
        *(volatile LAS u32*)(smem + LDS_CTL + C_LIM * 4) = 4;
        WAITL(0);
        u32 fl, tmp, c, r, t = 0, pend = 0, lim = 4;
        asm volatile(
            "s_mov_b64 exec, 0xffff\n"
            "1:\n\t"
            "s_barrier\n\t"
            "s_mov_b32 %[r], %[nspin]\n"
            "6:\n\t"
            "s_cmp_eq_u32 %[pend], 0\n\t"
            "s_cbranch_scc1 2f\n\t"
            "s_getreg_b32 %[c], hwreg(HW_REG_IB_STS, 0, 4)\n\t"
            "s_cmp_lg_u32 %[c], 0\n\t"
            "s_cbranch_scc1 4f\n\t"
            "s_waitcnt vmcnt(0)\n\t"
            "v_subrev_u32 %[fl], %[base], %[fl]\n\t"
            "v_max_i32 %[fl], 0, %[fl]\n\t"
            "v_mad_u32_u24 %[fl], %[fl], 7, %[pi8]\n\t"
            "v_xor_b32 %[tmp], %[kh], %[fl]\n\t"
            "v_and_b32 %[tmp], 1, %[tmp]\n\t"
            "v_cmp_gt_u32 vcc, %[nsh2], %[fl]\n\t"
            "s_mov_b32 %[pend], 0\n\t"
            "s_sub_u32 %[r], %[r], 2\n\t"
            "v_cndmask_b32 %[tmp], 0, %[tmp], vcc\n\t"
            "v_mad_u32_u24 %[fl], %[tmp], 7, %[fl]\n\t"
            "s_nop 1\n\t"
            "v_min_u32_dpp %[fl], %[fl], %[fl] row_ror:8 row_mask:0xf bank_mask:0xf\n\t"
            "s_nop 1\n\t"
            "v_min_u32_dpp %[fl], %[fl], %[fl] row_ror:4 row_mask:0xf bank_mask:0xf\n\t"
            "s_nop 1\n\t"
            "v_min_u32_dpp %[fl], %[fl], %[fl] row_ror:2 row_mask:0xf bank_mask:0xf\n\t"
            "s_nop 1\n\t"
            "v_min_u32_dpp %[fl], %[fl], %[fl] row_ror:1 row_mask:0xf bank_mask:0xf\n\t"
            "v_lshrrev_b32 %[tmp], 1, %[fl]\n\t"
            "v_subrev_u32 %[fl], %[nsh], %[fl]\n\t"
            "v_max_i32 %[fl], %[fl], %[tmp]\n\t"
            "s_nop 0\n\t"
            "v_readfirstlane_b32 %[c], %[fl]\n\t"
            "s_cmp_le_u32 %[c], %[lim]\n\t"
            "s_cbranch_scc1 2f\n\t"
            "ds_write_b32 %[lima], %[fl]\n\t"
            "s_mov_b32 %[lim], %[c]\n\t"
            "s_cmp_ge_u32 %[lim], %[nbar]\n\t"
            "s_cbranch_scc1 9f\n"
            "2:\n\t"
            "s_add_u32 %[c], %[t], %[reach]\n\t"
            "s_cmp_gt_u32 %[lim], %[c]\n\t"
            "s_cbranch_scc1 3f\n\t"
            "global_load_dword %[fl], %[vf], %[flagp] sc0 sc1\n\t"
            "s_mov_b32 %[pend], 1\n"
            "4:\n\t"
            "s_sub_u32 %[r], %[r], 1\n\t"
            "s_cmp_gt_i32 %[r], 0\n\t"
            "s_cbranch_scc1 6b\n"
            "3:\n\t"
            "s_add_u32 %[t], %[t], 1\n\t"
            "s_cmp_lt_u32 %[t], %[nbar]\n\t"
            "s_cbranch_scc1 1b\n"
            "9:\n\t"
            "s_waitcnt vmcnt(0) lgkmcnt(0)\n\t"
            "s_mov_b64 exec, -1"
            : [fl] "=&v"(fl), [tmp] "=&v"(tmp), [c] "=&s"(c), [r] "=&s"(r), [t] "+s"(t), [pend] "+s"(pend), [lim] "+s"(lim)
            : [base] "s"(call * 16), [pi8] "v"(pi8), [kh] "s"(kh), [lima] "v"(lima), [vf] "v"(vf), [flagp] "s"(flagp),
              [reach] "n"(2 + PMARGIN), [nbar] "n"(NBAR), [nspin] "n"(PSPIN), [nsh] "n"(NSH), [nsh2] "n"(2 * NSH)
            : "memory", "vcc", "scc");
        return;
    }

    if (wid == 9) {
        __builtin_amdgcn_s_setprio(2);
        u32 cnt;
        asm volatile("global_load_dword %0, %1, off sc1" : "=&v"(cnt) : "v"(cntp) : "memory");
        u32 vo[4], vr[4];
#pragma unroll
        for (int i = 0; i < 4; ++i) {
            const int row = 8 * i + (lane >> 3), ch = (lane & 7) ^ ((row >> 1) & 7);
            vo[i] = (rank * 512 + row0 + row) * 1024 + ch * 16 - i * 1024;
            vr[i] = (row0 + row) * 128 + ch * 16 - i * 1024;
        }
        const u32 gtab = lane < 4 ? (2 * lane + kh) * 128 : (u32)ktab << 16;
        const u32 ltab = lds0 + LDS_A + (lane & (RING - 1)) * ASLOT;
        auto issue = [&](const u32 (&v)[4], const char* b0, u32 gs, int u, int part) {
            const char* base = b0 + ((u32)__builtin_amdgcn_readlane(gtab, u) + part * 3 * gs);
            const u32 l = __builtin_amdgcn_readlane(ltab, u) + part * 12288;
            asm volatile("s_mov_b32 m0, %7\n\ts_nop 0\n\t"
                         "global_load_lds_dwordx4 %0, %4 offset:0 sc0 sc1\n\tglobal_load_lds_dwordx4 %1, %4 offset:1024 sc0 sc1\n\t"
                         "global_load_lds_dwordx4 %2, %4 offset:2048 sc0 sc1\n\tglobal_load_lds_dwordx4 %3, %4 offset:3072 sc0 sc1\n\t"
                         "s_add_u32 m0, m0, 0x1000\n\ts_nop 0\n\t"
                         "global_load_lds_dwordx4 %0, %5 offset:0 sc0 sc1\n\tglobal_load_lds_dwordx4 %1, %5 offset:1024 sc0 sc1\n\t"
                         "global_load_lds_dwordx4 %2, %5 offset:2048 sc0 sc1\n\tglobal_load_lds_dwordx4 %3, %5 offset:3072 sc0 sc1\n\t"
                         "s_add_u32 m0, m0, 0x1000\n\ts_cmp_lg_u32 %8, 0\n\t"
                         "global_load_lds_dwordx4 %0, %6 offset:0 sc0 sc1\n\tglobal_load_lds_dwordx4 %1, %6 offset:1024 sc0 sc1\n\t"
                         "s_cbranch_scc1 1f\n\t"
                         "global_load_lds_dwordx4 %2, %6 offset:2048 sc0 sc1\n\tglobal_load_lds_dwordx4 %3, %6 offset:3072 sc0 sc1\n"
                         "1:"
                         ::"v"(v[0]), "v"(v[1]), "v"(v[2]), "v"(v[3]), "s"(base), "s"(base + gs), "s"(base + 2 * gs), "s"(l), "s"(part)
                         : "memory", "scc");
        };
#pragma unroll 1
        for (int h = 0; h < 8; ++h) {
            issue(vo, P.x, 32768, h >> 1, h & 1);
            if ((h & 1) && h > 1) { WAITV(22); BAR(); }
        }
        asm volatile("" : "+v"(cnt));
        const u32 call = rfl(cnt) + 1;
        const char* stg = myws + WS_STAGE + (size_t)(call % NBUF) * STAGE_BYTES;
        auto block = [&](int need) -> int {
            const u64 t0 = rt();
            u32 spins = 0;
            for (;;) {
                u32 fl;
                asm volatile("s_mov_b64 exec, 0xffff\n\tglobal_load_dword %0, %1, %2 sc0 sc1\n\ts_mov_b64 exec, -1\n\ts_waitcnt vmcnt(0)"
                             : "=&v"(fl) : "v"(vf), "s"(flagp) : "memory");
                int c = pi8 + 7 * max((int)(fl - call * 16), 0);
                c += (c < 2 * NSH && ((c ^ kh) & 1)) ? 7 : 0;
                c = rfl(row16_min(c));
                c = max(c >> 1, c - NSH);
                if (need < c) return c;
                if (expired(tmo, call, t0, spins)) return 1000;
            }
        };
        const u32 va = vr[0], vb = vr[1];
        int t = 2, u = 4, lim = 4;
        for (;;) {
            u32 x, cap, c, so, r;
            asm volatile(
                "1:\n\t"
                "s_mov_b32 %[r], %[nspin]\n\t"
                "s_add_u32 %[cap], %[t], 3\n\t"
                "s_min_u32 %[cap], %[cap], %[nsh]\n"
                "6:\n\t"
                "s_cmp_ge_u32 %[u], %[cap]\n\t"
                "s_cbranch_scc1 3f\n\t"
                "s_cmp_ge_u32 %[u], %[lim]\n\t"
                "s_cbranch_scc1 8f\n"
                "2:\n\t"
                "v_readlane_b32 %[so], %[gtab], %[u]\n\t"
                "v_readlane_b32 %[c], %[ltab], %[u]\n\t"
                "s_add_u32 s60, %[b0], %[so]\n\t"
                "s_addc_u32 s61, %[b1], 0\n\t"
                "s_mov_b32 m0, %[c]\n\t"
                "s_add_u32 %[u], %[u], 1\n\t"
                ".rept 5\n\t"
                "global_load_lds_dwordx4 %[va], s[60:61] offset:0 sc0 sc1\n\tglobal_load_lds_dwordx4 %[vb], s[60:61] offset:1024 sc0 sc1\n\t"
                "global_load_lds_dwordx4 %[va], s[60:61] offset:2048 sc0 sc1\n\tglobal_load_lds_dwordx4 %[vb], s[60:61] offset:3072 sc0 sc1\n\t"
                "s_add_u32 m0, m0, 0x1000\n\t"
                "s_add_u32 s60, s60, 0x1000\n\t"
                "s_addc_u32 s61, s61, 0\n\t"
                ".endr\n\t"
                "global_load_lds_dwordx4 %[va], s[60:61] offset:0 sc0 sc1\n\tglobal_load_lds_dwordx4 %[vb], s[60:61] offset:1024 sc0 sc1\n\t"
                "s_mov_b32 %[r], 1\n\t"
                "s_cmp_lt_u32 %[u], %[cap]\n\t"
                "s_cbranch_scc1 6b\n"
                "3:\n\t"
                "s_sub_u32 %[c], %[u], %[t]\n\t"
                "s_cmp_lt_u32 %[c], 3\n\t"
                "s_cbranch_scc1 5f\n\t"
                "s_waitcnt vmcnt(22)\n"
                "4:\n\t"
                "s_add_u32 %[t], %[t], 1\n\t"
                "s_cmp_lt_u32 %[u], %[nsh]\n\t"
                "s_barrier\n\t"
                "s_cbranch_scc1 1b\n\t"
                "s_branch 9f\n"
                "8:\n\t"
                "ds_read_b32 %[x], %[lima]\n\t"
                "s_sub_u32 %[r], %[r], 1\n\t"
                "s_waitcnt lgkmcnt(0)\n\t"
                "v_readfirstlane_b32 %[so], %[x]\n\t"
                "s_max_u32 %[lim], %[lim], %[so]\n\t"
                "s_cmp_lt_u32 %[u], %[lim]\n\t"
                "s_cbranch_scc1 2b\n\t"
                "s_cmp_gt_i32 %[r], 0\n\t"
                "s_cbranch_scc1 8b\n\t"
                "s_branch 3b\n"
                "5:\n\t"
                "s_waitcnt vmcnt(0)\n\t"
                "s_cmp_eq_u32 %[c], 2\n\t"
                "s_cbranch_scc1 4b\n"
                "9:"
                : [x] "=&v"(x), [cap] "=&s"(cap), [c] "=&s"(c), [so] "=&s"(so), [r] "=&s"(r),
                  [t] "+s"(t), [u] "+s"(u), [lim] "+s"(lim)
                : [gtab] "v"(gtab), [ltab] "v"(ltab), [lima] "v"(lima),
                  [b0] "s"((u32)(size_t)stg), [b1] "s"((u32)((size_t)stg >> 32)), [va] "v"(va), [vb] "v"(vb),
                  [nsh] "n"(NSH), [nspin] "n"(ASPIN)
                : "memory", "scc", "s60", "s61");
            if (u >= NSH) break;
            lim = block(u);
            if (lim >= 1000) break;
        }
#pragma unroll 1
        for (; t < NBAR - 1; ++t) {
            if (u < NS && lim < 1000) {
                if (u >= lim) lim = max(lim, (int)rfl(*(volatile LAS u32*)(smem + LDS_CTL + C_LIM * 4)));
                if (u >= lim && u == t + 1) lim = block(u);
                if (lim == NS && u == t + 2) {
                    t += NS - u;
#pragma unroll 1
                    do {
                        issue(vr, stg, 4096, u, kh);
                        WAITV(10);
                        BAR();
                    } while (++u < NS);
                } else {
#pragma unroll 1
                    for (; u < min(min(lim, NS), t + 3) && lim < 1000; ++u) issue(vr, stg, 4096, u, kh);
                }
            }
            if (u == t + 3) WAITV(10); else WAITV(0);
            BAR();
        }
        STAMP(3);
        return;
    }

    const u32 cnt0 = __hip_atomic_load(cntp, __ATOMIC_RELAXED, __HIP_MEMORY_SCOPE_AGENT);
    if (wid == 0) STAMP(4);
    if (wid < 4) {
        if (kh) consume<6, 0>(P, smem, lds0, lane, wid, myws, cntp, tmo, cnt0, row0, 0, col0, 0, xcd);
        else    consume<5, 0>(P, smem, lds0, lane, wid, myws, cntp, tmo, cnt0, row0, 96, col0, 0, xcd);
    } else {
        if (kh) consume<5, 1>(P, smem, lds0, lane, wid, myws, cntp, tmo, cnt0, row0, 96, col0, 0, xcd);
        else    consume<6, 1>(P, smem, lds0, lane, wid, myws, cntp, tmo, cnt0, row0, 0, col0, tm == 2 ? 16 : 0, xcd);
    }
}

extern "C" void kernel_launch(int rank, int nranks, ncclComm_t comm,
                              hipStream_t stream,
                              void* const* d_in, const int* in_sizes, int n_in,
                              void* d_out, int out_size,
                              const int* in_dtypes, int out_dtype,
                              void* const* d_out_all,
                              void* const* d_ws_all, size_t ws_bytes) {
    if (nranks != NR || (unsigned)rank >= (unsigned)NR || ws_bytes < WS_NEED || n_in < 2 ||
        in_sizes[0] != 4096 * 512 || in_sizes[1] != 4096 * 4096 || out_size != 512 * 4096) {
        fprintf(stderr, "kernel_launch: unexpected problem shape\n");
        return;
    }
    HIP_CHECK(hipFuncSetAttribute((const void*)fused_a2a_gemm, hipFuncAttributeMaxDynamicSharedMemorySize, LDS_BYTES));
    Params P;
    P.x = (const char*)d_in[0];
    P.w = (const char*)d_in[1];
    P.out = (char*)d_out;
    for (int p = 0; p < NR; ++p) P.ws[p] = (char*)d_ws_all[p];
    P.rank = rank;
    P.pad = 0;
    hipLaunchKernelGGL(fused_a2a_gemm, dim3(NWG), dim3(NTHREADS), LDS_BYTES, stream, P);
    HIP_CHECK(hipGetLastError());
}
```

```cpp
#include <hip/hip_runtime.h>
#include <rccl/rccl.h>
#include <cstdio>

#define HIP_CHECK(expr) do { hipError_t _e = (expr); if (_e != hipSuccess) { \
    fprintf(stderr, "HIP error %s at %s:%d\n", hipGetErrorString(_e), __FILE__, __LINE__); \
    return; } } while (0)

#define LAS __attribute__((address_space(3)))
#define GAS __attribute__((address_space(1)))
typedef unsigned u32;
typedef unsigned long long u64;
typedef unsigned u32x4 __attribute__((ext_vector_type(4)));
typedef unsigned u32x2 __attribute__((ext_vector_type(2)));
typedef short s16x4 __attribute__((ext_vector_type(4)));
typedef short s16x8 __attribute__((ext_vector_type(8)));
typedef __bf16 bf16x8 __attribute__((ext_vector_type(8)));
typedef float f32x4 __attribute__((ext_vector_type(4)));
typedef float f32x16 __attribute__((ext_vector_type(16)));
typedef unsigned u32x16 __attribute__((ext_vector_type(16)));

constexpr int NR = 8;
constexpr int NWG = 256;
constexpr int NGEMM = 192;
constexpr int NTHREADS = 704;
constexpr int NG = 64;
#define NTAIL 6
#define XD 2
constexpr int NSH = (NG - NTAIL) / 2;
constexpr int NS = NSH + NTAIL;
static_assert(NTAIL % 2 == 0 && NTAIL >= 4 && XD >= 2 && XD <= 3, "");
constexpr size_t WS_CNT = 16384;
constexpr size_t WS_TMO = 0;
constexpr size_t WS_FLAG = 4096;
constexpr size_t WS_MAX = 512u << 10;
constexpr size_t WS_DIAG = 65536;
constexpr size_t WS_XCH = 9u << 20;
constexpr size_t WS_STAGE = 1u << 20;
constexpr size_t STAGE_BYTES = 4u << 20;
#define NBUF 2
static_assert(WS_STAGE + NBUF * STAGE_BYTES <= WS_XCH, "");
#define XSTR 9216
constexpr size_t WS_NEED = WS_XCH + (size_t)NGEMM * 4 * XSTR;
#define PUSH_LA 2
#define LD_LA 2
#define T0 4
#define PSPIN 5
#define ASPIN 2
#define KB 0
#define KBU 2
#define TW 26
#define GDELAY 0
#define S1DELAY 32
#define SPEC 640
#define OFL 1
#define OSUF " sc1"
#define ROTA 1
#define ROTB 1
#define PSPIN0 5
#define KMASK 63
constexpr int RING = 4;
constexpr int ASLOT = 22 * 1024;
constexpr int LDS_W = 0;
constexpr int LDS_A = RING * 16384;
constexpr int LDS_CTL = LDS_A + RING * ASLOT;
constexpr int LDS_SCHED = LDS_CTL + 256 + 256;
constexpr int LDS_BYTES = LDS_SCHED + 256;
static_assert(LDS_BYTES <= 160 * 1024 && LDS_BYTES > 80 * 1024, "one workgroup per CU");
constexpr int C_MAX = 0  , C_CALL = 32  , C_ST = 36  ;
#define PMARGIN 2
constexpr u32 BADMAX = 0xffffffffu;
constexpr u64 DEADLINE = 40000000ull;

struct Params {
    const char* x;
    const char* w;
    char* out;
    char* myws;
    int rank;
    int pad;
    const char* psrc[NR - 1];
    char* pws[NR - 1];
    char* ws[NR];
};

#define WAITV(n) asm volatile("s_waitcnt vmcnt(%0)" ::"n"(n) : "memory")
#define WAITL(n) asm volatile("s_waitcnt lgkmcnt(%0)" ::"n"(n) : "memory")
#define FENCE() asm volatile("" ::: "memory")
#define SB() __builtin_amdgcn_sched_barrier(0)
#define BAR() __builtin_amdgcn_s_barrier()

__device__ __forceinline__ u64 rt() { return __builtin_amdgcn_s_memrealtime(); }
__device__ __forceinline__ u32 rfl(u32 v) { return __builtin_amdgcn_readfirstlane(v); }
__device__ __forceinline__ u32 row16_max(u32 c) {
    c = max(c, (u32)__builtin_amdgcn_update_dpp((int)c, (int)c, 0x128, 0xf, 0xf, false));
    c = max(c, (u32)__builtin_amdgcn_update_dpp((int)c, (int)c, 0x124, 0xf, 0xf, false));
    c = max(c, (u32)__builtin_amdgcn_update_dpp((int)c, (int)c, 0x122, 0xf, 0xf, false));
    c = max(c, (u32)__builtin_amdgcn_update_dpp((int)c, (int)c, 0x121, 0xf, 0xf, false));
    return c;
}
__device__ __forceinline__ u32 wave_max(u32 v) {
    v = row16_max(v);
    return max(max((u32)__builtin_amdgcn_readlane(v, 0), (u32)__builtin_amdgcn_readlane(v, 16)),
               max((u32)__builtin_amdgcn_readlane(v, 32), (u32)__builtin_amdgcn_readlane(v, 48)));
}
__device__ __forceinline__ int row16_min(int c) {
    c = min(c, __builtin_amdgcn_update_dpp(c, c, 0x128, 0xf, 0xf, false));
    c = min(c, __builtin_amdgcn_update_dpp(c, c, 0x124, 0xf, 0xf, false));
    c = min(c, __builtin_amdgcn_update_dpp(c, c, 0x122, 0xf, 0xf, false));
    c = min(c, __builtin_amdgcn_update_dpp(c, c, 0x121, 0xf, 0xf, false));
    return c;
}
__device__ __forceinline__ u32 pk_bf16(float lo, float hi) {
    typedef __bf16 bf16x2 __attribute__((ext_vector_type(2)));
    const bf16x2 v = {(__bf16)lo, (__bf16)hi};
    return __builtin_bit_cast(u32, v);
}
__device__ __forceinline__ bool expired(GAS u32* tmo, u32 call, u64 t0, u32& spins) {
    if ((++spins & 63u) != 0) return false;
    if (rfl(__hip_atomic_load(tmo, __ATOMIC_RELAXED, __HIP_MEMORY_SCOPE_AGENT)) == call || rt() - t0 > DEADLINE) {
        __hip_atomic_store(tmo, call, __ATOMIC_RELAXED, __HIP_MEMORY_SCOPE_AGENT);
        return true;
    }
    return false;
}

#define STAMP(slot) do {} while (0)

struct Sched { int wl[8], ws[8]; };
constexpr Sched mk_sched() {
    Sched s{}; int posL[16] = {}, posS[16] = {}, n = 0;
    for (int j = 0; j < LD_LA; ++j) { n += 2; posL[j] = n; }
    for (int j = 0; j < 8 + PUSH_LA; ++j) {
        if (j >= PUSH_LA) s.ws[j - PUSH_LA] = n - posS[j - PUSH_LA];
        if (j < 8) { s.wl[j] = n - posL[j]; n += 2; posS[j] = n; if (j + LD_LA < 8) { n += 2; posL[j + LD_LA] = n; } }
    }
    return s;
}
constexpr Sched SCHED = mk_sched();

template <int J>
__device__ __forceinline__ void push_stage(u32x4 (&d)[8][2], u32 vs0, u32 vs1, const char* src, char* dst, char* flag,
                                           u32 call, int wid, u32 vd) {
    if constexpr (J >= PUSH_LA) {
        WAITV(SCHED.ws[J - PUSH_LA]);
        BAR();
        if (wid == 0) {
            const u32 val = call * 16 + (J - PUSH_LA) + 1, vz = 0;
            asm volatile("s_mov_b64 exec, 1\n\tglobal_store_dword %0, %1, %2 sc0 sc1\n\ts_mov_b64 exec, -1"
                         ::"v"(vz), "v"(val), "s"(flag) : "memory");
        }
    }
    if constexpr (J < 8) {
        if constexpr (J == 1 && S1DELAY > 0) __builtin_amdgcn_s_sleep(S1DELAY);
        WAITV(SCHED.wl[J]);
        char* p = dst + (size_t)J * 65536;
        asm volatile("global_store_dwordx4 %2, %0, %3 sc0 sc1\n\t"
                     "global_store_dwordx4 %2, %1, %3 offset:1024 sc0 sc1\n\ts_nop 1"
                     : "+v"(d[J][0]), "+v"(d[J][1]) : "v"(vd), "s"(p) : "memory");
        if constexpr (J + LD_LA < 8)
            asm volatile("global_load_dwordx4 %0, %2, %4 offset:%5\n\tglobal_load_dwordx4 %1, %3, %4 offset:%5"
                         : "=&v"(d[J + LD_LA][0]), "=&v"(d[J + LD_LA][1]) : "v"(vs0), "v"(vs1), "s"(src), "n"((J + LD_LA) * 128) : "memory");
    }
    if constexpr (J + 1 < 8 + PUSH_LA) push_stage<J + 1>(d, vs0, vs1, src, dst, flag, call, wid, vd);
}

__device__ __forceinline__ void warm_w(const char* w, u32 lds0, int ktab, int lane, int c, int wid, int xcd, int u0, int u1) {
    const u32 gtab = (u32)ktab << 19;
    const u32 vo = (c * 16 + wid * 4 + (lane >> 4)) * 8192 + (xcd >> 1) * 2048 + (lane & 15) * 128;
#pragma unroll 1
    for (int u = u0; u < min(u1, NSH); ++u)
        asm volatile("s_mov_b32 m0, %2\n\ts_nop 0\n\tglobal_load_lds_dword %0, %1\n\ts_waitcnt vmcnt(1)"
                     ::"v"(vo), "s"(w + (u32)__builtin_amdgcn_readlane(gtab, u)), "s"(lds0 + wid * 256) : "memory");
    const u32 vt = ((c * 2 + (wid & 1)) * 8 + (lane >> 3)) * 8192 +
                   (xcd >> 1) * 2048 + (((lane >> 1) & 3) * 2 + ((xcd & 1) ^ 1)) * 256 + (lane & 1) * 128;
#pragma unroll 1
    for (int u = max(u0, NSH) + (wid >> 1); u < u1; u += 2)
        asm volatile("s_mov_b32 m0, %2\n\ts_nop 0\n\tglobal_load_lds_dword %0, %1\n\ts_waitcnt vmcnt(1)"
                     ::"v"(vt), "s"(w + (u32)__builtin_amdgcn_readlane(gtab, u)), "s"(lds0 + wid * 256) : "memory");
    WAITV(0);
}

#define MORD 0
#define ML_MFMA                                                                                                     \
    ".if ml_n == 4\n"                                                                                               \
    "s_waitcnt lgkmcnt(6*(\\rd))\n"                                                                                 \
    ".endif\n"                                                                                                      \
    "v_mfma_f32_16x16x32_bf16 v[40+4*ml_n:43+4*ml_n], v[88+32*(\\ks)+4*(ml_n&1):91+32*(\\ks)+4*(ml_n&1)], "         \
    "v[96+32*(\\ks)+4*(ml_n>>1):99+32*(\\ks)+4*(ml_n>>1)], v[40+4*ml_n:43+4*ml_n]\n"
#define ML_MACROS                                                                                                   \
    ".macro RD k, ks, slot\n"                                                                                       \
    ".if (\\k) < 2\n"                                                                                               \
    "ds_read_b64_tr_b16 v[88+32*(\\ks)+2*(\\k):89+32*(\\ks)+2*(\\k)], %[pw0] offset:16384*(\\slot)+8192*(\\ks)+1024*(\\k)\n" \
    ".elseif (\\k) < 4\n"                                                                                           \
    "ds_read_b64_tr_b16 v[88+32*(\\ks)+2*(\\k):89+32*(\\ks)+2*(\\k)], %[pw1] offset:16384*(\\slot)+8192*(\\ks)+1024*((\\k)-2)\n" \
    ".elseif (\\ks) == 0 && (\\slot) < 2\n"                                                                         \
    "ds_read_b128 v[80+4*(\\k):83+4*(\\k)], %[pa00] offset:%c[aslot]*(\\slot)+2048*((\\k)-4)\n"                      \
    ".elseif (\\ks) == 0\n"                                                                                         \
    "ds_read_b128 v[80+4*(\\k):83+4*(\\k)], %[pa01] offset:%c[aslot]*((\\slot)-2)+2048*((\\k)-4)\n"                  \
    ".elseif (\\slot) < 2\n"                                                                                        \
    "ds_read_b128 v[112+4*(\\k):115+4*(\\k)], %[pa10] offset:%c[aslot]*(\\slot)+2048*((\\k)-4)\n"                    \
    ".else\n"                                                                                                       \
    "ds_read_b128 v[112+4*(\\k):115+4*(\\k)], %[pa11] offset:%c[aslot]*((\\slot)-2)+2048*((\\k)-4)\n"                \
    ".endif\n"                                                                                                      \
    ".endm\n"                                                                                                       \
    ".macro PH ks, slot, rd, cnt, xl=8\n"                                                                                 \
    "s_waitcnt lgkmcnt(%c[mt]-2)\n"                                                                                 \
    ".set ml_n, 0\n"                                                                                                \
    ".rept 2*%c[mt]\n"                                                                                              \
    ML_MFMA                                                                                                         \
    ".if (\\rd) && ml_n < 2\n"                                                                                      \
    "RD 2*ml_n, 1-(\\ks), \\slot\n"                                                                                 \
    "RD 2*ml_n+1, 1-(\\ks), \\slot\n"                                                                               \
    ".elseif (\\rd) && ml_n < %c[mt]+2\n"                                                                           \
    "RD ml_n+2, 1-(\\ks), \\slot\n"                                                                                 \
    ".elseif (\\cnt) && ml_n == %c[mt]+2\n"                                                                         \
    "s_sub_u32 %[cnt], %[cnt], 1\n"                                                                                 \
    ".elseif (\\cnt) && ml_n == %c[mt]+3\n"                                                                         \
    "s_cmp_eq_u32 %[cnt], 0\n"                                                                                      \
    ".endif\n"                                                                                                      \
    ".if (ml_n == 2 || ml_n == 5 || ml_n == 8) && (\\xl)+(ml_n-2)/3 < 8\n"                                          \
    "XL (\\xl)+(ml_n-2)/3\n"                                                                                       \
    ".endif\n"                                                                                                      \
    ".set ml_n, ml_n+1\n"                                                                                           \
    ".endr\n"                                                                                                       \
    ".endm\n"                                                                                                       \
    ".macro STEP s, cnt\n"                                                                                          \
    "PH 0, (\\s)&3, 1, 0\n"                                                                                         \
    "s_barrier\n"                                                                                                   \
    "PH 1, ((\\s)+1)&3, 1, \\cnt\n"                                                                                 \
    ".endm\n"
#define ML_LOOP                                                                                                     \
    ".set ml_n, 40\n.rept 8*%c[mt]\nv_mov_b32 v[ml_n], 0\n.set ml_n, ml_n+1\n.endr\n"                               \
    "s_mov_b32 %[cnt], (%c[nloop]+3)/4\n"                                                                           \
    "s_waitcnt lgkmcnt(0)\n"                                                                                        \
    "s_barrier\n"                                                                                                   \
    ".set ml_k, 0\n.rept %c[mt]+4\nRD ml_k, 0, 0\n.set ml_k, ml_k+1\n.endr\n"                                       \
    "1:\n"                                                                                                          \
    ".set ml_s, 0\n"                                                                                                \
    ".rept 4\n"                                                                                                     \
    ".if ml_s == ((%c[nloop]-1)&3)\n"                                                                               \
    "STEP ml_s, 1\n"                                                                                                \
    "s_cbranch_scc1 2f\n"                                                                                           \
    ".else\n"                                                                                                       \
    "STEP ml_s, 0\n"                                                                                                \
    ".endif\n"                                                                                                      \
    ".set ml_s, ml_s+1\n"                                                                                           \
    ".endr\n"                                                                                                       \
    "s_branch 1b\n"                                                                                                 \
    "2:\n"
#define ML_END "s_nop 7\ns_nop 1\n.purgem RD\n.purgem PH\n.purgem STEP"
#define ML_FRAGS                                                                                                    \
    "v88", "v89", "v90", "v91", "v92", "v93", "v94", "v95", "v96", "v97", "v98", "v99", "v100", "v101", "v102", "v103",       \
    "v104", "v105", "v106", "v107", "v108", "v109", "v110", "v111", "v112", "v113", "v114", "v115", "v116", "v117", "v118", "v119", \
    "v120", "v121", "v122", "v123", "v124", "v125", "v126", "v127", "v128", "v129", "v130", "v131", "v132", "v133", "v134", "v135", \
    "v136", "v137", "v138", "v139", "v140", "v141", "v142", "v143", "v144", "v145", "v146", "v147", "v148", "v149", "v150", "v151"

__device__ __forceinline__ void finish(const Params& P, char* smem, int lane, int wid, char* myws, GAS u32* tmo, u32 call, u32 mb,
                                       int row0, int col0) {
    const int r0 = wid * 8 + (lane >> 3), rlo = row0 == 336 ? 16 : 0;
    char* op = P.out + (size_t)(row0 + r0) * 4096 + col0 + (lane & 7) * 16;
    GAS u64* g0 = (GAS u64*)(myws + WS_MAX) + (wid * 96 + lane) * 16;
    GAS u64* g1 = lane < 32 ? g0 + 64 * 16 : (GAS u64*)(myws + WS_TMO);
    volatile LAS u32x2* st = (volatile LAS u32x2*)(smem + LDS_CTL + C_ST * 4);
    if (lane == 0) st[wid] = u32x2{0u, 0u};
    WAITL(0);
    BAR(); FENCE();
    f32x4 y[12];
    {
        LAS char* tp = (LAS char*)smem + r0 * 528 + (lane & 7) * 64;
#pragma unroll
        for (int n = 0; n < 12; ++n) y[n] = *(LAS f32x4*)(tp + (n >> 2) * (64 * 528) + (n & 3) * 16);
    }
    bool ok0 = false, ok1 = false, mine = false;
    u32 gm = mb, written = 0xfffffffeu, spins = 0;
    const u64 t0 = rt();
    for (;;) {
        if (!mine) {
            u64 v0 = 0, v1 = 0;
            if (!ok0) v0 = __hip_atomic_load(g0, __ATOMIC_RELAXED, __HIP_MEMORY_SCOPE_SYSTEM);
            if (!ok1) v1 = __hip_atomic_load(g1, __ATOMIC_RELAXED, __HIP_MEMORY_SCOPE_SYSTEM);
            if ((u32)(v0 >> 32) == call) { ok0 = true; gm = max(gm, (u32)v0); }
            if (lane < 32) { if ((u32)(v1 >> 32) == call) { ok1 = true; gm = max(gm, (u32)v1); } }
            else if ((u32)v1 == call) gm = BADMAX;
            u32 cnt = __popcll(__ballot(ok0)) + __popcll(__ballot(ok1));
            if (cnt < 96 && expired(tmo, call, t0, spins)) { gm = BADMAX; cnt = 96; }
            mine = cnt == 96;
            gm = wave_max(gm);
            if (lane == 0) st[wid] = u32x2{gm, cnt};
        }
        const u32x2 sv = st[lane & 7];
        const u32 cur = rfl(row16_max(sv[0]));
        u32 tot = sv[1];
        tot += __builtin_amdgcn_update_dpp(0, (int)tot, 0x124, 0xf, 0xf, false);
        tot += __builtin_amdgcn_update_dpp(0, (int)tot, 0x122, 0xf, 0xf, false);
        tot += __builtin_amdgcn_update_dpp(0, (int)tot, 0x121, 0xf, 0xf, false);
        tot = rfl(tot);
        if (tot >= SPEC && cur != written) {
            written = cur;
            const float amax = __uint_as_float(cur);
            const float inv = amax > 0.f ? 127.0f / amax : 0.f;
#pragma unroll
            for (int k = 0; k < 3; ++k) {
                u32x4 o;
#pragma unroll
                for (int g = 0; g < 4; ++g) {
                    u32 wd = 0;
#pragma unroll
                    for (int r = 0; r < 4; ++r)
                        wd = __builtin_amdgcn_cvt_pk_u8_f32(y[k * 4 + g][r] * inv, r, wd);
                    o[g] = cur == BADMAX ? 0x80808080u : wd;
                }
                const int r = r0 + 64 * k;
                if (r >= rlo && r < 176)
                    asm volatile("global_store_dwordx4 %0, %1, off" OSUF "\n\ts_nop 1" ::"v"(op + (size_t)k * 64 * 4096), "v"(o) : "memory");
            }
        }
        if (tot == 768) break;
    }
    if (wid == 0) STAMP(7);
}

template <int MT, int own>
__device__ __forceinline__ void consume(const Params& P, char* smem, u32 lds0, int lane, int wid, char* myws, GAS u32* cntp, GAS u32* tmo,
                                        int row0, int rh, int col0) {
    const int wn = wid & 3;
    u32 pa[2], pw[2];
#pragma unroll
    for (int ks = 0; ks < 2; ++ks)
        pa[ks] = lds0 + LDS_A + (rh + (lane & 15)) * 128 + (((4 * ks + (lane >> 4)) ^ ((lane >> 1) & 7)) * 16);
    {
        const u32 g = lane >> 4, q = (lane >> 2) & 3, p = lane & 3;
#pragma unroll
        for (int nb = 0; nb < 2; ++nb) {
            const u32 ch = (wn * 4 + nb * 2 + (p >> 1)) ^ ((q << 2) | ((g & 1) << 1));
            pw[nb] = lds0 + LDS_W + (8 * g + q) * 256 + ch * 16 + (p & 1) * 8;
        }
    }
    static_assert(RING == 4 && LDS_W == 0, "");
    f32x16 a0, a1, a2;
    u32 lc;
    constexpr int NV = 8 * MT, NQ = (NV + 5) / 6;
    u32 vl = lane * 16;
    char* xp = myws + WS_XCH + (size_t)(((blockIdx.x - 64) >> 1) * 8 + wid) * XSTR;
    char* xp2 = xp + 4096;
    asm volatile("" : "+s"(xp), "+s"(xp2), "+v"(vl));
#define ACC(n) ((n) < 16 ? a0[(n) & 15] : (n) < 32 ? a1[(n) & 15] : a2[(n) & 15])
    if constexpr (!own) {
        asm volatile(ML_MACROS ML_LOOP
                     "PH 0, %c[nloop]&3, 1, 0\n"
                     "PH 1, 0, 0, 0\n"
                     ML_END
                     : "=&{v[40:55]}"(a0), "=&{v[56:71]}"(a1), "=&{v[72:87]}"(a2), [cnt] "=&s"(lc)
                     : [pw0] "v"(pw[0]), [pw1] "v"(pw[1]), [pa00] "v"(pa[0]), [pa01] "v"(pa[0] + 2 * ASLOT),
                       [pa10] "v"(pa[1]), [pa11] "v"(pa[1] + 2 * ASLOT), [mt] "n"(MT), [aslot] "n"(ASLOT), [nloop] "n"(NSH - 1)
                     : "memory", "scc", ML_FRAGS);
        const u32x2 ct = *(volatile LAS u32x2*)(smem + LDS_CTL + C_CALL * 4);
        const u32 call = rfl(ct[0]), tag = rfl(ct[1]);
#pragma unroll
        for (int q = 0; q < NQ; ++q) {
            u32x4 o;
#pragma unroll
            for (int hf = 0; hf < 2; ++hf) {
                float f[3];
#pragma unroll
                for (int e = 0; e < 3; ++e) { const int n = q * 6 + hf * 3 + e; f[e] = n < NV ? ACC(n) : 0.f; }
                o[2 * hf] = pk_bf16(f[0], f[1]);
                o[2 * hf + 1] = (pk_bf16(f[2], 0.f) & 0xffffu) | tag;
            }
            asm volatile("global_store_dwordx4 %0, %1, %2 offset:%3 sc1\n\ts_nop 1"
                         ::"v"(vl), "v"(o), "s"(q < 4 ? xp : xp2), "n"((q & 3) * 1024) : "memory");
        }
        if (wid == 0) STAMP(5);
        if (wid == 0 && lane == 0) __hip_atomic_store(cntp, call, __ATOMIC_RELAXED, __HIP_MEMORY_SCOPE_AGENT);
        return;
    }
    if (wid == 0) {
        char* pw8 = P.ws[0];
#pragma unroll
        for (int p = 1; p < NR; ++p) pw8 = ((lane & 7) == p) ? P.ws[p] : pw8;
        if (lane < NR)
            ((volatile LAS u64*)(smem + LDS_CTL + 64))[lane] = (u64)(pw8 + WS_MAX + ((P.rank & 7) * 96 + ((blockIdx.x - 64) >> 1)) * 128);
    }
    u32x16 x0, x1;
    u32 call, tag;
    asm volatile(ML_MACROS ML_LOOP
                 ".macro XL k\n"
                 ".if (\\k) < 4\n"
                 "global_load_dwordx4 v[8+4*(\\k):11+4*(\\k)], %[vl], %[xp] offset:1024*(\\k) sc1\n"
                 ".elseif (\\k) < 7\n"
                 "global_load_dwordx4 v[8+4*(\\k):11+4*(\\k)], %[vl], %[xp2] offset:1024*((\\k)-4) sc1\n"
                 ".else\n"
                 "global_load_dwordx4 v[36:39], %[vl], %[xp2] offset:%c[xlast] sc1\n"
                 ".endif\n"
                 ".endm\n"
                 ".set ml_s, %c[nloop]\n.set ml_x, 0\n"
                 ".rept %c[xd]-1\n"
                 "PH 0, ml_s&3, 1, 0, ml_x\ns_barrier\nPH 1, (ml_s+1)&3, 1, 0, ml_x+3\n"
                 ".set ml_s, ml_s+1\n.set ml_x, ml_x+6\n.endr\n"
                 "PH 0, ml_s&3, 1, 0, ml_x\n"
                 "ds_read_b64 v[88:89], %[cta]\n"
                 "PH 1, 0, 0, 0\n"
                 "v_readfirstlane_b32 %[call], v88\n"
                 "v_readfirstlane_b32 %[tag], v89\n"
                 ".purgem XL\n"
                 "s_waitcnt vmcnt(0)\n"
                 ML_END
                 : "=&{v[40:55]}"(a0), "=&{v[56:71]}"(a1), "=&{v[72:87]}"(a2), "=&{v[8:23]}"(x0), "=&{v[24:39]}"(x1), [cnt] "=&s"(lc), [call] "=&s"(call), [tag] "=&s"(tag)
                 : [pw0] "v"(pw[0]), [pw1] "v"(pw[1]), [pa00] "v"(pa[0]), [pa01] "v"(pa[0] + 2 * ASLOT),
                   [pa10] "v"(pa[1]), [pa11] "v"(pa[1] + 2 * ASLOT), [mt] "n"(MT), [aslot] "n"(ASLOT), [nloop] "n"(NS - XD),
                   [vl] "v"(vl), [xp2] "s"(xp2), [xp] "s"(xp), [xlast] "n"((NQ - 5) * 1024), [xd] "n"(XD), [cta] "v"(lds0 + LDS_CTL + C_CALL * 4)
                 : "memory", "scc", ML_FRAGS);
    u32x4 in[8];
#pragma unroll
    for (int q = 0; q < 8; ++q)
#pragma unroll
        for (int e = 0; e < 4; ++e) in[q][e] = q < 4 ? x0[4 * q + e] : x1[4 * (q - 4) + e];
    f32x4 acc[2 * MT];
#pragma unroll
    for (int n = 0; n < NV; ++n) acc[n >> 2][n & 3] = ACC(n);
#define XLOADS(TAIL)                                                                                                             \
    asm volatile("global_load_dwordx4 %0, %8, %10 sc1\n\tglobal_load_dwordx4 %1, %8, %10 offset:1024 sc1\n\t"                    \
                 "global_load_dwordx4 %2, %8, %10 offset:2048 sc1\n\tglobal_load_dwordx4 %3, %8, %10 offset:3072 sc1\n\t"        \
                 "global_load_dwordx4 %4, %8, %9 sc1\n\tglobal_load_dwordx4 %5, %8, %9 offset:1024 sc1\n\t"                    \
                 "global_load_dwordx4 %6, %8, %9 offset:2048 sc1\n\tglobal_load_dwordx4 %7, %8, %9 offset:%11 sc1" TAIL        \
                 : "=&v"(in[0]), "=&v"(in[1]), "=&v"(in[2]), "=&v"(in[3]), "=&v"(in[4]), "=&v"(in[5]), "=&v"(in[6]), "=&v"(in[7]) \
                 : "v"(vl), "s"(xp2), "s"(xp), "n"((NQ - 5) * 1024) : "memory")
    if (wid == 7) STAMP(20);
    {
        u32 bad = 0;
#pragma unroll
        for (int q = 0; q < 8; ++q) bad |= (in[q][1] ^ tag) | (in[q][3] ^ tag);
        if (!__all(bad < 0x10000u)) {
            const u64 t0 = rt();
            u32 spins = 0;
            for (;;) {
                XLOADS("\n\ts_waitcnt vmcnt(0)");
                bad = 0;
#pragma unroll
                for (int q = 0; q < 8; ++q) bad |= (in[q][1] ^ tag) | (in[q][3] ^ tag);
                if (__all(bad < 0x10000u)) break;
                if (expired(tmo, call, t0, spins)) break;
            }
        }
    }
    if (wid == 0) STAMP(21);
    volatile LAS u32* ctlp = (volatile LAS u32*)(smem + LDS_CTL);
    float m = 0.f;
    {
#pragma unroll
        for (int n = 0; n < NV; ++n) {
            const int q = n / 6, hf = (n % 6) / 3, e = n % 3;
            const u32 w = in[q][2 * hf + (e >> 1)];
            acc[n >> 2][n & 3] += __uint_as_float(e == 1 ? w & 0xffff0000u : w << 16);
            m = fmaxf(m, acc[n >> 2][n & 3]);
        }
    }
    const u32 wm = wave_max(__float_as_uint(m));
    if (lane == 0) ctlp[C_MAX + wid] = wm;
    WAITL(0);
    BAR();
    const u32x4 mv = *(volatile LAS u32x4*)(smem + LDS_CTL + C_MAX * 4), mw = *(volatile LAS u32x4*)(smem + LDS_CTL + C_MAX * 4 + 16);
    const u32 mb = max(max(max(mv[0], mv[1]), max(mv[2], mv[3])), max(max(mw[0], mw[1]), max(mw[2], mw[3])));
    if (wid == 0) {
        GAS u64* mxp = (GAS u64*)((volatile LAS u64*)(smem + LDS_CTL + 64))[lane & 7];
        if (lane < NR) __hip_atomic_store(mxp, ((u64)call << 32) | mb, __ATOMIC_RELAXED, __HIP_MEMORY_SCOPE_SYSTEM);
        if (lane == 0) __hip_atomic_store(cntp, call, __ATOMIC_RELAXED, __HIP_MEMORY_SCOPE_AGENT);
        STAMP(18);
    }
    {
        LAS char* tp = (LAS char*)smem + (rh + (lane & 15)) * 528 + wn * 128 + (lane >> 4) * 16;
#pragma unroll
        for (int n = 0; n < 2 * MT; ++n) *(LAS f32x4*)(tp + (n >> 1) * (16 * 528) + (n & 1) * 64) = acc[n];
    }
    finish(P, smem, lane, wid, myws, tmo, call, mb, row0, col0);
}

__global__ void __launch_bounds__(NTHREADS) fused_a2a_gemm(Params P) {
    extern __shared__ __attribute__((aligned(1024))) char smem[];
    const u32 lds0 = (u32)(size_t)(LAS char*)smem;
    int lane = threadIdx.x & 63;
    const int wid = rfl(threadIdx.x >> 6);
    const int q = ((blockIdx.x & 31) * 37) >> 8, pi = (blockIdx.x & 31) - 7 * q;
    const char* psrc = P.psrc[pi];
    char* pws = P.pws[pi];
    char* myws_ = P.myws;
    const char *px = P.x, *pw_ = P.w;
    int rank_ = P.rank;
    asm volatile("" : "+s"(psrc), "+s"(pws), "+s"(myws_), "+s"(rank_), "+s"(px), "+s"(pw_));
    const int rank = rank_ & 7;
    const int xcd = blockIdx.x & 7, idx = 31 - (blockIdx.x >> 3);
    char* const myws = myws_;
    GAS u32* const cntp = (GAS u32*)(myws + WS_CNT) + blockIdx.x * 32;
    GAS u32* const tmo = (GAS u32*)(myws + WS_TMO);

    if (blockIdx.x < 28 && wid < 8) {
        const int row0 = q * 128 + wid * 16;
        const char* src = psrc + row0 * 1024;
        const u32 vs0 = (lane >> 3) * 1024 + (lane & 7) * 16, vs1 = vs0 + 8192;
        u32x4 d[8][2];
        u32 cnt;
#pragma unroll
        for (int j = 0; j < LD_LA; ++j) {
            asm volatile("global_load_dwordx4 %0, %2, %4 offset:%5\n\tglobal_load_dwordx4 %1, %3, %4 offset:%5"
                         : "=&v"(d[j][0]), "=&v"(d[j][1]) : "v"(vs0), "v"(vs1), "s"(src), "n"(j * 128) : "memory");
            if (j == 0) asm volatile("global_load_dword %0, %1, off sc1" : "=&v"(cnt) : "v"(cntp) : "memory");
        }
        char* dst = pws + WS_STAGE + (size_t)(rank * 8) * 65536 + (size_t)row0 * 128;
        char* flag = pws + WS_FLAG + (rank * 4 + q) * 128;
        asm volatile("s_waitcnt vmcnt(%3)" : "+v"(cnt), "+s"(dst), "+s"(flag) : "n"(2 * LD_LA - 2) : "memory");
        const u32 call = rfl(cnt) + 1;
        dst += (size_t)(call % NBUF) * STAGE_BYTES;
        push_stage<0>(d, vs0, vs1, src, dst, flag, call, wid, lane * 16);
        if (threadIdx.x == 0) __hip_atomic_store(cntp, call, __ATOMIC_RELAXED, __HIP_MEMORY_SCOPE_AGENT);
        if (wid == 0) STAMP(1);
    }

    const int kh = xcd & 1;
    const int tm = idx % 3, tn = idx / 3;
    const int row0 = tm == 2 ? 336 : tm * 176;
    const int col0 = (xcd >> 1) * 1024 + tn * 128;
    const int owner = (tn ^ kh) & 1;
    const int ns = owner ? NS : NSH;

    if (idx < 24 && wid < 8) {
        if (wid < 4 && tm < 2) {
            if (owner) consume<6, 1>(P, smem, lds0, lane, wid, myws, cntp, tmo, row0, 0, col0);
            else       consume<6, 0>(P, smem, lds0, lane, wid, myws, cntp, tmo, row0, 0, col0);
        } else {
            const int rh = wid < 4 ? 16 : 96;
            if (owner) consume<5, 1>(P, smem, lds0, lane, wid, myws, cntp, tmo, row0, rh, col0);
            else       consume<5, 0>(P, smem, lds0, lane, wid, myws, cntp, tmo, row0, rh, col0);
        }
        return;
    }
    asm volatile("" : "+v"(lane));

    const int rot = idx < 24 ? (ROTA * tm + ROTB * tn) & 3 : 0;
    int ktab, kfull;
    {
        const int sl = lane < 4 ? (lane + rot) & 3 : lane;
        const int g = min(sl < NSH ? 2 * sl + kh : sl + NSH, NG - 1);
        const int tt = g - 8, j = (tt * 37) >> 8, i = tt - 7 * j;
        kfull = g < 8 ? rank * 8 + g : ((rank + 1 + i) & 7) * 8 + j;
        ktab = kfull & KMASK;
    }

    if (idx >= 28) {
        const int g = blockIdx.x;
        if (wid < 4) warm_w(pw_, lds0, ktab, lane, g >> 3, wid, xcd, TW, NS);
        if (wid == 0) STAMP(23);
        return;
    }

    if (idx >= 24) {
        if (wid < 4) warm_w(pw_, lds0, ktab, lane, idx - 24, wid, xcd, T0, TW - KB * 3 / 2);
        if (wid == 0) STAMP(19);
        return;
    }

    const u32 scha = lds0 + LDS_SCHED + lane * 4;
#define LOADER_LOOP(ISSUE, WAITN)                                                                                     \
    "1:\n\t"                                                                                                          \
    "s_mov_b32 %[r], %[nspin]\n\t"                                                                                    \
    "s_add_u32 %[cap], %[t], 3\n\t"                                                                                   \
    "s_min_u32 %[cap], %[cap], %[ns]\n"                                                                               \
    "6:\n\t"                                                                                                          \
    "s_cmp_ge_u32 %[u], %[cap]\n\t"                                                                                   \
    "s_cbranch_scc1 3f\n\t"                                                                                           \
    "s_cmp_ge_u32 %[u], %[lim]\n\t"                                                                                   \
    "s_cbranch_scc1 8f\n"                                                                                             \
    "2:\n\t"                                                                                                          \
    "v_readlane_b32 %[so], %[x], %[u]\n\t"                                                                            \
    "v_readlane_b32 %[c], %[ltab], %[u]\n\t"                                                                          \
    "s_lshl_b32 %[so], %[so], %[sh]\n\t"                                                                              \
    "s_mov_b32 m0, %[c]\n\t"                                                                                          \
    "s_add_u32 s60, %[b0], %[so]\n\t"                                                                                 \
    "s_addc_u32 s61, %[b1], 0\n\t"                                                                                    \
    ISSUE                                                                                                             \
    "s_add_u32 %[u], %[u], 1\n\t"                                                                                     \
    "s_mov_b32 %[r], 1\n\t"                                                                                           \
    "s_cmp_lt_u32 %[u], %[cap]\n\t"                                                                                   \
    "s_cbranch_scc1 6b\n"                                                                                             \
    "3:\n\t"                                                                                                          \
    "s_sub_u32 %[c], %[u], %[t]\n\t"                                                                                  \
    "s_cmp_lt_u32 %[c], 3\n\t"                                                                                        \
    "s_cbranch_scc1 5f\n\t"                                                                                           \
    WAITN                                                                                                             \
    "4:\n\t"                                                                                                          \
    "s_add_u32 %[t], %[t], 1\n\t"                                                                                     \
    "s_cmp_lt_u32 %[t], %[tend]\n\t"                                                                                  \
    "s_barrier\n\t"                                                                                                   \
    "s_cbranch_scc1 1b\n\t"                                                                                           \
    "s_branch 9f\n"                                                                                                   \
    "8:\n\t"              \
    "ds_read_b32 %[x], %[scha]\n\t"                                                                                   \
    "s_sub_u32 %[r], %[r], 1\n\t"                                                                                     \
    "s_sub_u32 %[c], %[u], %[t]\n\t"                                                                                  \
    "s_waitcnt lgkmcnt(0)\n\t"                                                                                        \
    "v_cmp_eq_u32 vcc, 0, %[x]\n\t"                                                                                   \
    "s_ff1_i32_b64 %[lim], vcc\n\t"                                                                                   \
    "s_cmp_lt_u32 %[u], %[lim]\n\t"                                                                                   \
    "s_cbranch_scc1 2b\n\t"                                                                                           \
    "s_cmp_lt_u32 %[c], 2\n\t"                                                                                        \
    "s_cbranch_scc1 8b\n\t"                                                                                           \
    "s_cmp_gt_i32 %[r], 0\n\t"                                                                                        \
    "s_cbranch_scc1 8b\n\t"                                                                                           \
    "s_getreg_b32 %[c], hwreg(HW_REG_IB_STS)\n\t"                         \
    "s_and_b32 %[c], %[c], 0xc0000f\n\t"                                                                              \
    "s_cbranch_scc1 8b\n\t"                                                                                           \
    "s_branch 3b\n"                                                                                                   \
    "5:\n\t"                                                                                                          \
    "s_waitcnt vmcnt(0)\n\t"                                                                                          \
    "s_branch 4b\n"                                                                                                   \
    "9:"

    if (wid == 8) {
        __builtin_amdgcn_s_setprio(2);
        u32 v[8];
#pragma unroll
        for (int i = 0; i < 8; ++i) {
            const u32 q = lane >> 4, ch = (lane & 15) ^ ((q << 2) | (((i >> 1) & 1) << 1));
            v[i] = (4 * i + q) * 8192 + col0 * 2 + ch * 16 - (i & 3) * 1024;
        }
        const u32 ltab = lds0 + LDS_W + (lane & (RING - 1)) * 16384;
        const char* wb = pw_ - (1u << 19);
        u32 x = ktab + 1, cap, c, so, r;
        int t = -1, u = 0, lim = 4;
#define W_ISSUE                                                                                                       \
    "global_load_lds_dwordx4 %[v0], s[60:61] offset:0\n\tglobal_load_lds_dwordx4 %[v1], s[60:61] offset:1024\n\t"     \
    "global_load_lds_dwordx4 %[v2], s[60:61] offset:2048\n\tglobal_load_lds_dwordx4 %[v3], s[60:61] offset:3072\n\t"  \
    "s_add_u32 m0, m0, 0x1000\n\ts_nop 0\n\t"                                                                         \
    "global_load_lds_dwordx4 %[v4], s[60:61] offset:0\n\tglobal_load_lds_dwordx4 %[v5], s[60:61] offset:1024\n\t"     \
    "global_load_lds_dwordx4 %[v6], s[60:61] offset:2048\n\tglobal_load_lds_dwordx4 %[v7], s[60:61] offset:3072\n\t"  \
    "s_add_u32 m0, m0, 0x1000\n\ts_add_u32 s60, s60, 0x40000\n\ts_addc_u32 s61, s61, 0\n\t"                           \
    "global_load_lds_dwordx4 %[v0], s[60:61] offset:0\n\tglobal_load_lds_dwordx4 %[v1], s[60:61] offset:1024\n\t"     \
    "global_load_lds_dwordx4 %[v2], s[60:61] offset:2048\n\tglobal_load_lds_dwordx4 %[v3], s[60:61] offset:3072\n\t"  \
    "s_add_u32 m0, m0, 0x1000\n\ts_nop 0\n\t"                                                                         \
    "global_load_lds_dwordx4 %[v4], s[60:61] offset:0\n\tglobal_load_lds_dwordx4 %[v5], s[60:61] offset:1024\n\t"     \
    "global_load_lds_dwordx4 %[v6], s[60:61] offset:2048\n\tglobal_load_lds_dwordx4 %[v7], s[60:61] offset:3072\n\t"
        auto run = [&](int tend) {
            asm volatile(LOADER_LOOP(W_ISSUE, "s_waitcnt vmcnt(16)\n")
                         : [x] "+v"(x), [cap] "=&s"(cap), [c] "=&s"(c), [so] "=&s"(so), [r] "=&s"(r), [t] "+s"(t), [u] "+s"(u), [lim] "+s"(lim)
                         : [ltab] "v"(ltab), [scha] "v"(scha), [b0] "s"((u32)(size_t)wb), [b1] "s"((u32)((size_t)wb >> 32)),
                           [v0] "v"(v[0]), [v1] "v"(v[1]), [v2] "v"(v[2]), [v3] "v"(v[3]), [v4] "v"(v[4]), [v5] "v"(v[5]), [v6] "v"(v[6]), [v7] "v"(v[7]),
                           [sh] "n"(19), [ns] "s"(ns), [nspin] "n"(ASPIN), [tend] "s"(tend)
                         : "memory", "scc", "vcc", "s60", "s61");
        };
        if (GDELAY) __builtin_amdgcn_s_sleep(GDELAY);
        run(ns - 1);
        STAMP(2);
        return;
    }

    if (wid == 10) {
        static_assert(NS - 4 <= 32, "");
        const u32 call = rfl(__hip_atomic_load(cntp, __ATOMIC_RELAXED, __HIP_MEMORY_SCOPE_AGENT)) + 1;
        *(volatile LAS u32*)(size_t)scha = lane < 4 ? ktab + 1 : 0;
        *(volatile LAS u32x2*)(smem + LDS_CTL + C_CALL * 4) = u32x2{call, (call % 65535u + 1u) << 16};
        const int ck = __shfl(kfull, min(4 + (lane & 31), NS - 1));
        const u32 vf = ((ck >> 3) * 4 + ((lane & 32) ? (row0 + 175) >> 7 : row0 >> 7)) * 128;
        const u32 thr = call * 16 + (ck & 7) + 1;
        const u32 ent = (ck & KMASK) + 1;
        constexpr u32 ALL = NS - 4 == 32 ? 0xffffffffu : (1u << ((NS - 4) & 31)) - 1, FULL = (1u << (NSH - 4)) - 1;
        u32 tvo[KB > 0 ? KB : 1];
#pragma unroll
        for (int k = 0; k < KB; ++k) {
            const int n = idx + 24 * k;
            tvo[k] = ((u32)__builtin_amdgcn_readlane(ktab, TW - 1 - (n >> 4)) << 19) + ((n & 15) * 4 + (lane >> 4)) * 8192 + (xcd >> 1) * 2048 + (lane & 15) * 128;
        }
        WAITL(0);
#pragma unroll 1
        for (int e = 0; e < KBU; ++e) BAR();
#pragma unroll
        for (int k = 0; k < KB; ++k)
            asm volatile("s_mov_b32 m0, %2\n\ts_nop 0\n\tglobal_load_lds_dword %0, %1" ::"v"(tvo[k]), "s"(pw_), "s"(lds0 + LDS_CTL + 256) : "memory");
        u32 fl, tmp, c, r, nw, t = KBU, pend = 0, lim = 4, done = ~(owner ? ALL : FULL), spins = 0, resume = 0;
        u64 t0 = 0;
        for (;;) {
            asm volatile(
                "s_cmp_lg_u32 %[res], 0\n\t"
                "s_cbranch_scc1 7f\n"
                "1:\n\t"
                "s_barrier\n"
                "7:\n\t"
                "s_mov_b32 %[res], 64\n\t"
                "s_cmp_lt_u32 %[t], 3\n\t"
                "s_cselect_b32 %[r], %[nspin0], %[nspin]\n"
                "6:\n\t"
                "s_cmp_eq_u32 %[pend], 0\n\t"
                "s_cbranch_scc1 2f\n\t"
                "s_getreg_b32 %[c], hwreg(HW_REG_IB_STS, 0, 4)\n\t"
                "s_cmp_lg_u32 %[c], 0\n\t"
                "s_cbranch_scc1 4f\n"
                "5:\n\t"
                "s_waitcnt vmcnt(0)\n\t"
                "v_cmp_le_u32 vcc, %[thr], %[fl]\n\t"
                "s_mov_b32 %[pend], 0\n\t"
                "s_and_b32 %[nw], vcc_lo, vcc_hi\n\t"
                "s_andn2_b32 %[nw], %[nw], %[done]\n\t"
                "s_cbranch_scc0 2f\n\t"
                "v_mbcnt_lo_u32_b32 %[tmp], %[nw], 0\n\t"
                "s_or_b32 %[done], %[done], %[nw]\n\t"
                "v_add_u32 %[tmp], %[lim], %[tmp]\n\t"
                "s_bcnt1_i32_b32 %[c], %[nw]\n\t"
                "v_lshl_add_u32 %[tmp], %[tmp], 2, %[sch0]\n\t"
                "s_add_u32 %[lim], %[lim], %[c]\n\t"
                "s_mov_b32 exec_lo, %[nw]\n\t"
                "s_mov_b32 exec_hi, 0\n\t"
                "ds_write_b32 %[tmp], %[ent]\n\t"
                "s_mov_b64 exec, -1\n\t"
                "s_sub_u32 %[r], %[r], 1\n\t"
                "s_cmp_ge_u32 %[lim], %[ns]\n\t"
                "s_cbranch_scc1 9f\n"
                "2:\n\t"
                "s_add_u32 %[c], %[t], %[reach]\n\t"
                "s_cmp_gt_u32 %[lim], %[c]\n\t"
                "s_cbranch_scc1 3f\n\t"
                "global_load_dword %[fl], %[vf], %[flagp] sc0 sc1\n\t"
                "s_mov_b32 %[pend], 1\n"
                "4:\n\t"
                "s_sub_u32 %[r], %[r], 1\n\t"
                "s_cmp_gt_i32 %[r], 0\n\t"
                "s_cbranch_scc1 6b\n\t"
                "s_add_u32 %[c], %[t], 1\n\t"
                "s_cmp_gt_u32 %[lim], %[c]\n\t"
                "s_cbranch_scc1 3f\n\t"
                "s_sub_u32 %[res], %[res], 1\n\t"
                "s_cmp_gt_u32 %[res], 1\n\t"
                "s_cbranch_scc1 5b\n\t"
                "s_branch 9f\n"
                "3:\n\t"
                "s_add_u32 %[t], %[t], 1\n\t"
                "s_branch 1b\n"
                "9:\n\t"
                "s_waitcnt vmcnt(0) lgkmcnt(0)\n\t"
                "s_mov_b32 %[pend], 0"
                : [fl] "=&v"(fl), [tmp] "=&v"(tmp), [c] "=&s"(c), [r] "=&s"(r), [nw] "=&s"(nw), [t] "+s"(t), [pend] "+s"(pend),
                  [lim] "+s"(lim), [done] "+s"(done), [res] "+s"(resume)
                : [thr] "v"(thr), [ent] "v"(ent), [vf] "v"(vf), [flagp] "s"(myws + WS_FLAG), [sch0] "s"(lds0 + LDS_SCHED),
                  [reach] "n"(2 + PMARGIN), [ns] "s"(ns), [nspin] "n"(PSPIN), [nspin0] "n"(PSPIN0)
                : "memory", "vcc", "scc");
            if ((int)lim >= ns) break;
            if (t0 == 0) t0 = rt();
            spins = 63;
            if (expired(tmo, call, t0, spins)) {
                if (lane >= (int)lim && lane < ns) *(volatile LAS u32*)(size_t)scha = ((rank * 8) & KMASK) + 1;
                WAITL(0);
                break;
            }
        }
        return;
    }

    if (wid == 9) {
        __builtin_amdgcn_s_setprio(2);
        u32 vo[4];
#pragma unroll
        for (int i = 0; i < 4; ++i) {
            const int row = 8 * i + (lane >> 3), ch = (lane & 7) ^ ((row >> 1) & 7);
            vo[i] = (rank * 512 + row0 + row) * 1024 + ch * 16 - i * 1024;
        }
        u32 va = (row0 + (lane >> 3)) * 128 + ((lane & 7) ^ (lane >> 4)) * 16, vb = (row0 + (lane >> 3)) * 128 + ((lane & 7) ^ (4 + (lane >> 4))) * 16;
        u32 ltab = lds0 + LDS_A + (lane & (RING - 1)) * ASLOT;
        asm volatile("" : "+v"(va), "+v"(vb), "+v"(ltab));
        if (GDELAY) __builtin_amdgcn_s_sleep(GDELAY);
#pragma unroll 1
        for (int u = 0; u < 4; ++u) {
            const char* base = px + (2 * ((u + rot) & 3) + kh) * 128;
            asm volatile("s_mov_b32 m0, %6\n\ts_mov_b32 s60, %4\n\ts_mov_b32 s61, %5\n\t"
                         ".rept 5\n\t"
                         "global_load_lds_dwordx4 %0, s[60:61] offset:0\n\tglobal_load_lds_dwordx4 %1, s[60:61] offset:1024\n\t"
                         "global_load_lds_dwordx4 %2, s[60:61] offset:2048\n\tglobal_load_lds_dwordx4 %3, s[60:61] offset:3072\n\t"
                         "s_add_u32 m0, m0, 0x1000\n\ts_add_u32 s60, s60, 0x8000\n\ts_addc_u32 s61, s61, 0\n\t"
                         ".endr\n\t"
                         "global_load_lds_dwordx4 %0, s[60:61] offset:0\n\tglobal_load_lds_dwordx4 %1, s[60:61] offset:1024"
                         ::"v"(vo[0]), "v"(vo[1]), "v"(vo[2]), "v"(vo[3]), "s"((u32)(size_t)base), "s"((u32)((size_t)base >> 32)),
                           "s"(lds0 + LDS_A + u * ASLOT) : "memory", "scc", "s60", "s61");
            if (u > 0) { WAITV(22); BAR(); }
        }
        FENCE();
        const u32 call = rfl(*(volatile LAS u32*)(smem + LDS_CTL + C_CALL * 4));
        const char* stg = myws + WS_STAGE + (size_t)(call % NBUF) * STAGE_BYTES - 65536;
        u32 x = 0, cap, c, so, r;
        int t = 2, u = 4, lim = 4;
#define A_G4                                                                                                                          \
    "global_load_lds_dwordx4 %[va], s[60:61] offset:0 sc0 sc1\n\tglobal_load_lds_dwordx4 %[vb], s[60:61] offset:1024 sc0 sc1\n\t"     \
    "global_load_lds_dwordx4 %[va], s[60:61] offset:2048 sc0 sc1\n\tglobal_load_lds_dwordx4 %[vb], s[60:61] offset:3072 sc0 sc1\n\t"  \
    "s_add_u32 m0, m0, 0x1000\n\ts_add_u32 s60, s60, 0x1000\n\ts_addc_u32 s61, s61, 0\n\t"
#define A_ISSUE A_G4 A_G4 A_G4 A_G4 A_G4                                                                                              \
    "global_load_lds_dwordx4 %[va], s[60:61] offset:0 sc0 sc1\n\tglobal_load_lds_dwordx4 %[vb], s[60:61] offset:1024 sc0 sc1\n\t"
        asm volatile(LOADER_LOOP(A_ISSUE, "s_waitcnt vmcnt(22)\n")
                     : [x] "+v"(x), [cap] "=&s"(cap), [c] "=&s"(c), [so] "=&s"(so), [r] "=&s"(r), [t] "+s"(t), [u] "+s"(u), [lim] "+s"(lim)
                     : [ltab] "v"(ltab), [scha] "v"(scha), [b0] "s"((u32)(size_t)stg), [b1] "s"((u32)((size_t)stg >> 32)),
                       [va] "v"(va), [vb] "v"(vb), [sh] "n"(16), [ns] "s"(ns), [nspin] "n"(ASPIN), [tend] "s"(ns - 1)
                     : "memory", "scc", "vcc", "s60", "s61");
        STAMP(3);
        return;
    }

}

extern "C" void kernel_launch(int rank, int nranks, ncclComm_t comm,
                              hipStream_t stream,
                              void* const* d_in, const int* in_sizes, int n_in,
                              void* d_out, int out_size,
                              const int* in_dtypes, int out_dtype,
                              void* const* d_out_all,
                              void* const* d_ws_all, size_t ws_bytes) {
    if (nranks != NR || (unsigned)rank >= (unsigned)NR || ws_bytes < WS_NEED || n_in < 2 ||
        in_sizes[0] != 4096 * 512 || in_sizes[1] != 4096 * 4096 || out_size != 512 * 4096) {
        fprintf(stderr, "kernel_launch: unexpected problem shape\n");
        return;
    }
    HIP_CHECK(hipFuncSetAttribute((const void*)fused_a2a_gemm, hipFuncAttributeMaxDynamicSharedMemorySize, LDS_BYTES));
    Params P;
    P.x = (const char*)d_in[0];
    P.w = (const char*)d_in[1];
    P.out = (char*)d_out;
    for (int p = 0; p < NR; ++p) P.ws[p] = (char*)d_ws_all[p];
    P.myws = P.ws[rank];
    for (int i = 0; i < NR - 1; ++i) {
        const int p = (rank + 1 + i) % NR;
        P.psrc[i] = P.x + (size_t)p * 512 * 1024;
        P.pws[i] = P.ws[p];
    }
    P.rank = rank;
    P.pad = 0;
    hipLaunchKernelGGL(fused_a2a_gemm, dim3(NWG), dim3(NTHREADS), LDS_BYTES, stream, P);
    HIP_CHECK(hipGetLastError());
}
```

```cpp
#include <hip/hip_runtime.h>
#include <rccl/rccl.h>
#include <cstdio>

#define HIP_CHECK(expr) do { hipError_t _e = (expr); if (_e != hipSuccess) { \
    fprintf(stderr, "HIP error %s at %s:%d\n", hipGetErrorString(_e), __FILE__, __LINE__); \
    return; } } while (0)

#define LAS __attribute__((address_space(3)))
#define GAS __attribute__((address_space(1)))
typedef unsigned u32;
typedef unsigned long long u64;
typedef unsigned u32x4 __attribute__((ext_vector_type(4)));
typedef unsigned u32x2 __attribute__((ext_vector_type(2)));
typedef short s16x4 __attribute__((ext_vector_type(4)));
typedef short s16x8 __attribute__((ext_vector_type(8)));
typedef __bf16 bf16x8 __attribute__((ext_vector_type(8)));
typedef float f32x4 __attribute__((ext_vector_type(4)));
typedef float f32x16 __attribute__((ext_vector_type(16)));
typedef unsigned u32x16 __attribute__((ext_vector_type(16)));

constexpr int NR = 8;
constexpr int NWG = 256;
constexpr int NGEMM = 192;
constexpr int NTHREADS = 704;
constexpr int NG = 64;
#define NTAIL 6
#define XD 2
constexpr int NSH = (NG - NTAIL) / 2;
constexpr int NS = NSH + NTAIL;
static_assert(NTAIL % 2 == 0 && NTAIL >= 4 && XD >= 2 && XD <= 3, "");
constexpr size_t WS_CNT = 16384;
constexpr size_t WS_TMO = 0;
constexpr size_t WS_FLAG = 4096;
constexpr size_t WS_MAX = 512u << 10;
constexpr size_t WS_SUM = 8192;
constexpr size_t WS_DIAG = 65536;
constexpr size_t WS_XCH = 9u << 20;
constexpr size_t WS_STAGE = 1u << 20;
constexpr size_t STAGE_BYTES = 4u << 20;
#define NBUF 2
static_assert(WS_STAGE + NBUF * STAGE_BYTES <= WS_XCH, "");
#define XSTR 9216
constexpr size_t WS_NEED = WS_XCH + (size_t)NGEMM * 4 * XSTR;
#define PUSH_LA 2
#define LD_LA 2
#define T0 2
#define PSPIN 5
#define ASPIN 2
#define KBU 3
#define TW 28
#define S1DELAY 32
#define SPEC 640
#define OFL 1
#define OSUF " sc1"
#define ROTA 1
#define ROTB 1
#define PSPIN0 5
#define EARLYP 1
#define SUMM 1
#define SQUIT 24
#define SSLEEP 4
#define B0EARLY 0
#define KMASK 63
constexpr int RING = 4;
constexpr int ASLOT = 22 * 1024;
constexpr int LDS_W = 0;
constexpr int LDS_A = RING * 16384;
constexpr int LDS_CTL = LDS_A + RING * ASLOT;
constexpr int LDS_SCHED = LDS_CTL + 256 + 256;
constexpr int LDS_BYTES = LDS_SCHED + 256;
static_assert(LDS_BYTES <= 160 * 1024 && LDS_BYTES > 80 * 1024, "one workgroup per CU");
constexpr int C_ST = 0  , C_OWN = 8, C_CNTA = 9, C_CNTB = 10, C_CALL = 32  ;
#define PMARGIN 2
constexpr u32 BADMAX = 0xffffffffu;
constexpr u64 DEADLINE = 40000000ull;

struct Params {
    const char* x;
    const char* w;
    char* out;
    char* myws;
    int rank;
    int pad;
    const char* psrc[NR - 1];
    char* pws[NR - 1];
    char* ws[NR];
};

#define WAITV(n) asm volatile("s_waitcnt vmcnt(%0)" ::"n"(n) : "memory")
#define WAITL(n) asm volatile("s_waitcnt lgkmcnt(%0)" ::"n"(n) : "memory")
#define FENCE() asm volatile("" ::: "memory")
#define SB() __builtin_amdgcn_sched_barrier(0)
#define BAR() __builtin_amdgcn_s_barrier()

__device__ __forceinline__ u64 rt() { return __builtin_amdgcn_s_memrealtime(); }
__device__ __forceinline__ u32 rfl(u32 v) { return __builtin_amdgcn_readfirstlane(v); }
__device__ __forceinline__ u32 row16_max(u32 c) {
    c = max(c, (u32)__builtin_amdgcn_update_dpp((int)c, (int)c, 0x128, 0xf, 0xf, false));
    c = max(c, (u32)__builtin_amdgcn_update_dpp((int)c, (int)c, 0x124, 0xf, 0xf, false));
    c = max(c, (u32)__builtin_amdgcn_update_dpp((int)c, (int)c, 0x122, 0xf, 0xf, false));
    c = max(c, (u32)__builtin_amdgcn_update_dpp((int)c, (int)c, 0x121, 0xf, 0xf, false));
    return c;
}
__device__ __forceinline__ u32 wave_max(u32 v) {
    v = row16_max(v);
    return max(max((u32)__builtin_amdgcn_readlane(v, 0), (u32)__builtin_amdgcn_readlane(v, 16)),
               max((u32)__builtin_amdgcn_readlane(v, 32), (u32)__builtin_amdgcn_readlane(v, 48)));
}
__device__ __forceinline__ int row16_min(int c) {
    c = min(c, __builtin_amdgcn_update_dpp(c, c, 0x128, 0xf, 0xf, false));
    c = min(c, __builtin_amdgcn_update_dpp(c, c, 0x124, 0xf, 0xf, false));
    c = min(c, __builtin_amdgcn_update_dpp(c, c, 0x122, 0xf, 0xf, false));
    c = min(c, __builtin_amdgcn_update_dpp(c, c, 0x121, 0xf, 0xf, false));
    return c;
}
__device__ __forceinline__ u32 pk_bf16(float lo, float hi) {
    typedef __bf16 bf16x2 __attribute__((ext_vector_type(2)));
    const bf16x2 v = {(__bf16)lo, (__bf16)hi};
    return __builtin_bit_cast(u32, v);
}
__device__ __forceinline__ bool expired(GAS u32* tmo, u32 call, u64 t0, u32& spins) {
    if ((++spins & 63u) != 0) return false;
    if (rfl(__hip_atomic_load(tmo, __ATOMIC_RELAXED, __HIP_MEMORY_SCOPE_AGENT)) == call || rt() - t0 > DEADLINE) {
        __hip_atomic_store(tmo, call, __ATOMIC_RELAXED, __HIP_MEMORY_SCOPE_AGENT);
        return true;
    }
    return false;
}

#define STAMP(slot) do {} while (0)

struct Sched { int wl[8], ws[8]; };
constexpr Sched mk_sched() {
    Sched s{}; int posL[16] = {}, posS[16] = {}, n = 0;
    for (int j = 0; j < LD_LA; ++j) { n += 2; posL[j] = n; }
    for (int j = 0; j < 8 + PUSH_LA; ++j) {
        if (j >= PUSH_LA) s.ws[j - PUSH_LA] = n - posS[j - PUSH_LA];
        if (j < 8) { s.wl[j] = n - posL[j]; n += 2; posS[j] = n; if (j + LD_LA < 8) { n += 2; posL[j + LD_LA] = n; } }
    }
    return s;
}
constexpr Sched SCHED = mk_sched();

template <int J>
__device__ __forceinline__ void push_stage(u32x4 (&d)[8][2], u32 vs0, u32 vs1, const char* src, char* dst, char* flag,
                                           u32 call, int wid, u32 vd) {
    if constexpr (J >= PUSH_LA) {
        WAITV(SCHED.ws[J - PUSH_LA]);
        BAR();
        if (wid == 0) {
            const u32 val = call * 16 + (J - PUSH_LA) + 1, vz = 0;
            asm volatile("s_mov_b64 exec, 1\n\tglobal_store_dword %0, %1, %2 sc0 sc1\n\ts_mov_b64 exec, -1"
                         ::"v"(vz), "v"(val), "s"(flag) : "memory");
        }
    }
    if constexpr (J < 8) {
        if constexpr (J == 1 && S1DELAY > 0) __builtin_amdgcn_s_sleep(S1DELAY);
        WAITV(SCHED.wl[J]);
        char* p = dst + (size_t)J * 65536;
        asm volatile("global_store_dwordx4 %2, %0, %3 sc0 sc1\n\t"
                     "global_store_dwordx4 %2, %1, %3 offset:1024 sc0 sc1\n\ts_nop 1"
                     : "+v"(d[J][0]), "+v"(d[J][1]) : "v"(vd), "s"(p) : "memory");
        if constexpr (J + LD_LA < 8)
            asm volatile("global_load_dwordx4 %0, %2, %4 offset:%5\n\tglobal_load_dwordx4 %1, %3, %4 offset:%5"
                         : "=&v"(d[J + LD_LA][0]), "=&v"(d[J + LD_LA][1]) : "v"(vs0), "v"(vs1), "s"(src), "n"((J + LD_LA) * 128) : "memory");
    }
    if constexpr (J + 1 < 8 + PUSH_LA) push_stage<J + 1>(d, vs0, vs1, src, dst, flag, call, wid, vd);
}

__device__ __forceinline__ void warm_w(const char* w, u32 lds, int ktab, int lane, int id, int nw, int xcd, int u0, int u1) {
    const u32 gtab = (u32)ktab << 19;
    const u32 vo = (lane >> 4) * 8192 + (xcd >> 1) * 2048 + (lane & 15) * 128;
    const int n1 = (min(u1, NSH) - u0) * 16;
    int n = id;
#pragma unroll 1
    for (; n < n1; n += nw)
        asm volatile("s_mov_b32 m0, %2\n\ts_nop 0\n\tglobal_load_lds_dword %0, %1\n\ts_waitcnt vmcnt(1)"
                     ::"v"(vo), "s"(w + ((u32)__builtin_amdgcn_readlane(gtab, u0 + (n >> 4)) + (n & 15) * 4 * 8192)), "s"(lds) : "memory");
    const u32 vt = (lane >> 3) * 8192 + (xcd >> 1) * 2048 + (((lane >> 1) & 3) * 2 + ((xcd & 1) ^ 1)) * 256 + (lane & 1) * 128;
#pragma unroll 1
    for (n -= n1; n < (u1 - NSH) * 8; n += nw)
        asm volatile("s_mov_b32 m0, %2\n\ts_nop 0\n\tglobal_load_lds_dword %0, %1\n\ts_waitcnt vmcnt(1)"
                     ::"v"(vt), "s"(w + ((u32)__builtin_amdgcn_readlane(gtab, NSH + (n >> 3)) + (n & 7) * 8 * 8192)), "s"(lds) : "memory");
    WAITV(0);
}

#define MORD 0
#define ML_MFMA                                                                                                     \
    ".if ml_n == 4\n"                                                                                               \
    "s_waitcnt lgkmcnt(6*(\\rd))\n"                                                                                 \
    ".endif\n"                                                                                                      \
    "v_mfma_f32_16x16x32_bf16 v[40+4*ml_n:43+4*ml_n], v[88+32*(\\ks)+4*(ml_n&1):91+32*(\\ks)+4*(ml_n&1)], "         \
    "v[96+32*(\\ks)+4*(ml_n>>1):99+32*(\\ks)+4*(ml_n>>1)], v[40+4*ml_n:43+4*ml_n]\n"
#define ML_MACROS                                                                                                   \
    ".macro RD k, ks, slot\n"                                                                                       \
    ".if (\\k) < 2\n"                                                                                               \
    "ds_read_b64_tr_b16 v[88+32*(\\ks)+2*(\\k):89+32*(\\ks)+2*(\\k)], %[pw0] offset:16384*(\\slot)+8192*(\\ks)+1024*(\\k)\n" \
    ".elseif (\\k) < 4\n"                                                                                           \
    "ds_read_b64_tr_b16 v[88+32*(\\ks)+2*(\\k):89+32*(\\ks)+2*(\\k)], %[pw1] offset:16384*(\\slot)+8192*(\\ks)+1024*((\\k)-2)\n" \
    ".elseif (\\ks) == 0 && (\\slot) < 2\n"                                                                         \
    "ds_read_b128 v[80+4*(\\k):83+4*(\\k)], %[pa00] offset:%c[aslot]*(\\slot)+2048*((\\k)-4)\n"                      \
    ".elseif (\\ks) == 0\n"                                                                                         \
    "ds_read_b128 v[80+4*(\\k):83+4*(\\k)], %[pa01] offset:%c[aslot]*((\\slot)-2)+2048*((\\k)-4)\n"                  \
    ".elseif (\\slot) < 2\n"                                                                                        \
    "ds_read_b128 v[112+4*(\\k):115+4*(\\k)], %[pa10] offset:%c[aslot]*(\\slot)+2048*((\\k)-4)\n"                    \
    ".else\n"                                                                                                       \
    "ds_read_b128 v[112+4*(\\k):115+4*(\\k)], %[pa11] offset:%c[aslot]*((\\slot)-2)+2048*((\\k)-4)\n"                \
    ".endif\n"                                                                                                      \
    ".endm\n"                                                                                                       \
    ".macro PH ks, slot, rd, cnt, xl=8\n"                                                                                 \
    "s_waitcnt lgkmcnt(%c[mt]-2)\n"                                                                                 \
    ".set ml_n, 0\n"                                                                                                \
    ".rept 2*%c[mt]\n"                                                                                              \
    ML_MFMA                                                                                                         \
    ".if (\\rd) && ml_n < 2\n"                                                                                      \
    "RD 2*ml_n, 1-(\\ks), \\slot\n"                                                                                 \
    "RD 2*ml_n+1, 1-(\\ks), \\slot\n"                                                                               \
    ".elseif (\\rd) && ml_n < %c[mt]+2\n"                                                                           \
    "RD ml_n+2, 1-(\\ks), \\slot\n"                                                                                 \
    ".elseif (\\cnt) && ml_n == %c[mt]+2\n"                                                                         \
    "s_sub_u32 %[cnt], %[cnt], 1\n"                                                                                 \
    ".elseif (\\cnt) && ml_n == %c[mt]+3\n"                                                                         \
    "s_cmp_eq_u32 %[cnt], 0\n"                                                                                      \
    ".endif\n"                                                                                                      \
    ".if (ml_n == 2 || ml_n == 5 || ml_n == 8) && (\\xl)+(ml_n-2)/3 < 8\n"                                          \
    "XL (\\xl)+(ml_n-2)/3\n"                                                                                       \
    ".endif\n"                                                                                                      \
    ".set ml_n, ml_n+1\n"                                                                                           \
    ".endr\n"                                                                                                       \
    ".endm\n"                                                                                                       \
    ".macro STEP s, cnt\n"                                                                                          \
    "PH 0, (\\s)&3, 1, 0\n"                                                                                         \
    "s_barrier\n"                                                                                                   \
    "PH 1, ((\\s)+1)&3, 1, \\cnt\n"                                                                                 \
    ".endm\n"
#define ML_LOOP                                                                                                     \
    ".set ml_n, 40\n.rept 8*%c[mt]\nv_mov_b32 v[ml_n], 0\n.set ml_n, ml_n+1\n.endr\n"                               \
    "s_mov_b32 %[cnt], (%c[nloop]+3)/4\n"                                                                           \
    "s_waitcnt lgkmcnt(0)\n"                                                                                        \
    "s_barrier\n"                                                                                                   \
    ".set ml_k, 0\n.rept %c[mt]+4\nRD ml_k, 0, 0\n.set ml_k, ml_k+1\n.endr\n"                                       \
    "1:\n"                                                                                                          \
    ".set ml_s, 0\n"                                                                                                \
    ".rept 4\n"                                                                                                     \
    ".if ml_s == ((%c[nloop]-1)&3)\n"                                                                               \
    "STEP ml_s, 1\n"                                                                                                \
    "s_cbranch_scc1 2f\n"                                                                                           \
    ".else\n"                                                                                                       \
    "STEP ml_s, 0\n"                                                                                                \
    ".endif\n"                                                                                                      \
    ".set ml_s, ml_s+1\n"                                                                                           \
    ".endr\n"                                                                                                       \
    "s_branch 1b\n"                                                                                                 \
    "2:\n"
#define ML_END "s_nop 7\ns_nop 1\n.purgem RD\n.purgem PH\n.purgem STEP"
#define ML_FRAGS                                                                                                    \
    "v88", "v89", "v90", "v91", "v92", "v93", "v94", "v95", "v96", "v97", "v98", "v99", "v100", "v101", "v102", "v103",       \
    "v104", "v105", "v106", "v107", "v108", "v109", "v110", "v111", "v112", "v113", "v114", "v115", "v116", "v117", "v118", "v119", \
    "v120", "v121", "v122", "v123", "v124", "v125", "v126", "v127", "v128", "v129", "v130", "v131", "v132", "v133", "v134", "v135", \
    "v136", "v137", "v138", "v139", "v140", "v141", "v142", "v143", "v144", "v145", "v146", "v147", "v148", "v149", "v150", "v151"

__device__ __forceinline__ int tile_base(int r) {
    static_assert(((NS - 1) & (RING - 1)) == 2 && 48 * 528 <= 2 * 16384 && 3 * 16384 + 112 * 528 <= LDS_A + 2 * ASLOT, "");
    return r < 48 ? 0 : (r < 160 ? 3 * 16384 - 48 * 528 : LDS_A + 3 * ASLOT - 160 * 528);
}

struct RankPoll {
    u64 p0 = ~0ull, p1 = 0xffffffffull;
    u32x2 v0 = {0, 0}, v1 = {0, 0};
};
__device__ __forceinline__ u32 rank_poll(RankPoll& r, const char* g, u32 o0, u32 o1, u32 call, u32 mxa) {
    u32 left, c;
    asm volatile("s_mov_b64 exec, %[p0]\n\tglobal_load_dwordx2 %[v0], %[o0], %[g] sc0 sc1\n\t"
                 "s_mov_b64 exec, %[p1]\n\ts_bitset1_b32 exec_hi, 31\n\tglobal_load_dwordx2 %[v1], %[o1], %[g] sc0 sc1\n\t"
                 "s_mov_b64 exec, -1\n\t"
                 "s_waitcnt vmcnt(0)"
                 : [v0] "+v"(r.v0), [v1] "+v"(r.v1) : [p0] "s"(r.p0), [p1] "s"(r.p1), [o0] "v"(o0), [o1] "v"(o1), [g] "s"(g) : "memory");
    const u64 q0 = __ballot(r.v0[1] != call), q1 = __ballot(r.v1[1] != call) & 0xffffffffull;
    asm volatile("s_andn2_b64 exec, %[p0], %[q0]\n\tds_max_u32 %[mxa], %[a]\n\t"
                 "s_andn2_b64 exec, %[p1], %[q1]\n\tds_max_u32 %[mxa], %[b]\n\t"
                 "s_mov_b64 exec, -1"
                 :: [p0] "s"(r.p0), [p1] "s"(r.p1), [q0] "s"(q0), [q1] "s"(q1), [mxa] "v"(mxa), [a] "v"(r.v0[0]), [b] "v"(r.v1[0]) : "memory", "scc");
    r.p0 = q0; r.p1 = q1;
    left = __popcll(q0) + __popcll(q1);
    (void)c;
    return left;
}

__device__ __forceinline__ void summarise(char* smem, u32 lds0, int lane, int w, int rank, char* myws, GAS u32* tmo, u32 call) {
    volatile LAS u32* mx = (volatile LAS u32*)(smem + 4096) + w;
    volatile LAS u32* quit = (volatile LAS u32*)(smem + 4096) + 8;
    *mx = 0;
    if (w == 0) *quit = 0;
    WAITL(0);
    BAR();
    RankPoll r;
    const u32 o0 = WS_MAX + (w * 96 + lane) * 128, o1 = lane < 32 ? o0 + 8192 : (u32)WS_TMO;
    const u64 t0 = rt();
    u32 spins = 0;
    for (;;) {
        const u32 left = rank_poll(r, myws, o0, o1, call, lds0 + 4096 + w * 4);
        if (w == rank && left < SQUIT) { *quit = 1; return; }
        if (rfl(*quit)) return;
        if (left == 0) break;
        if (expired(tmo, call, t0, spins)) return;
        __builtin_amdgcn_s_sleep(SSLEEP);
    }
    const u32 m = *mx;
    if (lane == 0) __hip_atomic_store((GAS u64*)(myws + WS_SUM) + w * 16, ((u64)call << 32) | m, __ATOMIC_RELAXED, __HIP_MEMORY_SCOPE_AGENT);
}

__device__ __forceinline__ void watch_summaries(char* smem, u32 lds0, int lane, char* myws) {
    const u32 call = rfl(*(volatile LAS u32*)(smem + LDS_CTL + C_CALL * 4));
    GAS u64* g = (GAS u64*)(myws + WS_SUM) + (lane & 7) * 16;
    do {
        if (lane < 8) {
            const u64 v = __hip_atomic_load(g, __ATOMIC_RELAXED, __HIP_MEMORY_SCOPE_AGENT);
            if ((u32)(v >> 32) == call)
                asm volatile("ds_max_u32 %0, %1\n\tds_max_u32 %2, %3"
                             ::"v"(lds0 + LDS_CTL + C_OWN * 4), "v"((u32)v), "v"(lds0 + LDS_CTL + (C_ST + lane) * 4), "v"(96u) : "memory");
        }
        __builtin_amdgcn_s_sleep(2);
    } while (rfl(*(volatile LAS u32*)(smem + LDS_CTL + C_CNTA * 4)) < 8);
}

__device__ __forceinline__ void finish(const Params& P, char* smem, u32 lds0, int lane, int wid, char* myws, GAS u32* tmo, u32 call,
                                       int row0, int col0) {
    const int r0 = wid * 8 + (lane >> 3), rlo = row0 == 336 ? 16 : 0;
    char* op = P.out + (size_t)(row0 + r0) * 4096 + col0 + (lane & 7) * 16;
    asm volatile("" : "+v"(op));
    volatile LAS u32* ctlp = (volatile LAS u32*)(smem + LDS_CTL);
    const u32 o0 = WS_MAX + (wid * 96 + lane) * 128, o1 = lane < 32 ? o0 + 8192 : (u32)WS_TMO;
    RankPoll rp;
    while (rfl(ctlp[C_CNTB]) < 8) {}
    FENCE();
    f32x4 y[12];
#pragma unroll
    for (int k = 0; k < 3; ++k) {
        LAS char* tp = (LAS char*)smem + tile_base(wid * 8 + 64 * k) + (r0 + 64 * k) * 528 + (lane & 7) * 64;
#pragma unroll
        for (int e = 0; e < 4; ++e) y[k * 4 + e] = *(LAS f32x4*)(tp + e * 16);
    }
    u32 written = 0xfffffffeu, spins = 0;
    const u64 t0 = rt();
    for (;;) {
        const u32x4 sa = *(volatile LAS u32x4*)(smem + LDS_CTL + C_ST * 4), sb = *(volatile LAS u32x4*)(smem + LDS_CTL + C_ST * 4 + 16);
        const u32 cur = rfl(ctlp[C_OWN]), mine = rfl(ctlp[C_ST + wid]);
        const u32 tot = rfl(sa[0] + sa[1] + sa[2] + sa[3] + sb[0] + sb[1] + sb[2] + sb[3]);
        if (tot >= SPEC && cur != written) {
            written = cur;
            const float amax = __uint_as_float(cur);
            const float inv = amax > 0.f ? 127.0f / amax : 0.f;
#pragma unroll
            for (int k = 0; k < 3; ++k) {
                u32x4 o;
#pragma unroll
                for (int g = 0; g < 4; ++g) {
                    u32 wd = 0;
#pragma unroll
                    for (int r = 0; r < 4; ++r)
                        wd = __builtin_amdgcn_cvt_pk_u8_f32(y[k * 4 + g][r] * inv, r, wd);
                    o[g] = cur == BADMAX ? 0x80808080u : wd;
                }
                const int r = r0 + 64 * k;
                if (r >= rlo && r < 176)
                    asm volatile("global_store_dwordx4 %0, %1, off" OSUF "\n\ts_nop 1" ::"v"(op + (size_t)k * 64 * 4096), "v"(o) : "memory");
            }
        }
        if (tot == 768) break;
        if (mine < 96) {
            u32 left = rank_poll(rp, myws, o0, o1, call, lds0 + LDS_CTL + C_OWN * 4);
            if ((u32)__builtin_amdgcn_readlane(rp.v1[0], 63) == call || (left && expired(tmo, call, t0, spins))) {
                ctlp[C_OWN] = BADMAX;
                left = 0;
            }
            asm volatile("s_mov_b64 exec, 1\n\tds_max_u32 %0, %1\n\ts_mov_b64 exec, -1"
                         ::"v"(lds0 + LDS_CTL + (C_ST + wid) * 4), "v"(96 - left) : "memory");
        }
    }
    if (wid == 0) STAMP(7);
}

template <int MT, int own>
__device__ __forceinline__ void consume(const Params& P, char* smem, u32 lds0, int lane, int wid, char* myws, GAS u32* cntp, GAS u32* tmo,
                                        int row0, int rh, int col0) {
    const int wn = wid & 3;
    u32 pa[2], pw[2];
#pragma unroll
    for (int ks = 0; ks < 2; ++ks)
        pa[ks] = lds0 + LDS_A + (rh + (lane & 15)) * 128 + (((4 * ks + (lane >> 4)) ^ ((lane >> 1) & 7)) * 16);
    {
        const u32 g = lane >> 4, q = (lane >> 2) & 3, p = lane & 3;
#pragma unroll
        for (int nb = 0; nb < 2; ++nb) {
            const u32 ch = (wn * 4 + nb * 2 + (p >> 1)) ^ ((q << 2) | ((g & 1) << 1));
            pw[nb] = lds0 + LDS_W + (8 * g + q) * 256 + ch * 16 + (p & 1) * 8;
        }
    }
    static_assert(RING == 4 && LDS_W == 0, "");
    f32x16 a0, a1, a2;
    u32 lc;
    constexpr int NV = 8 * MT, NQ = (NV + 5) / 6;
    u32 vl = lane * 16;
    char* xp = myws + WS_XCH + (size_t)(((blockIdx.x - 64) >> 1) * 8 + wid) * XSTR;
    char* xp2 = xp + 4096;
    asm volatile("" : "+s"(xp), "+s"(xp2), "+v"(vl));
#define ACC(n) ((n) < 16 ? a0[(n) & 15] : (n) < 32 ? a1[(n) & 15] : a2[(n) & 15])
    if constexpr (!own) {
        asm volatile(ML_MACROS ML_LOOP
                     "PH 0, %c[nloop]&3, 1, 0\n"
                     "PH 1, 0, 0, 0\n"
                     ML_END
                     : "=&{v[40:55]}"(a0), "=&{v[56:71]}"(a1), "=&{v[72:87]}"(a2), [cnt] "=&s"(lc)
                     : [pw0] "v"(pw[0]), [pw1] "v"(pw[1]), [pa00] "v"(pa[0]), [pa01] "v"(pa[0] + 2 * ASLOT),
                       [pa10] "v"(pa[1]), [pa11] "v"(pa[1] + 2 * ASLOT), [mt] "n"(MT), [aslot] "n"(ASLOT), [nloop] "n"(NSH - 1)
                     : "memory", "scc", ML_FRAGS);
        const u32x2 ct = *(volatile LAS u32x2*)(smem + LDS_CTL + C_CALL * 4);
        const u32 call = rfl(ct[0]), tag = rfl(ct[1]);
#pragma unroll
        for (int q = 0; q < NQ; ++q) {
            u32x4 o;
#pragma unroll
            for (int hf = 0; hf < 2; ++hf) {
                float f[3];
#pragma unroll
                for (int e = 0; e < 3; ++e) { const int n = q * 6 + hf * 3 + e; f[e] = n < NV ? ACC(n) : 0.f; }
                o[2 * hf] = pk_bf16(f[0], f[1]);
                o[2 * hf + 1] = (pk_bf16(f[2], 0.f) & 0xffffu) | tag;
            }
            asm volatile("global_store_dwordx4 %0, %1, %2 offset:%3 sc1\n\ts_nop 1"
                         ::"v"(vl), "v"(o), "s"(q < 4 ? xp : xp2), "n"((q & 3) * 1024) : "memory");
        }
        if (wid == 0) STAMP(5);
        if (wid == 0 && lane == 0) __hip_atomic_store(cntp, call, __ATOMIC_RELAXED, __HIP_MEMORY_SCOPE_AGENT);
        return;
    }
    if (wid == 0) {
        char* pw8 = P.ws[0];
#pragma unroll
        for (int p = 1; p < NR; ++p) pw8 = ((lane & 7) == p) ? P.ws[p] : pw8;
        if (lane < NR)
            ((volatile LAS u64*)(smem + LDS_CTL + 64))[lane] = (u64)(pw8 + WS_MAX + ((P.rank & 7) * 96 + ((blockIdx.x - 64) >> 1)) * 128);
    }
    u32x16 x0, x1;
    u32 call, tag;
    asm volatile(ML_MACROS ML_LOOP
                 ".macro XL k\n"
                 ".if (\\k) < 4\n"
                 "global_load_dwordx4 v[8+4*(\\k):11+4*(\\k)], %[vl], %[xp] offset:1024*(\\k) sc1\n"
                 ".elseif (\\k) < 7\n"
                 "global_load_dwordx4 v[8+4*(\\k):11+4*(\\k)], %[vl], %[xp2] offset:1024*((\\k)-4) sc1\n"
                 ".else\n"
                 "global_load_dwordx4 v[36:39], %[vl], %[xp2] offset:%c[xlast] sc1\n"
                 ".endif\n"
                 ".endm\n"
                 ".set ml_s, %c[nloop]\n.set ml_x, 0\n"
                 ".rept %c[xd]-1\n"
                 "PH 0, ml_s&3, 1, 0, ml_x\ns_barrier\nPH 1, (ml_s+1)&3, 1, 0, ml_x+3\n"
                 ".set ml_s, ml_s+1\n.set ml_x, ml_x+6\n.endr\n"
                 "PH 0, ml_s&3, 1, 0, ml_x\n"
                 "ds_read_b64 v[88:89], %[cta]\n"
                 "PH 1, 0, 0, 0\n"
                 "v_readfirstlane_b32 %[call], v88\n"
                 "v_readfirstlane_b32 %[tag], v89\n"
                 ".purgem XL\n"
                 "s_waitcnt vmcnt(0)\n"
                 ML_END
                 : "=&{v[40:55]}"(a0), "=&{v[56:71]}"(a1), "=&{v[72:87]}"(a2), "=&{v[8:23]}"(x0), "=&{v[24:39]}"(x1), [cnt] "=&s"(lc), [call] "=&s"(call), [tag] "=&s"(tag)
                 : [pw0] "v"(pw[0]), [pw1] "v"(pw[1]), [pa00] "v"(pa[0]), [pa01] "v"(pa[0] + 2 * ASLOT),
                   [pa10] "v"(pa[1]), [pa11] "v"(pa[1] + 2 * ASLOT), [mt] "n"(MT), [aslot] "n"(ASLOT), [nloop] "n"(NS - XD),
                   [vl] "v"(vl), [xp2] "s"(xp2), [xp] "s"(xp), [xlast] "n"((NQ - 5) * 1024), [xd] "n"(XD), [cta] "v"(lds0 + LDS_CTL + C_CALL * 4)
                 : "memory", "scc", ML_FRAGS);
    u32x4 in[8];
#pragma unroll
    for (int q = 0; q < 8; ++q)
#pragma unroll
        for (int e = 0; e < 4; ++e) in[q][e] = q < 4 ? x0[4 * q + e] : x1[4 * (q - 4) + e];
    f32x4 acc[2 * MT];
#pragma unroll
    for (int n = 0; n < NV; ++n) acc[n >> 2][n & 3] = ACC(n);
#define XLOADS(TAIL)                                                                                                             \
    asm volatile("global_load_dwordx4 %0, %8, %10 sc1\n\tglobal_load_dwordx4 %1, %8, %10 offset:1024 sc1\n\t"                    \
                 "global_load_dwordx4 %2, %8, %10 offset:2048 sc1\n\tglobal_load_dwordx4 %3, %8, %10 offset:3072 sc1\n\t"        \
                 "global_load_dwordx4 %4, %8, %9 sc1\n\tglobal_load_dwordx4 %5, %8, %9 offset:1024 sc1\n\t"                    \
                 "global_load_dwordx4 %6, %8, %9 offset:2048 sc1\n\tglobal_load_dwordx4 %7, %8, %9 offset:%11 sc1" TAIL        \
                 : "=&v"(in[0]), "=&v"(in[1]), "=&v"(in[2]), "=&v"(in[3]), "=&v"(in[4]), "=&v"(in[5]), "=&v"(in[6]), "=&v"(in[7]) \
                 : "v"(vl), "s"(xp2), "s"(xp), "n"((NQ - 5) * 1024) : "memory")
    if (wid == 7) STAMP(20);
    {
        u32 bad = 0;
#pragma unroll
        for (int q = 0; q < 8; ++q) bad |= (in[q][1] ^ tag) | (in[q][3] ^ tag);
        if (!__all(bad < 0x10000u)) {
            const u64 t0 = rt();
            u32 spins = 0;
            for (;;) {
                XLOADS("\n\ts_waitcnt vmcnt(0)");
                bad = 0;
#pragma unroll
                for (int q = 0; q < 8; ++q) bad |= (in[q][1] ^ tag) | (in[q][3] ^ tag);
                if (__all(bad < 0x10000u)) break;
                if (expired(tmo, call, t0, spins)) break;
            }
        }
    }
    if (wid == 0) STAMP(21);
    float m = 0.f;
    {
#pragma unroll
        for (int n = 0; n < NV; ++n) {
            const int q = n / 6, hf = (n % 6) / 3, e = n % 3;
            const u32 w = in[q][2 * hf + (e >> 1)];
            acc[n >> 2][n & 3] += __uint_as_float(e == 1 ? w & 0xffff0000u : w << 16);
            m = fmaxf(m, acc[n >> 2][n & 3]);
        }
    }
    const u32 wm = wave_max(__float_as_uint(m));
    u32 arr, mb;
    u64 mxp;
    asm volatile("s_mov_b64 exec, 1\n\t"
                 "ds_max_u32 %3, %4 offset:%6\n\t"
                 "ds_add_rtn_u32 %0, %3, %5 offset:%7\n\t"
                 "s_mov_b64 exec, -1\n\t"
                 "ds_read_b32 %1, %3 offset:%6\n\t"
                 "ds_read_b64 %2, %8\n\t"
                 "s_waitcnt lgkmcnt(0)"
                 : "=&v"(arr), "=&v"(mb), "=&v"(mxp)
                 : "v"(lds0 + LDS_CTL), "v"(wm), "v"(1u), "n"(C_OWN * 4), "n"(C_CNTA * 4), "v"(lds0 + LDS_CTL + 64 + (lane & 7) * 8) : "memory");
    if (rfl(arr) == 7) {
        if (lane < NR) __hip_atomic_store((GAS u64*)mxp, ((u64)call << 32) | mb, __ATOMIC_RELAXED, __HIP_MEMORY_SCOPE_SYSTEM);
        if (lane == 0) __hip_atomic_store(cntp, call, __ATOMIC_RELAXED, __HIP_MEMORY_SCOPE_AGENT);
        STAMP(18);
    }
    {
        LAS char* tp = (LAS char*)smem + (rh + (lane & 15)) * 528 + wn * 128 + (lane >> 4) * 16;
#pragma unroll
        for (int n = 0; n < 2 * MT; ++n)
            *(LAS f32x4*)(tp + tile_base(rh + (n >> 1) * 16) + (n >> 1) * (16 * 528) + (n & 1) * 64) = acc[n];
        asm volatile("s_mov_b64 exec, 1\n\tds_add_u32 %0, %1 offset:%2\n\ts_mov_b64 exec, -1"
                     ::"v"(lds0 + LDS_CTL), "v"(1u), "n"(C_CNTB * 4) : "memory");
    }
    finish(P, smem, lds0, lane, wid, myws, tmo, call, row0, col0);
}

#define G4S(GS)                                                                                                       \
    "global_load_lds_dwordx4 %[v0], s[60:61] offset:0\n\tglobal_load_lds_dwordx4 %[v1], s[60:61] offset:1024\n\t"     \
    "global_load_lds_dwordx4 %[v2], s[60:61] offset:2048\n\tglobal_load_lds_dwordx4 %[v3], s[60:61] offset:3072\n\t"  \
    "s_add_u32 m0, m0, 0x1000\n\ts_add_u32 s60, s60, " GS "\n\ts_addc_u32 s61, s61, 0\n\t"
#define OWN_ISSUE                                                                                                     \
    "s_mov_b32 m0, %[l]\n\ts_mov_b32 s60, %[b0]\n\ts_mov_b32 s61, %[b1]\n\t"                                          \
    G4S("%[gs]") G4S("%[gs]") G4S("%[gs]") G4S("%[gs]")                                                               \
    "s_cmp_lg_u32 %[isw], 0\n\ts_cbranch_scc1 1f\n\t"                                                                 \
    G4S("%[gs]")                                                                                                      \
    "global_load_lds_dwordx4 %[v0], s[60:61] offset:0\n\tglobal_load_lds_dwordx4 %[v1], s[60:61] offset:1024\n"       \
    "1:"

__global__ void __launch_bounds__(NTHREADS) fused_a2a_gemm(Params P) {
    extern __shared__ __attribute__((aligned(1024))) char smem[];
    const u32 lds0 = (u32)(size_t)(LAS char*)smem;
    int lane = threadIdx.x & 63;
    const int wid = rfl(threadIdx.x >> 6);
    const int q = ((blockIdx.x & 31) * 37) >> 8, pi = (blockIdx.x & 31) - 7 * q;
    const char* psrc = P.psrc[pi];
    char* pws = P.pws[pi];
    char* myws_ = P.myws;
    const char *px = P.x, *pw_ = P.w;
    int rank_ = P.rank;
    asm volatile("" : "+s"(psrc), "+s"(pws), "+s"(myws_), "+s"(rank_), "+s"(px), "+s"(pw_));
    const int rank = rank_ & 7;
    const int xcd = blockIdx.x & 7, idx = 31 - (blockIdx.x >> 3);
    char* const myws = myws_;
    GAS u32* const cntp = (GAS u32*)(myws + WS_CNT) + blockIdx.x * 32;
    GAS u32* const tmo = (GAS u32*)(myws + WS_TMO);

    u32 pcall = 0;
    if (blockIdx.x < 21 && wid < (q == 2 ? 10 : 11)) {
        const int row0 = q * 176 + wid * 16;
        const char* src = psrc + row0 * 1024;
        const u32 vs0 = (lane >> 3) * 1024 + (lane & 7) * 16, vs1 = vs0 + 8192;
        u32x4 d[8][2];
        u32 cnt;
#pragma unroll
        for (int j = 0; j < LD_LA; ++j) {
            asm volatile("global_load_dwordx4 %0, %2, %4 offset:%5\n\tglobal_load_dwordx4 %1, %3, %4 offset:%5"
                         : "=&v"(d[j][0]), "=&v"(d[j][1]) : "v"(vs0), "v"(vs1), "s"(src), "n"(j * 128) : "memory");
            if (j == 0) asm volatile("global_load_dword %0, %1, off sc1" : "=&v"(cnt) : "v"(cntp) : "memory");
        }
        char* dst = pws + WS_STAGE + (size_t)(rank * 8) * 65536 + (size_t)row0 * 128;
        char* flag = pws + WS_FLAG + (rank * 4 + q) * 128;
        asm volatile("s_waitcnt vmcnt(%3)" : "+v"(cnt), "+s"(dst), "+s"(flag) : "n"(2 * LD_LA - 2) : "memory");
        const u32 call = rfl(cnt) + 1;
        dst += (size_t)(call % NBUF) * STAGE_BYTES;
        pcall = call;
        push_stage<0>(d, vs0, vs1, src, dst, flag, call, wid, lane * 16);
        if (threadIdx.x == 0) __hip_atomic_store(cntp, call, __ATOMIC_RELAXED, __HIP_MEMORY_SCOPE_AGENT);
        if (wid == 0) STAMP(1);
    }

    const int kh = xcd & 1;
    const int tm = idx % 3, tn = idx / 3;
    const int row0 = tm == 2 ? 336 : tm * 176;
    const int col0 = (xcd >> 1) * 1024 + tn * 128;
    const int owner = (tn ^ kh) & 1;
    const int ns = owner ? NS : NSH;

    const int rot = idx < 24 ? (ROTA * tm + ROTB * tn) & 3 : 0;
    u32 lv[4];
    if (idx < 24 && (wid == 8 || wid == 9)) {
        const bool isw = wid == 8;
        const char* base = isw ? pw_ + ((size_t)((rank * 8 + 2 * rot + kh) & KMASK) << 19) + col0 * 2
                               : px + (size_t)(rank * 512 + row0) * 1024 + (2 * rot + kh) * 128;
        const u32 bw = (lane >> 4) * 8192 + ((lane & 15) ^ ((lane >> 4) << 2)) * 16, ba = (lane >> 3) * 1024 + ((lane & 7) ^ (lane >> 4)) * 16;
        const u32 sel = 0u - (u32)(9 - wid), b0 = (bw & sel) | (ba & ~sel);
        const u32 st = isw ? 4 * 8192 - 1024 : 8192 - 1024, x1 = isw ? 0 : 64, x2 = isw ? 32 : 0;
        lv[0] = b0; lv[1] = (b0 ^ x1) + st; lv[2] = (b0 ^ x2) + 2 * st; lv[3] = (b0 ^ (x1 | x2)) + 3 * st;
        asm volatile(OWN_ISSUE ::[v0] "v"(lv[0]), [v1] "v"(lv[1]), [v2] "v"(lv[2]), [v3] "v"(lv[3]),
                     [b0] "s"((u32)(size_t)base), [b1] "s"((u32)((size_t)base >> 32)), [l] "s"(lds0 + (isw ? LDS_W : LDS_A)),
                     [gs] "s"(isw ? 16 * 8192 : 32768), [isw] "s"(9 - wid) : "memory", "scc", "s60", "s61");
    }

    if (idx < 24 && wid < 8) {
        if (wid < 4 && tm < 2) {
            if (owner) consume<6, 1>(P, smem, lds0, lane, wid, myws, cntp, tmo, row0, 0, col0);
            else       consume<6, 0>(P, smem, lds0, lane, wid, myws, cntp, tmo, row0, 0, col0);
        } else {
            const int rh = wid < 4 ? 16 : 96;
            if (owner) consume<5, 1>(P, smem, lds0, lane, wid, myws, cntp, tmo, row0, rh, col0);
            else       consume<5, 0>(P, smem, lds0, lane, wid, myws, cntp, tmo, row0, rh, col0);
        }
        return;
    }
    asm volatile("" : "+v"(lane));

    int ktab, kfull;
    {
        const int sl = lane < 4 ? (lane + rot) & 3 : lane;
        const int g = min(sl < NSH ? 2 * sl + kh : sl + NSH, NG - 1);
        const int tt = g - 8, j = (tt * 37) >> 8, i = tt - 7 * j;
        kfull = g < 8 ? rank * 8 + g : ((rank + 1 + i) & 7) * 8 + j;
        ktab = kfull & KMASK;
    }

    if (idx >= 24) {
        static_assert(TW <= NSH, "");
        if (wid < 4) {
            if (idx >= 29) warm_w(pw_, lds0 + wid * 256, ktab, lane, (idx - 29) * 4 + wid, 12, xcd, TW, NS);
            else           warm_w(pw_, lds0 + wid * 256, ktab, lane, (idx - 24) * 4 + wid, 20, xcd, T0, TW);
        }
        if (wid == 0) STAMP(idx >= 29 ? 23 : 19);
        if (SUMM && blockIdx.x == 0 && wid < 8) summarise(smem, lds0, lane, wid, rank, myws, tmo, pcall);
        return;
    }

    const u32 scha = lds0 + LDS_SCHED + lane * 4;
#define LOADER_LOOP(ISSUE, WAITN)                                                                                     \
    "1:\n\t"                                                                                                          \
    "s_mov_b32 %[r], %[nspin]\n\t"                                                                                    \
    "s_add_u32 %[cap], %[t], 3\n\t"                                                                                   \
    "s_min_u32 %[cap], %[cap], %[ns]\n"                                                                               \
    "6:\n\t"                                                                                                          \
    "s_cmp_ge_u32 %[u], %[cap]\n\t"                                                                                   \
    "s_cbranch_scc1 3f\n\t"                                                                                           \
    "s_cmp_ge_u32 %[u], %[lim]\n\t"                                                                                   \
    "s_cbranch_scc1 8f\n"                                                                                             \
    "2:\n\t"                                                                                                          \
    "v_readlane_b32 %[so], %[x], %[u]\n\t"                                                                            \
    "v_readlane_b32 %[c], %[ltab], %[u]\n\t"                                                                          \
    "s_lshl_b32 %[so], %[so], %[sh]\n\t"                                                                              \
    "s_mov_b32 m0, %[c]\n\t"                                                                                          \
    "s_add_u32 s60, %[b0], %[so]\n\t"                                                                                 \
    "s_addc_u32 s61, %[b1], 0\n\t"                                                                                    \
    ISSUE                                                                                                             \
    "s_add_u32 %[u], %[u], 1\n\t"                                                                                     \
    "s_mov_b32 %[r], 1\n\t"                                                                                           \
    "s_cmp_lt_u32 %[u], %[cap]\n\t"                                                                                   \
    "s_cbranch_scc1 6b\n"                                                                                             \
    "3:\n\t"                                                                                                          \
    "s_sub_u32 %[c], %[u], %[t]\n\t"                                                                                  \
    "s_cmp_lt_u32 %[c], 3\n\t"                                                                                        \
    "s_cbranch_scc1 5f\n\t"                                                                                           \
    WAITN                                                                                                             \
    "4:\n\t"                                                                                                          \
    "s_add_u32 %[t], %[t], 1\n\t"                                                                                     \
    "s_cmp_lt_u32 %[t], %[tend]\n\t"                                                                                  \
    "s_barrier\n\t"                                                                                                   \
    "s_cbranch_scc1 1b\n\t"                                                                                           \
    "s_branch 9f\n"                                                                                                   \
    "8:\n\t"              \
    "ds_read_b32 %[x], %[scha]\n\t"                                                                                   \
    "s_sub_u32 %[r], %[r], 1\n\t"                                                                                     \
    "s_sub_u32 %[c], %[u], %[t]\n\t"                                                                                  \
    "s_waitcnt lgkmcnt(0)\n\t"                                                                                        \
    "v_cmp_eq_u32 vcc, 0, %[x]\n\t"                                                                                   \
    "s_ff1_i32_b64 %[lim], vcc\n\t"                                                                                   \
    "s_cmp_lt_u32 %[u], %[lim]\n\t"                                                                                   \
    "s_cbranch_scc1 2b\n\t"                                                                                           \
    "s_cmp_lt_u32 %[c], 2\n\t"                                                                                        \
    "s_cbranch_scc1 8b\n\t"                                                                                           \
    "s_cmp_gt_i32 %[r], 0\n\t"                                                                                        \
    "s_cbranch_scc1 8b\n\t"                                                                                           \
    "s_getreg_b32 %[c], hwreg(HW_REG_IB_STS)\n\t"                         \
    "s_and_b32 %[c], %[c], 0xc0000f\n\t"                                                                              \
    "s_cbranch_scc1 8b\n\t"                                                                                           \
    "s_branch 3b\n"                                                                                                   \
    "5:\n\t"                                                                                                          \
    "s_waitcnt vmcnt(0)\n\t"                                                                                          \
    "s_branch 4b\n"                                                                                                   \
    "9:"

    if (wid == 8) {
        __builtin_amdgcn_s_setprio(2);
        const u32 ltab = lds0 + LDS_W + (lane & (RING - 1)) * 16384;
        const char* wb = pw_ - (1u << 19) + col0 * 2;
        u32 x = ktab + 1, cap, c, so, r;
        int t = -1, u = 1, lim = 4;
        if (B0EARLY) { WAITV(0); BAR(); t = 0; }
#define W_ISSUE G4S("0x20000") G4S("0x20000") G4S("0x20000") G4S("0x20000")
        auto run = [&](int tend) {
            asm volatile(LOADER_LOOP(W_ISSUE, "s_waitcnt vmcnt(16)\n")
                         : [x] "+v"(x), [cap] "=&s"(cap), [c] "=&s"(c), [so] "=&s"(so), [r] "=&s"(r), [t] "+s"(t), [u] "+s"(u), [lim] "+s"(lim)
                         : [ltab] "v"(ltab), [scha] "v"(scha), [b0] "s"((u32)(size_t)wb), [b1] "s"((u32)((size_t)wb >> 32)),
                           [v0] "v"(lv[0]), [v1] "v"(lv[1]), [v2] "v"(lv[2]), [v3] "v"(lv[3]),
                           [sh] "n"(19), [ns] "s"(ns), [nspin] "n"(ASPIN), [tend] "s"(tend)
                         : "memory", "scc", "vcc", "s60", "s61");
        };
        run(ns - 1);
        STAMP(2);
        if (owner && SUMM) watch_summaries(smem, lds0, lane, myws);
        return;
    }

    if (wid == 10) {
        static_assert(NS - 4 <= 32, "");
        const int ck = __shfl(kfull, min(4 + (lane & 31), NS - 1));
        const u32 vf = ((ck >> 3) * 4 + tm) * 128;
        u32 cnt, fl0;
        asm volatile("global_load_dword %0, %2, off sc1\n\tglobal_load_dword %1, %3, %4 sc0 sc1\n\ts_waitcnt vmcnt(0)"
                     : "=&v"(cnt), "=&v"(fl0) : "v"(cntp), "v"(vf), "s"(myws + WS_FLAG) : "memory");
        const u32 call = rfl(cnt) + 1;
        const u32 thr = call * 16 + (ck & 7) + 1;
        const u32 ent = (ck & KMASK) + 1;
        constexpr u32 ALL = NS - 4 == 32 ? 0xffffffffu : (1u << ((NS - 4) & 31)) - 1, FULL = (1u << (NSH - 4)) - 1;
        const u32 nw0 = EARLYP ? (u32)__ballot(fl0 >= thr) & (owner ? ALL : FULL) : 0u;
        *(volatile LAS u32*)(size_t)scha = lane < 4 ? ktab + 1 : 0;
        if (lane < 16) ((volatile LAS u32*)(smem + LDS_CTL))[lane] = 0;
        *(volatile LAS u32x2*)(smem + LDS_CTL + C_CALL * 4) = u32x2{call, (call % 65535u + 1u) << 16};
        if (lane < 32 && ((nw0 >> lane) & 1))
            *(volatile LAS u32*)(smem + LDS_SCHED + (4 + __popc(nw0 & ((1u << lane) - 1))) * 4) = ent;
        WAITL(0);
        u32 fl, tmp, c, r, nw, t = KBU - 1, pend = 0, lim = 4 + __popc(nw0), done = ~(owner ? ALL : FULL) | nw0, spins = 0, resume = 1;
#pragma unroll 1
        for (int e = 0; e < KBU; ++e) BAR();
        u64 t0 = 0;
        while ((int)lim < ns) {
            asm volatile(
                "s_cmp_lg_u32 %[res], 0\n\t"
                "s_cbranch_scc1 7f\n"
                "1:\n\t"
                "s_barrier\n"
                "7:\n\t"
                "s_mov_b32 %[res], 64\n\t"
                "s_cmp_lt_u32 %[t], 3\n\t"
                "s_cselect_b32 %[r], %[nspin0], %[nspin]\n"
                "6:\n\t"
                "s_cmp_eq_u32 %[pend], 0\n\t"
                "s_cbranch_scc1 2f\n\t"
                "s_getreg_b32 %[c], hwreg(HW_REG_IB_STS, 0, 4)\n\t"
                "s_cmp_lg_u32 %[c], 0\n\t"
                "s_cbranch_scc1 4f\n"
                "5:\n\t"
                "s_waitcnt vmcnt(0)\n\t"
                "v_cmp_le_u32 vcc, %[thr], %[fl]\n\t"
                "s_mov_b32 %[pend], 0\n\t"
                "s_andn2_b32 %[nw], vcc_lo, %[done]\n\t"
                "s_cbranch_scc0 2f\n\t"
                "v_mbcnt_lo_u32_b32 %[tmp], %[nw], 0\n\t"
                "s_or_b32 %[done], %[done], %[nw]\n\t"
                "v_add_u32 %[tmp], %[lim], %[tmp]\n\t"
                "s_bcnt1_i32_b32 %[c], %[nw]\n\t"
                "v_lshl_add_u32 %[tmp], %[tmp], 2, %[sch0]\n\t"
                "s_add_u32 %[lim], %[lim], %[c]\n\t"
                "s_mov_b32 exec_lo, %[nw]\n\t"
                "s_mov_b32 exec_hi, 0\n\t"
                "ds_write_b32 %[tmp], %[ent]\n\t"
                "s_mov_b64 exec, -1\n\t"
                "s_sub_u32 %[r], %[r], 1\n\t"
                "s_cmp_ge_u32 %[lim], %[ns]\n\t"
                "s_cbranch_scc1 9f\n"
                "2:\n\t"
                "s_add_u32 %[c], %[t], %[reach]\n\t"
                "s_cmp_gt_u32 %[lim], %[c]\n\t"
                "s_cbranch_scc1 3f\n\t"
                "s_mov_b32 exec_hi, 0\n\t"
                "global_load_dword %[fl], %[vf], %[flagp] sc0 sc1\n\t"
                "s_mov_b32 exec_hi, -1\n\t"
                "s_mov_b32 %[pend], 1\n"
                "4:\n\t"
                "s_sub_u32 %[r], %[r], 1\n\t"
                "s_cmp_gt_i32 %[r], 0\n\t"
                "s_cbranch_scc1 6b\n\t"
                "s_add_u32 %[c], %[t], 1\n\t"
                "s_cmp_gt_u32 %[lim], %[c]\n\t"
                "s_cbranch_scc1 3f\n\t"
                "s_sub_u32 %[res], %[res], 1\n\t"
                "s_cmp_gt_u32 %[res], 1\n\t"
                "s_cbranch_scc1 5b\n\t"
                "s_branch 9f\n"
                "3:\n\t"
                "s_add_u32 %[t], %[t], 1\n\t"
                "s_branch 1b\n"
                "9:\n\t"
                "s_waitcnt vmcnt(0) lgkmcnt(0)\n\t"
                "s_mov_b32 %[pend], 0"
                : [fl] "=&v"(fl), [tmp] "=&v"(tmp), [c] "=&s"(c), [r] "=&s"(r), [nw] "=&s"(nw), [t] "+s"(t), [pend] "+s"(pend),
                  [lim] "+s"(lim), [done] "+s"(done), [res] "+s"(resume)
                : [thr] "v"(thr), [ent] "v"(ent), [vf] "v"(vf), [flagp] "s"(myws + WS_FLAG), [sch0] "s"(lds0 + LDS_SCHED),
                  [reach] "n"(2 + PMARGIN), [ns] "s"(ns), [nspin] "n"(PSPIN), [nspin0] "n"(PSPIN0)
                : "memory", "vcc", "scc");
            if ((int)lim >= ns) break;
            if (t0 == 0) t0 = rt();
            spins = 63;
            if (expired(tmo, call, t0, spins)) {
                if (lane >= (int)lim && lane < ns) *(volatile LAS u32*)(size_t)scha = ((rank * 8) & KMASK) + 1;
                WAITL(0);
                break;
            }
        }
        return;
    }

    if (wid == 9) {
        __builtin_amdgcn_s_setprio(2);
        u32 va = (row0 + (lane >> 3)) * 128 + ((lane & 7) ^ (lane >> 4)) * 16, vb = (row0 + (lane >> 3)) * 128 + ((lane & 7) ^ (4 + (lane >> 4))) * 16;
        u32 ltab = lds0 + LDS_A + (lane & (RING - 1)) * ASLOT;
        asm volatile("" : "+v"(va), "+v"(vb), "+v"(ltab));
        if (B0EARLY) { WAITV(0); BAR(); }
#pragma unroll 1
        for (int u = 1; u < 4; ++u) {
            const char* base = px + (size_t)(rank * 512 + row0) * 1024 + (2 * ((u + rot) & 3) + kh) * 128;
            asm volatile(OWN_ISSUE ::[v0] "v"(lv[0]), [v1] "v"(lv[1]), [v2] "v"(lv[2]), [v3] "v"(lv[3]),
                         [b0] "s"((u32)(size_t)base), [b1] "s"((u32)((size_t)base >> 32)), [l] "s"(lds0 + LDS_A + u * ASLOT),
                         [gs] "s"(32768), [isw] "s"(0u) : "memory", "scc", "s60", "s61");
            if (u > B0EARLY) { WAITV(22); BAR(); }
        }
        FENCE();
        const u32 call = rfl(*(volatile LAS u32*)(smem + LDS_CTL + C_CALL * 4));
        const char* stg = myws + WS_STAGE + (size_t)(call % NBUF) * STAGE_BYTES - 65536;
        u32 x = 0, cap, c, so, r;
        int t = 2, u = 4, lim = 4;
#define A_G4                                                                                                                          \
    "global_load_lds_dwordx4 %[va], s[60:61] offset:0 sc0 sc1\n\tglobal_load_lds_dwordx4 %[vb], s[60:61] offset:1024 sc0 sc1\n\t"     \
    "global_load_lds_dwordx4 %[va], s[60:61] offset:2048 sc0 sc1\n\tglobal_load_lds_dwordx4 %[vb], s[60:61] offset:3072 sc0 sc1\n\t"  \
    "s_add_u32 m0, m0, 0x1000\n\ts_add_u32 s60, s60, 0x1000\n\ts_addc_u32 s61, s61, 0\n\t"
#define A_ISSUE A_G4 A_G4 A_G4 A_G4 A_G4                                                                                              \
    "global_load_lds_dwordx4 %[va], s[60:61] offset:0 sc0 sc1\n\tglobal_load_lds_dwordx4 %[vb], s[60:61] offset:1024 sc0 sc1\n\t"
        asm volatile(LOADER_LOOP(A_ISSUE, "s_waitcnt vmcnt(22)\n")
                     : [x] "+v"(x), [cap] "=&s"(cap), [c] "=&s"(c), [so] "=&s"(so), [r] "=&s"(r), [t] "+s"(t), [u] "+s"(u), [lim] "+s"(lim)
                     : [ltab] "v"(ltab), [scha] "v"(scha), [b0] "s"((u32)(size_t)stg), [b1] "s"((u32)((size_t)stg >> 32)),
                       [va] "v"(va), [vb] "v"(vb), [sh] "n"(16), [ns] "s"(ns), [nspin] "n"(ASPIN), [tend] "s"(ns - 1)
                     : "memory", "scc", "vcc", "s60", "s61");
        STAMP(3);
        return;
    }

}

extern "C" void kernel_launch(int rank, int nranks, ncclComm_t comm,
                              hipStream_t stream,
                              void* const* d_in, const int* in_sizes, int n_in,
                              void* d_out, int out_size,
                              const int* in_dtypes, int out_dtype,
                              void* const* d_out_all,
                              void* const* d_ws_all, size_t ws_bytes) {
    if (nranks != NR || (unsigned)rank >= (unsigned)NR || ws_bytes < WS_NEED || n_in < 2 ||
        in_sizes[0] != 4096 * 512 || in_sizes[1] != 4096 * 4096 || out_size != 512 * 4096) {
        fprintf(stderr, "kernel_launch: unexpected problem shape\n");
        return;
    }
    HIP_CHECK(hipFuncSetAttribute((const void*)fused_a2a_gemm, hipFuncAttributeMaxDynamicSharedMemorySize, LDS_BYTES));
    Params P;
    P.x = (const char*)d_in[0];
    P.w = (const char*)d_in[1];
    P.out = (char*)d_out;
    for (int p = 0; p < NR; ++p) P.ws[p] = (char*)d_ws_all[p];
    P.myws = P.ws[rank];
    for (int i = 0; i < NR - 1; ++i) {
        const int p = (rank + 1 + i) % NR;
        P.psrc[i] = P.x + (size_t)p * 512 * 1024;
        P.pws[i] = P.ws[p];
    }
    P.rank = rank;
    P.pad = 0;
    hipLaunchKernelGGL(fused_a2a_gemm, dim3(NWG), dim3(NTHREADS), LDS_BYTES, stream, P);
    HIP_CHECK(hipGetLastError());
}
```

```cpp
#include <hip/hip_runtime.h>
#include <rccl/rccl.h>
#include <cstdio>

#define HIP_CHECK(expr) do { hipError_t _e = (expr); if (_e != hipSuccess) { \
    fprintf(stderr, "HIP error %s at %s:%d\n", hipGetErrorString(_e), __FILE__, __LINE__); \
    return; } } while (0)

#define LAS __attribute__((address_space(3)))
#define GAS __attribute__((address_space(1)))
typedef unsigned u32;
typedef unsigned long long u64;
typedef unsigned u32x4 __attribute__((ext_vector_type(4)));
typedef unsigned u32x2 __attribute__((ext_vector_type(2)));
typedef float f32x4 __attribute__((ext_vector_type(4)));
typedef float f32x16 __attribute__((ext_vector_type(16)));
typedef unsigned u32x16 __attribute__((ext_vector_type(16)));

constexpr int NR = 8;
constexpr int NWG = 256;
constexpr int NGEMM = 192;
constexpr int NTHREADS = 704;
constexpr int NG = 64;
#define NTAIL 5
#define XD 2
constexpr int nsh_of(int kh) { return (NG - NTAIL + 1 - kh) / 2; }
constexpr int ns_of(int kh) { return nsh_of(kh) + NTAIL; }
static_assert(NTAIL >= 4 && XD >= 2 && XD <= 3, "");
constexpr size_t WS_CNT = 16384;
constexpr size_t WS_TMO = 0;
constexpr size_t WS_FLAG = 4096;
constexpr size_t WS_MAX = 512u << 10;
constexpr size_t WS_XCH = 9u << 20;
constexpr size_t WS_STAGE = 1u << 20;
constexpr size_t STAGE_BYTES = 4u << 20;
#define NBUF 2
static_assert(WS_STAGE + NBUF * STAGE_BYTES <= WS_XCH, "");
#define XSTR 9216
constexpr size_t WS_NEED = WS_XCH + (size_t)NGEMM * 4 * XSTR;
#define PUSH_LA 2
#define LD_LA 1
#define T0 2
#define PSPIN 5
#define ASPIN 2
#define KBU 3
#define TW 28
#define GS 64
#define XN 3
#define XS 0
#define STR_(x) #x
#define STR(x) STR_(x)
#define XNS STR(XN)
#define XSS STR(XS)
static_assert((2 * XD - XS) * XN >= 8 && XN <= 5, "");
constexpr int RING = 4;
constexpr int ASLOT = 22 * 1024;
constexpr int LDS_W = 0;
constexpr int LDS_A = RING * 16384;
constexpr int LDS_CTL = LDS_A + RING * ASLOT;
constexpr int LDS_SCHED = LDS_CTL + 256 + 256;
constexpr int LDS_BYTES = LDS_SCHED + 256;
static_assert(LDS_BYTES <= 160 * 1024 && LDS_BYTES > 80 * 1024, "one workgroup per CU");
constexpr int C_OWN = 0, C_CNTC = 1, C_CNTA = 2, C_CNTB = 3, C_CALL = 32  ;
#define PMARGIN 1
constexpr u32 BADMAX = 0xffffffffu;
constexpr u64 DEADLINE = 40000000ull;

struct Params {
    const char* x;
    const char* w;
    char* out;
    char* myws;
    int rank;
    int pad;
    const char* psrc[NR - 1];
    char* pws[NR - 1];
    char* ws[NR];
};

#define WAITV(n) asm volatile("s_waitcnt vmcnt(%0)" ::"n"(n) : "memory")
#define WAITL(n) asm volatile("s_waitcnt lgkmcnt(%0)" ::"n"(n) : "memory")
#define FENCE() asm volatile("" ::: "memory")
#define BAR() __builtin_amdgcn_s_barrier()

__device__ __forceinline__ u64 rt() { return __builtin_amdgcn_s_memrealtime(); }
__device__ __forceinline__ u32 rfl(u32 v) { return __builtin_amdgcn_readfirstlane(v); }
__device__ __forceinline__ u32 pk_bf16(float lo, float hi) {
    typedef __bf16 bf16x2 __attribute__((ext_vector_type(2)));
    const bf16x2 v = {(__bf16)lo, (__bf16)hi};
    return __builtin_bit_cast(u32, v);
}
__device__ __forceinline__ bool expired(GAS u32* tmo, u32 call, u64 t0, u32& spins) {
    if ((++spins & 63u) != 0) return false;
    if (rfl(__hip_atomic_load(tmo, __ATOMIC_RELAXED, __HIP_MEMORY_SCOPE_AGENT)) == call || rt() - t0 > DEADLINE) {
        __hip_atomic_store(tmo, call, __ATOMIC_RELAXED, __HIP_MEMORY_SCOPE_AGENT);
        return true;
    }
    return false;
}

struct Sched { int wl[8], ws[8]; };
constexpr Sched mk_sched() {
    Sched s{}; int posL[16] = {}, posS[16] = {}, n = 0;
    for (int j = 0; j < LD_LA; ++j) { n += 2; posL[j] = n; }
    for (int j = 0; j < 8 + PUSH_LA; ++j) {
        if (j >= PUSH_LA) s.ws[j - PUSH_LA] = n - posS[j - PUSH_LA];
        if (j < 8) { s.wl[j] = n - posL[j]; n += 2; posS[j] = n; if (j + LD_LA < 8) { n += 2; posL[j + LD_LA] = n; } }
    }
    return s;
}
constexpr Sched SCHED = mk_sched();

template <int J>
__device__ __forceinline__ void push_stage(u32x4 (&d)[8][2], u32 vs0, u32 vs1, const char* src, char* dst, char* flag,
                                           u32 call, int wid, u32 vd) {
    if constexpr (J >= PUSH_LA) {
        WAITV(SCHED.ws[J - PUSH_LA]);
        BAR();
        if (wid == 0) {
            const u32 val = call * 16 + (J - PUSH_LA) + 1, vz = 0;
            asm volatile("s_mov_b64 exec, 1\n\tglobal_store_dword %0, %1, %2 sc0 sc1\n\ts_mov_b64 exec, -1"
                         ::"v"(vz), "v"(val), "s"(flag) : "memory");
        }
    }
    if constexpr (J < 8) {
        WAITV(SCHED.wl[J]);
        char* p = dst + (size_t)J * 65536;
        asm volatile("global_store_dwordx4 %2, %0, %3 sc0 sc1\n\t"
                     "global_store_dwordx4 %2, %1, %3 offset:1024 sc0 sc1\n\ts_nop 1"
                     : "+v"(d[J][0]), "+v"(d[J][1]) : "v"(vd), "s"(p) : "memory");
        if constexpr (J + LD_LA < 8)
            asm volatile("global_load_dwordx4 %0, %2, %4 offset:%5\n\tglobal_load_dwordx4 %1, %3, %4 offset:%5"
                         : "=&v"(d[J + LD_LA][0]), "=&v"(d[J + LD_LA][1]) : "v"(vs0), "v"(vs1), "s"(src), "n"((J + LD_LA) * 128) : "memory");
    }
    if constexpr (J + 1 < 8 + PUSH_LA) push_stage<J + 1>(d, vs0, vs1, src, dst, flag, call, wid, vd);
}

__device__ __forceinline__ void warm_w(const char* w, u32 lds, int ktab, int lane, int id, int nw, int xcd, int u0, int u1) {
    const u32 gtab = (u32)ktab << 19;
    const u32 vo = (lane >> 4) * 8192 + (xcd >> 1) * 2048 + (lane & 15) * 128;
    const int NSH = nsh_of(xcd & 1);
    const int n1 = (min(u1, NSH) - u0) * 16;
    int n = id;
#pragma unroll 1
    for (; n < n1; n += nw)
        asm volatile("s_mov_b32 m0, %2\n\ts_nop 0\n\tglobal_load_lds_dword %0, %1\n\ts_waitcnt vmcnt(1)"
                     ::"v"(vo), "s"(w + ((u32)__builtin_amdgcn_readlane(gtab, u0 + (n >> 4)) + (n & 15) * 4 * 8192)), "s"(lds) : "memory");
    const u32 vt = (lane >> 3) * 8192 + (xcd >> 1) * 2048 + (((lane >> 1) & 3) * 2 + ((xcd & 1) ^ 1)) * 256 + (lane & 1) * 128;
#pragma unroll 1
    for (n -= n1; n < (u1 - NSH) * 8; n += nw)
        asm volatile("s_mov_b32 m0, %2\n\ts_nop 0\n\tglobal_load_lds_dword %0, %1\n\ts_waitcnt vmcnt(1)"
                     ::"v"(vt), "s"(w + ((u32)__builtin_amdgcn_readlane(gtab, NSH + (n >> 3)) + (n & 7) * 8 * 8192)), "s"(lds) : "memory");
    WAITV(0);
}

#define ML_MFMA                                                                                                     \
    ".if ml_n == 4\n"                                                                                               \
    "s_waitcnt lgkmcnt(6*(\\rd))\n"                                                                                 \
    ".endif\n"                                                                                                      \
    "v_mfma_f32_16x16x32_bf16 v[40+4*ml_n:43+4*ml_n], v[88+32*(\\ks)+4*(ml_n&1):91+32*(\\ks)+4*(ml_n&1)], "         \
    "v[96+32*(\\ks)+4*(ml_n>>1):99+32*(\\ks)+4*(ml_n>>1)], v[40+4*ml_n:43+4*ml_n]\n"
#define ML_MACROS                                                                                                   \
    ".macro RD k, ks, slot\n"                                                                                       \
    ".if (\\k) < 2\n"                                                                                               \
    "ds_read_b64_tr_b16 v[88+32*(\\ks)+2*(\\k):89+32*(\\ks)+2*(\\k)], %[pw0] offset:16384*(\\slot)+8192*(\\ks)+1024*(\\k)\n" \
    ".elseif (\\k) < 4\n"                                                                                           \
    "ds_read_b64_tr_b16 v[88+32*(\\ks)+2*(\\k):89+32*(\\ks)+2*(\\k)], %[pw1] offset:16384*(\\slot)+8192*(\\ks)+1024*((\\k)-2)\n" \
    ".elseif (\\ks) == 0 && (\\slot) < 2\n"                                                                         \
    "ds_read_b128 v[80+4*(\\k):83+4*(\\k)], %[pa00] offset:%c[aslot]*(\\slot)+2048*((\\k)-4)\n"                      \
    ".elseif (\\ks) == 0\n"                                                                                         \
    "ds_read_b128 v[80+4*(\\k):83+4*(\\k)], %[pa01] offset:%c[aslot]*((\\slot)-2)+2048*((\\k)-4)\n"                  \
    ".elseif (\\slot) < 2\n"                                                                                        \
    "ds_read_b128 v[112+4*(\\k):115+4*(\\k)], %[pa10] offset:%c[aslot]*(\\slot)+2048*((\\k)-4)\n"                    \
    ".else\n"                                                                                                       \
    "ds_read_b128 v[112+4*(\\k):115+4*(\\k)], %[pa11] offset:%c[aslot]*((\\slot)-2)+2048*((\\k)-4)\n"                \
    ".endif\n"                                                                                                      \
    ".endm\n"                                                                                                       \
    ".macro PH ks, slot, rd, cnt, xl=8\n"                                                                                 \
    "s_waitcnt lgkmcnt(%c[mt]-2)\n"                                                                                 \
    ".set ml_n, 0\n"                                                                                                \
    ".rept 2*%c[mt]\n"                                                                                              \
    ML_MFMA                                                                                                         \
    ".if (\\rd) && ml_n < 2\n"                                                                                      \
    "RD 2*ml_n, 1-(\\ks), \\slot\n"                                                                                 \
    "RD 2*ml_n+1, 1-(\\ks), \\slot\n"                                                                               \
    ".elseif (\\rd) && ml_n < %c[mt]+2\n"                                                                           \
    "RD ml_n+2, 1-(\\ks), \\slot\n"                                                                                 \
    ".elseif (\\cnt) && ml_n == %c[mt]+2\n"                                                                         \
    "s_sub_u32 %[cnt], %[cnt], 1\n"                                                                                 \
    ".elseif (\\cnt) && ml_n == %c[mt]+3\n"                                                                         \
    "s_cmp_eq_u32 %[cnt], 0\n"                                                                                      \
    ".endif\n"                                                                                                      \
    ".if (ml_n & 1) && (ml_n >> 1) < " XNS " && (\\xl)+(ml_n>>1) < 8 && (\\xl) >= 0\n"                              \
    "XL (\\xl)+(ml_n>>1)\n"                                                                                        \
    ".endif\n"                                                                                                      \
    ".set ml_n, ml_n+1\n"                                                                                           \
    ".endr\n"                                                                                                       \
    ".endm\n"                                                                                                       \
    ".macro STEP s, cnt\n"                                                                                          \
    "PH 0, (\\s)&3, 1, 0\n"                                                                                         \
    "s_barrier\n"                                                                                                   \
    "PH 1, ((\\s)+1)&3, 1, \\cnt\n"                                                                                 \
    ".endm\n"
#define ML_LOOP                                                                                                     \
    ".set ml_n, 40\n.rept 8*%c[mt]\nv_mov_b32 v[ml_n], 0\n.set ml_n, ml_n+1\n.endr\n"                               \
    "s_mov_b32 %[cnt], (%c[nloop]+3)/4\n"                                                                           \
    "s_waitcnt lgkmcnt(0)\n"                                                                                        \
    "s_barrier\n"                                                                                                   \
    ".set ml_k, 0\n.rept %c[mt]+4\nRD ml_k, 0, 0\n.set ml_k, ml_k+1\n.endr\n"                                       \
    "1:\n"                                                                                                          \
    ".set ml_s, 0\n"                                                                                                \
    ".rept 4\n"                                                                                                     \
    ".if ml_s == ((%c[nloop]-1)&3)\n"                                                                               \
    "STEP ml_s, 1\n"                                                                                                \
    "s_cbranch_scc1 2f\n"                                                                                           \
    ".else\n"                                                                                                       \
    "STEP ml_s, 0\n"                                                                                                \
    ".endif\n"                                                                                                      \
    ".set ml_s, ml_s+1\n"                                                                                           \
    ".endr\n"                                                                                                       \
    "s_branch 1b\n"                                                                                                 \
    "2:\n"
#define ML_END "s_nop 7\ns_nop 1\n.purgem RD\n.purgem PH\n.purgem STEP"
#define ML_FRAGS                                                                                                    \
    "v88", "v89", "v90", "v91", "v92", "v93", "v94", "v95", "v96", "v97", "v98", "v99", "v100", "v101", "v102", "v103",       \
    "v104", "v105", "v106", "v107", "v108", "v109", "v110", "v111", "v112", "v113", "v114", "v115", "v116", "v117", "v118", "v119", \
    "v120", "v121", "v122", "v123", "v124", "v125", "v126", "v127", "v128", "v129", "v130", "v131", "v132", "v133", "v134", "v135", \
    "v136", "v137", "v138", "v139", "v140", "v141", "v142", "v143", "v144", "v145", "v146", "v147", "v148", "v149", "v150", "v151"

template <int KH>
__device__ __forceinline__ int tile_base(int r) {
    constexpr int L = (ns_of(KH) - 1) & (RING - 1);
    static_assert(L == 1 || L == 2 || L == 3, "");
    static_assert(48 * 528 <= 2 * 16384 && 3 * 16384 + 112 * 528 <= LDS_A + 2 * ASLOT, "");
    static_assert(2 * 16384 + 96 * 528 <= LDS_A + ASLOT && 80 * 528 <= 2 * ASLOT, "");
    static_assert(80 * 528 <= 3 * 16384 && 96 * 528 <= 3 * ASLOT, "");
    if (L == 1) return r < 96 ? 2 * 16384 : LDS_A + 2 * ASLOT - 96 * 528;
    if (L == 3) return r < 80 ? 0 : LDS_A - 80 * 528;
    return r < 48 ? 0 : (r < 160 ? 3 * 16384 - 48 * 528 : LDS_A + 3 * ASLOT - 160 * 528);
}

template <int KH>
__device__ __forceinline__ void finish(const Params& P, char* smem, u32 lds0, int lane, int wid, char* myws, GAS u32* tmo, u32 call,
                                       int row0, int col0) {
    const int r0 = wid * 8 + (lane >> 3), rlo = row0 == 336 ? 16 : 0;
    char* op = P.out + (size_t)(row0 + r0) * 4096 + col0 + (lane & 7) * 16;
    volatile LAS u32* ctlp = (volatile LAS u32*)(smem + LDS_CTL);
    u32 ta[3];
#pragma unroll
    for (int k = 0; k < 3; ++k) {
        const int rb = min(wid * 8 + 64 * k, 168);
        ta[k] = lds0 + tile_base<KH>(rb) + (rb + (lane >> 3)) * 528 + (lane & 7) * 64;
    }
    while (rfl(ctlp[C_CNTB]) < 8) {}
    FENCE();
    f32x4 y[12];
#pragma unroll
    for (int n = 0; n < 12; ++n) y[n] = *(LAS f32x4*)((LAS char*)(size_t)ta[n >> 2] + (n & 3) * 16);
    char *o1p = op + (size_t)64 * 4096, *o2p = op + (size_t)128 * 4096;
    u64 m0 = __ballot(r0 >= rlo), m2 = __ballot(r0 + 128 < 176);
    asm volatile("" : "+v"(op), "+v"(o1p), "+v"(o2p), "+s"(m0), "+s"(m2));
    {
        const u32 o0 = WS_MAX + (wid * 96 + lane) * GS, o1 = lane < 32 ? o0 + 64 * GS : (u32)WS_TMO;
        u64 p0 = ~0ull, t0 = 0;
        u32 p1 = ~0u, n, mx = 0;
        for (;;) {
            u32x2 a, b;
            asm volatile("s_mov_b32 %[n], 64\n"
                         "1:\n\t"
                         "s_mov_b64 exec, %[p0]\n\tglobal_load_dwordx2 v[2:3], %[o0], %[g] sc0 sc1\n\t"
                         "s_mov_b32 exec_lo, %[p1]\n\ts_mov_b32 exec_hi, 0x80000000\n\tglobal_load_dwordx2 v[4:5], %[o1], %[g] sc0 sc1\n\t"
                         "s_mov_b32 exec_hi, 0\n\t"
                         "s_sub_u32 %[n], %[n], 1\n\t"
                         "s_waitcnt vmcnt(0)\n\t"
                         "v_cmpx_eq_u32 vcc, %[call], v5\n\t"
                         "v_max_u32 %[mx], %[mx], v4\n\t"
                         "s_andn2_b32 %[p1], %[p1], vcc_lo\n\t"
                         "s_mov_b64 exec, %[p0]\n\t"
                         "v_cmpx_eq_u32 vcc, %[call], v3\n\t"
                         "v_max_u32 %[mx], %[mx], v2\n\t"
                         "s_andn2_b64 %[p0], %[p0], vcc\n\t"
                         "s_cbranch_scc1 2f\n\t"
                         "s_cmp_eq_u32 %[p1], 0\n\t"
                         "s_cbranch_scc1 9f\n"
                         "2:\n\t"
                         "s_cmp_lg_u32 %[n], 0\n\t"
                         "s_cbranch_scc1 1b\n"
                         "9:\n\t"
                         "s_mov_b64 exec, -1"
                         : "=&{v[2:3]}"(a), "=&{v[4:5]}"(b), [mx] "+v"(mx), [p0] "+s"(p0), [p1] "+s"(p1), [n] "=&s"(n)
                         : [o0] "v"(o0), [o1] "v"(o1), [g] "s"(myws), [call] "s"(call) : "memory", "vcc", "scc");
            p0 = ((u64)rfl((u32)(p0 >> 32)) << 32) | rfl((u32)p0); p1 = rfl(p1);
            if ((u32)__builtin_amdgcn_readlane(b[0], 63) == call) { mx = BADMAX; break; }
            if (!p0 && !p1) break;
            if (t0 == 0) t0 = rt();
            u32 spins = 63;
            if (expired(tmo, call, t0, spins)) { mx = BADMAX; break; }
        }
        asm volatile("s_nop 1\n\tv_max_u32_dpp %0, %0, %0 row_ror:8 row_mask:0xf bank_mask:0xf\n\t"
                     "s_nop 1\n\tv_max_u32_dpp %0, %0, %0 row_ror:4 row_mask:0xf bank_mask:0xf\n\t"
                     "s_nop 1\n\tv_max_u32_dpp %0, %0, %0 row_ror:2 row_mask:0xf bank_mask:0xf\n\t"
                     "s_mov_b32 exec_lo, 0x30003\n\ts_mov_b32 exec_hi, 0x30003\n\t"
                     "ds_max_u32 %1, %0 offset:%3\n\t"
                     "s_mov_b64 exec, 1\n\t"
                     "ds_add_u32 %1, %2 offset:%4\n\t"
                     "s_mov_b64 exec, -1"
                     : "+v"(mx) : "v"(lds0 + LDS_CTL), "v"(1u), "n"(C_OWN * 4), "n"(C_CNTC * 4) : "memory");
    }
    u32x2 st;
    do st = *(volatile LAS u32x2*)(smem + LDS_CTL + C_OWN * 4); while (rfl(st[1]) < 8);
    const float amax = __uint_as_float(st[0]);
    float inv = amax > 0.f ? 127.0f / amax : 0.f;
    if (rfl(st[0]) == BADMAX) {
        inv = 1.f;
#pragma unroll
        for (int n = 0; n < 12; ++n) y[n] = f32x4{128.f, 128.f, 128.f, 128.f};
    }
#pragma unroll
    for (int k = 0; k < 3; ++k) {
        typedef float f32x2 __attribute__((ext_vector_type(2)));
        const f32x2 iv = {inv, inv};
        f32x2 z[8];
#pragma unroll
        for (int e = 0; e < 8; ++e) {
            const f32x2 in = {y[k * 4 + (e >> 1)][2 * (e & 1)], y[k * 4 + (e >> 1)][2 * (e & 1) + 1]};
            asm volatile("v_pk_mul_f32 %0, %1, %2" : "=v"(z[e]) : "v"(in), "v"(iv));
        }
        u32x4 o = {0, 0, 0, 0};
#pragma unroll
        for (int r = 0; r < 4; ++r) {
#pragma unroll
            for (int g = 0; g < 4; ++g)
                o[g] = __builtin_amdgcn_cvt_pk_u8_f32(z[2 * g + (r >> 1)][r & 1], r, o[g]);
            __builtin_amdgcn_sched_barrier(0);
        }
        asm volatile("s_mov_b64 exec, %2\n\tglobal_store_dwordx4 %0, %1, off\n\ts_mov_b64 exec, -1\n\ts_nop 0"
                     ::"v"(k == 0 ? op : k == 1 ? o1p : o2p), "v"(o), "s"(k == 0 ? m0 : k == 1 ? ~0ull : m2) : "memory");
    }
}

template <int MT, int own, int KH>
__device__ __forceinline__ void consume(const Params& P, char* smem, u32 lds0, int lane, int wid, char* myws, GAS u32* cntp, GAS u32* tmo,
                                        int row0, int rh, int col0) {
    const int wn = wid & 3;
    u32 pa[2], pw[2];
#pragma unroll
    for (int ks = 0; ks < 2; ++ks)
        pa[ks] = lds0 + LDS_A + (rh + (lane & 15)) * 128 + (((4 * ks + (lane >> 4)) ^ ((lane >> 1) & 7)) * 16);
    {
        const u32 g = lane >> 4, q = (lane >> 2) & 3, p = lane & 3;
#pragma unroll
        for (int nb = 0; nb < 2; ++nb) {
            const u32 ch = (wn * 4 + nb * 2 + (p >> 1)) ^ ((q << 2) | ((g & 1) << 1));
            pw[nb] = lds0 + LDS_W + (8 * g + q) * 256 + ch * 16 + (p & 1) * 8;
        }
    }
    static_assert(RING == 4 && LDS_W == 0, "");
    f32x16 a0, a1, a2;
    u32 lc;
    constexpr int NV = 8 * MT, NQ = (NV + 5) / 6;
    u32 vl = lane * 16;
    char* xp = myws + WS_XCH + (size_t)(((blockIdx.x - 64) >> 1) * 8 + wid) * XSTR;
    char* xp2 = xp + 4096;
    asm volatile("" : "+s"(xp), "+s"(xp2), "+v"(vl));
#define ACC(n) ((n) < 16 ? a0[(n) & 15] : (n) < 32 ? a1[(n) & 15] : a2[(n) & 15])
    if constexpr (!own) {
        asm volatile(ML_MACROS ML_LOOP
                     "PH 0, %c[nloop]&3, 1, 0\n"
                     "PH 1, 0, 0, 0\n"
                     ML_END
                     : "=&{v[40:55]}"(a0), "=&{v[56:71]}"(a1), "=&{v[72:87]}"(a2), [cnt] "=&s"(lc)
                     : [pw0] "v"(pw[0]), [pw1] "v"(pw[1]), [pa00] "v"(pa[0]), [pa01] "v"(pa[0] + 2 * ASLOT),
                       [pa10] "v"(pa[1]), [pa11] "v"(pa[1] + 2 * ASLOT), [mt] "n"(MT), [aslot] "n"(ASLOT), [nloop] "n"(nsh_of(KH) - 1)
                     : "memory", "scc", ML_FRAGS);
        const u32x2 ct = *(volatile LAS u32x2*)(smem + LDS_CTL + C_CALL * 4);
        const u32 call = rfl(ct[0]), tag = rfl(ct[1]);
#pragma unroll
        for (int q = 0; q < NQ; ++q) {
            u32x4 o;
#pragma unroll
            for (int hf = 0; hf < 2; ++hf) {
                float f[3];
#pragma unroll
                for (int e = 0; e < 3; ++e) { const int n = q * 6 + hf * 3 + e; f[e] = n < NV ? ACC(n) : 0.f; }
                o[2 * hf] = pk_bf16(f[0], f[1]);
                o[2 * hf + 1] = (pk_bf16(f[2], 0.f) & 0xffffu) | tag;
            }
            asm volatile("global_store_dwordx4 %0, %1, %2 offset:%3 sc1\n\ts_nop 1"
                         ::"v"(vl), "v"(o), "s"(q < 4 ? xp : xp2), "n"((q & 3) * 1024) : "memory");
        }
        if (wid == 0 && lane == 0) __hip_atomic_store(cntp, call, __ATOMIC_RELAXED, __HIP_MEMORY_SCOPE_AGENT);
        return;
    }
    if (wid == 0) {
        char* pw8 = P.ws[0];
#pragma unroll
        for (int p = 1; p < NR; ++p) pw8 = ((lane & 7) == p) ? P.ws[p] : pw8;
        if (lane < NR)
            ((volatile LAS u64*)(smem + LDS_CTL + 64))[lane] = (u64)(pw8 + WS_MAX + ((P.rank & 7) * 96 + ((blockIdx.x - 64) >> 1)) * GS);
    }
    u32x16 x0, x1;
    u32 call, tag;
    asm volatile(ML_MACROS ML_LOOP
                 ".macro XL k\n"
                 ".if (\\k) < 4\n"
                 "global_load_dwordx4 v[8+4*(\\k):11+4*(\\k)], %[vl], %[xp] offset:1024*(\\k) sc1\n"
                 ".elseif (\\k) < %c[nq]\n"
                 "global_load_dwordx4 v[8+4*(\\k):11+4*(\\k)], %[vl], %[xp2] offset:1024*((\\k)-4) sc1\n"
                 ".endif\n"
                 ".endm\n"
                 ".set ml_s, %c[nloop]\n.set ml_x, -" XNS "*" XSS "\n"
                 ".rept %c[xd]-1\n"
                 "PH 0, ml_s&3, 1, 0, ml_x\ns_barrier\nPH 1, (ml_s+1)&3, 1, 0, ml_x+" XNS "\n"
                 ".set ml_s, ml_s+1\n.set ml_x, ml_x+2*" XNS "\n.endr\n"
                 "PH 0, ml_s&3, 1, 0, ml_x\n"
                 "ds_read_b64 v[88:89], %[cta]\n"
                 "PH 1, 0, 0, 0, ml_x+" XNS "\n"
                 "v_readfirstlane_b32 %[call], v88\n"
                 "v_readfirstlane_b32 %[tag], v89\n"
                 ".purgem XL\n"
                 "s_waitcnt vmcnt(0)\n"
                 ML_END
                 : "=&{v[40:55]}"(a0), "=&{v[56:71]}"(a1), "=&{v[72:87]}"(a2), "=&{v[8:23]}"(x0), "=&{v[24:39]}"(x1), [cnt] "=&s"(lc), [call] "=&s"(call), [tag] "=&s"(tag)
                 : [pw0] "v"(pw[0]), [pw1] "v"(pw[1]), [pa00] "v"(pa[0]), [pa01] "v"(pa[0] + 2 * ASLOT),
                   [pa10] "v"(pa[1]), [pa11] "v"(pa[1] + 2 * ASLOT), [mt] "n"(MT), [aslot] "n"(ASLOT), [nloop] "n"(ns_of(KH) - XD),
                   [vl] "v"(vl), [xp2] "s"(xp2), [xp] "s"(xp), [nq] "n"(NQ), [xd] "n"(XD), [cta] "v"(lds0 + LDS_CTL + C_CALL * 4)
                 : "memory", "scc", ML_FRAGS);
#define XW(i) ((i) < 16 ? x0[(i) & 15] : x1[(i) & 15])
    auto tags_ok = [&]() {
        u32 bad = 0;
#pragma unroll
        for (int q = 0; q < NQ; ++q) bad |= __builtin_amdgcn_bitop3_b32(XW(4 * q + 1), XW(4 * q + 3), tag, 0x7e);
        return __all(bad < 0x10000u);
    };
    if (!tags_ok()) {
        const u64 t0 = rt();
        u32 spins = 0;
        do {
            asm volatile("global_load_dwordx4 v[8:11], %2, %3 sc1\n\tglobal_load_dwordx4 v[12:15], %2, %3 offset:1024 sc1\n\t"
                         "global_load_dwordx4 v[16:19], %2, %3 offset:2048 sc1\n\tglobal_load_dwordx4 v[20:23], %2, %3 offset:3072 sc1\n\t"
                         "global_load_dwordx4 v[24:27], %2, %4 sc1\n\tglobal_load_dwordx4 v[28:31], %2, %4 offset:1024 sc1\n\t"
                         "global_load_dwordx4 v[32:35], %2, %4 offset:2048 sc1\n\t"
                         ".if %c5 > 7\n\tglobal_load_dwordx4 v[36:39], %2, %4 offset:3072 sc1\n\t.endif\n\t"
                         "s_waitcnt vmcnt(0)"
                         : "=&{v[8:23]}"(x0), "=&{v[24:39]}"(x1) : "v"(vl), "s"(xp), "s"(xp2), "n"(NQ) : "memory");
        } while (!tags_ok() && !expired(tmo, call, t0, spins));
    }
    f32x4 acc[2 * MT];
#pragma unroll
    for (int n = 0; n < NV; ++n) acc[n >> 2][n & 3] = ACC(n);
    float m4[4] = {0.f, 0.f, 0.f, 0.f};
    {
#pragma unroll
        for (int n = 0; n < NV; ++n) {
            const int q = n / 6, hf = (n % 6) / 3, e = n % 3;
            const u32 w = XW(4 * q + 2 * hf + (e >> 1));
            acc[n >> 2][n & 3] += __uint_as_float(e == 1 ? w & 0xffff0000u : w << 16);
            m4[(n >> 1) & 3] = fmaxf(m4[(n >> 1) & 3], acc[n >> 2][n & 3]);
        }
    }
    const float m = fmaxf(fmaxf(m4[0], m4[1]), fmaxf(m4[2], m4[3]));
    u32 arr, mb, wm = __float_as_uint(m);
    u64 mxp;
    asm volatile("s_nop 1\n\tv_max_u32_dpp %3, %3, %3 row_ror:8 row_mask:0xf bank_mask:0xf\n\t"
                 "s_nop 1\n\tv_max_u32_dpp %3, %3, %3 row_ror:4 row_mask:0xf bank_mask:0xf\n\t"
                 "s_nop 1\n\tv_max_u32_dpp %3, %3, %3 row_ror:2 row_mask:0xf bank_mask:0xf\n\t"
                 "s_mov_b32 exec_lo, 0x30003\n\ts_mov_b32 exec_hi, 0x30003\n\t"
                 "ds_max_u32 %4, %3 offset:%6\n\t"
                 "s_mov_b64 exec, 1\n\t"
                 "ds_add_rtn_u32 %0, %4, %5 offset:%7\n\t"
                 "s_mov_b64 exec, -1\n\t"
                 "ds_read_b32 %1, %4 offset:%6\n\t"
                 "ds_read_b64 %2, %8\n\t"
                 "s_waitcnt lgkmcnt(0)"
                 : "=&v"(arr), "=&v"(mb), "=&v"(mxp), "+v"(wm)
                 : "v"(lds0 + LDS_CTL), "v"(1u), "n"(C_OWN * 4), "n"(C_CNTA * 4), "v"(lds0 + LDS_CTL + 64 + (lane & 7) * 8) : "memory");
    if (rfl(arr) == 7) {
        if (lane < NR) __hip_atomic_store((GAS u64*)mxp, ((u64)call << 32) | mb, __ATOMIC_RELAXED, __HIP_MEMORY_SCOPE_SYSTEM);
        if (lane == 0) __hip_atomic_store(cntp, call, __ATOMIC_RELAXED, __HIP_MEMORY_SCOPE_AGENT);
    }
    {
        LAS char* tp = (LAS char*)smem + (rh + (lane & 15)) * 528 + wn * 128 + (lane >> 4) * 16;
#pragma unroll
        for (int n = 0; n < 2 * MT; ++n)
            *(LAS f32x4*)(tp + tile_base<KH>(rh + (n >> 1) * 16) + (n >> 1) * (16 * 528) + (n & 1) * 64) = acc[n];
        asm volatile("s_mov_b64 exec, 1\n\tds_add_u32 %0, %1 offset:%2\n\ts_mov_b64 exec, -1"
                     ::"v"(lds0 + LDS_CTL), "v"(1u), "n"(C_CNTB * 4) : "memory");
    }
    finish<KH>(P, smem, lds0, lane, wid, myws, tmo, call, row0, col0);
}

#define G4S(GS)                                                                                                       \
    "global_load_lds_dwordx4 %[v0], s[60:61] offset:0\n\tglobal_load_lds_dwordx4 %[v1], s[60:61] offset:1024\n\t"     \
    "global_load_lds_dwordx4 %[v2], s[60:61] offset:2048\n\tglobal_load_lds_dwordx4 %[v3], s[60:61] offset:3072\n\t"  \
    "s_add_u32 m0, m0, 0x1000\n\ts_add_u32 s60, s60, " GS "\n\ts_addc_u32 s61, s61, 0\n\t"
#define OWN_ISSUE                                                                                                     \
    "s_mov_b32 m0, %[l]\n\ts_mov_b32 s60, %[b0]\n\ts_mov_b32 s61, %[b1]\n\t"                                          \
    G4S("%[gs]") G4S("%[gs]") G4S("%[gs]") G4S("%[gs]")                                                               \
    "s_cmp_lg_u32 %[isw], 0\n\ts_cbranch_scc1 1f\n\t"                                                                 \
    G4S("%[gs]")                                                                                                      \
    "global_load_lds_dwordx4 %[v0], s[60:61] offset:0\n\tglobal_load_lds_dwordx4 %[v1], s[60:61] offset:1024\n"       \
    "1:"

__global__ void __launch_bounds__(NTHREADS) fused_a2a_gemm(Params P) {
    extern __shared__ __attribute__((aligned(1024))) char smem[];
    const u32 lds0 = (u32)(size_t)(LAS char*)smem;
    int lane = threadIdx.x & 63;
    const int wid = rfl(threadIdx.x >> 6);
    const int q = ((blockIdx.x & 31) * 37) >> 8, pi = (blockIdx.x & 31) - 7 * q;
    const char* psrc = P.psrc[pi];
    char* pws = P.pws[pi];
    char* myws_ = P.myws;
    const char *px = P.x, *pw_ = P.w;
    int rank_ = P.rank;
    asm volatile("" : "+s"(psrc), "+s"(pws), "+s"(myws_), "+s"(rank_), "+s"(px), "+s"(pw_));
    const int rank = rank_ & 7;
    const int xcd = blockIdx.x & 7, idx = 31 - (blockIdx.x >> 3);
    char* const myws = myws_;
    GAS u32* const cntp = (GAS u32*)(myws + WS_CNT) + blockIdx.x * 32;
    GAS u32* const tmo = (GAS u32*)(myws + WS_TMO);

    if (blockIdx.x < 21 && wid < (q == 2 ? 10 : 11)) {
        const int row0 = q * 176 + wid * 16;
        const char* src = psrc + row0 * 1024;
        const u32 vs0 = (lane >> 3) * 1024 + (lane & 7) * 16, vs1 = vs0 + 8192;
        u32x4 d[8][2];
        u32 cnt;
#pragma unroll
        for (int j = 0; j < LD_LA; ++j) {
            asm volatile("global_load_dwordx4 %0, %2, %4 offset:%5\n\tglobal_load_dwordx4 %1, %3, %4 offset:%5"
                         : "=&v"(d[j][0]), "=&v"(d[j][1]) : "v"(vs0), "v"(vs1), "s"(src), "n"(j * 128) : "memory");
            if (j == 0) asm volatile("global_load_dword %0, %1, off sc1" : "=&v"(cnt) : "v"(cntp) : "memory");
        }
        char* dst = pws + WS_STAGE + (size_t)(rank * 8) * 65536 + (size_t)row0 * 128;
        char* flag = pws + WS_FLAG + (rank * 4 + q) * 128;
        asm volatile("s_waitcnt vmcnt(%3)" : "+v"(cnt), "+s"(dst), "+s"(flag) : "n"(2 * LD_LA - 2) : "memory");
        const u32 call = rfl(cnt) + 1;
        dst += (size_t)(call % NBUF) * STAGE_BYTES;
        push_stage<0>(d, vs0, vs1, src, dst, flag, call, wid, lane * 16);
        if (threadIdx.x == 0) __hip_atomic_store(cntp, call, __ATOMIC_RELAXED, __HIP_MEMORY_SCOPE_AGENT);
    }

    const int kh = xcd & 1;
    const int tm = idx % 3, tn = idx / 3;
    const int row0 = tm == 2 ? 336 : tm * 176;
    const int col0 = (xcd >> 1) * 1024 + tn * 128;
    const int owner = (tn ^ kh) & 1;
    const int NSH = nsh_of(kh), NS = ns_of(kh);
    const int ns = owner ? NS : NSH;

    const int rot = idx < 24 ? (tm + tn) & 3 : 0;
    u32 lv[4];
    if (idx < 24 && (wid == 8 || wid == 9)) {
        const bool isw = wid == 8;
        const char* base = isw ? pw_ + ((size_t)(rank * 8 + 2 * rot + kh) << 19) + col0 * 2
                               : px + (size_t)(rank * 512 + row0) * 1024 + (2 * rot + kh) * 128;
        const u32 bw = (lane >> 4) * 8192 + ((lane & 15) ^ ((lane >> 4) << 2)) * 16, ba = (lane >> 3) * 1024 + ((lane & 7) ^ (lane >> 4)) * 16;
        const u32 sel = 0u - (u32)(9 - wid), b0 = (bw & sel) | (ba & ~sel);
        const u32 st = isw ? 4 * 8192 - 1024 : 8192 - 1024, x1 = isw ? 0 : 64, x2 = isw ? 32 : 0;
        lv[0] = b0; lv[1] = (b0 ^ x1) + st; lv[2] = (b0 ^ x2) + 2 * st; lv[3] = (b0 ^ (x1 | x2)) + 3 * st;
        asm volatile(OWN_ISSUE ::[v0] "v"(lv[0]), [v1] "v"(lv[1]), [v2] "v"(lv[2]), [v3] "v"(lv[3]),
                     [b0] "s"((u32)(size_t)base), [b1] "s"((u32)((size_t)base >> 32)), [l] "s"(lds0 + (isw ? LDS_W : LDS_A)),
                     [gs] "s"(isw ? 16 * 8192 : 32768), [isw] "s"(9 - wid) : "memory", "scc", "s60", "s61");
    }

    if (idx < 24 && wid < 8) {
#define CONSUME(MT, RH)                                                                                    \
    if (owner) { if (kh) consume<MT, 1, (NTAIL & 1)>(P, smem, lds0, lane, wid, myws, cntp, tmo, row0, RH, col0);  \
                 else    consume<MT, 1, 0>(P, smem, lds0, lane, wid, myws, cntp, tmo, row0, RH, col0); }          \
    else       { if (kh) consume<MT, 0, (NTAIL & 1)>(P, smem, lds0, lane, wid, myws, cntp, tmo, row0, RH, col0);  \
                 else    consume<MT, 0, 0>(P, smem, lds0, lane, wid, myws, cntp, tmo, row0, RH, col0); }
        if (wid < 4 && tm < 2) { CONSUME(6, 0) }
        else { const int rh = wid < 4 ? 16 : 96; CONSUME(5, rh) }
        return;
    }
    asm volatile("" : "+v"(lane));

    int ktab;
    {
        const int sl = lane < 4 ? (lane + rot) & 3 : lane;
        const int g = min(sl < NSH ? 2 * sl + kh : sl - NSH + NG - NTAIL, NG - 1);
        const int tt = g - 8, j = (tt * 37) >> 8, i = tt - 7 * j;
        ktab = g < 8 ? rank * 8 + g : ((rank + 1 + i) & 7) * 8 + j;
    }

    if (idx >= 24) {
        static_assert(TW <= nsh_of(1), "");
        if (wid < 4) {
            if (idx >= 29) warm_w(pw_, lds0 + wid * 256, ktab, lane, (idx - 29) * 4 + wid, 12, xcd, TW, NS);
            else           warm_w(pw_, lds0 + wid * 256, ktab, lane, (idx - 24) * 4 + wid, 20, xcd, T0, TW);
        }
        return;
    }

    const u32 scha = lds0 + LDS_SCHED + lane * 4;
#define LOADER_LOOP(ISSUE, WAITN)                                                                                     \
    "1:\n\t"                                                                                                          \
    "s_mov_b32 %[r], %[nspin]\n\t"                                                                                    \
    "s_add_u32 %[cap], %[t], 3\n\t"                                                                                   \
    "s_min_u32 %[cap], %[cap], %[ns]\n"                                                                               \
    "6:\n\t"                                                                                                          \
    "s_cmp_ge_u32 %[u], %[cap]\n\t"                                                                                   \
    "s_cbranch_scc1 3f\n\t"                                                                                           \
    "s_cmp_ge_u32 %[u], %[lim]\n\t"                                                                                   \
    "s_cbranch_scc1 8f\n"                                                                                             \
    "2:\n\t"                                                                                                          \
    "v_readlane_b32 %[so], %[x], %[u]\n\t"                                                                            \
    "v_readlane_b32 %[c], %[ltab], %[u]\n\t"                                                                          \
    "s_lshl_b32 %[so], %[so], %[sh]\n\t"                                                                              \
    "s_mov_b32 m0, %[c]\n\t"                                                                                          \
    "s_add_u32 s60, %[b0], %[so]\n\t"                                                                                 \
    "s_addc_u32 s61, %[b1], 0\n\t"                                                                                    \
    ISSUE                                                                                                             \
    "s_add_u32 %[u], %[u], 1\n\t"                                                                                     \
    "s_mov_b32 %[r], 1\n\t"                                                                                           \
    "s_cmp_lt_u32 %[u], %[cap]\n\t"                                                                                   \
    "s_cbranch_scc1 6b\n"                                                                                             \
    "3:\n\t"                                                                                                          \
    "s_sub_u32 %[c], %[u], %[t]\n\t"                                                                                  \
    "s_cmp_lt_u32 %[c], 3\n\t"                                                                                        \
    "s_cbranch_scc1 5f\n\t"                                                                                           \
    WAITN                                                                                                             \
    "4:\n\t"                                                                                                          \
    "s_add_u32 %[t], %[t], 1\n\t"                                                                                     \
    "s_cmp_lt_u32 %[t], %[tend]\n\t"                                                                                  \
    "s_barrier\n\t"                                                                                                   \
    "s_cbranch_scc1 1b\n\t"                                                                                           \
    "s_branch 9f\n"                                                                                                   \
    "8:\n\t"              \
    "ds_read_b32 %[x], %[scha]\n\t"                                                                                   \
    "s_sub_u32 %[r], %[r], 1\n\t"                                                                                     \
    "s_sub_u32 %[c], %[u], %[t]\n\t"                                                                                  \
    "s_waitcnt lgkmcnt(0)\n\t"                                                                                        \
    "v_cmp_eq_u32 vcc, 0, %[x]\n\t"                                                                                   \
    "s_ff1_i32_b64 %[lim], vcc\n\t"                                                                                   \
    "s_cmp_lt_u32 %[u], %[lim]\n\t"                                                                                   \
    "s_cbranch_scc1 2b\n\t"                                                                                           \
    "s_cmp_lt_u32 %[c], 2\n\t"                                                                                        \
    "s_cbranch_scc1 8b\n\t"                                                                                           \
    "s_cmp_gt_i32 %[r], 0\n\t"                                                                                        \
    "s_cbranch_scc1 8b\n\t"                                                                                           \
    "s_getreg_b32 %[c], hwreg(HW_REG_IB_STS)\n\t"                         \
    "s_and_b32 %[c], %[c], 0xc0000f\n\t"                                                                              \
    "s_cbranch_scc1 8b\n\t"                                                                                           \
    "s_branch 3b\n"                                                                                                   \
    "5:\n\t"                                                                                                          \
    "s_waitcnt vmcnt(0)\n\t"                                                                                          \
    "s_branch 4b\n"                                                                                                   \
    "9:"

    if (wid == 8) {
        __builtin_amdgcn_s_setprio(2);
        const u32 ltab = lds0 + LDS_W + (lane & (RING - 1)) * 16384;
        const char* wb = pw_ - (1u << 19) + col0 * 2;
        u32 x = ktab + 1, cap, c, so, r;
        int t = -1, u = 1, lim = 4;
#define W_ISSUE G4S("0x20000") G4S("0x20000") G4S("0x20000") G4S("0x20000")
        auto run = [&](int tend) {
            asm volatile(LOADER_LOOP(W_ISSUE, "s_waitcnt vmcnt(16)\n")
                         : [x] "+v"(x), [cap] "=&s"(cap), [c] "=&s"(c), [so] "=&s"(so), [r] "=&s"(r), [t] "+s"(t), [u] "+s"(u), [lim] "+s"(lim)
                         : [ltab] "v"(ltab), [scha] "v"(scha), [b0] "s"((u32)(size_t)wb), [b1] "s"((u32)((size_t)wb >> 32)),
                           [v0] "v"(lv[0]), [v1] "v"(lv[1]), [v2] "v"(lv[2]), [v3] "v"(lv[3]),
                           [sh] "n"(19), [ns] "s"(ns), [nspin] "n"(ASPIN), [tend] "s"(tend)
                         : "memory", "scc", "vcc", "s60", "s61");
        };
        run(ns - 1);
        return;
    }

    if (wid == 10) {
        static_assert(ns_of(0) - 4 <= 32, "");
        const int ck = __shfl(ktab, min(4 + (lane & 31), NS - 1));
        const u32 vf = ((ck >> 3) * 4 + tm) * 128;
        u32 cnt, fl0;
        asm volatile("global_load_dword %0, %2, off sc1\n\tglobal_load_dword %1, %3, %4 sc0 sc1\n\ts_waitcnt vmcnt(0)"
                     : "=&v"(cnt), "=&v"(fl0) : "v"(cntp), "v"(vf), "s"(myws + WS_FLAG) : "memory");
        const u32 call = rfl(cnt) + 1;
        const u32 thr = call * 16 + (ck & 7) + 1;
        const u32 ent = ck + 1;
        const u32 mine = 0xffffffffu >> (36 - ns);
        const u32 nw0 = (u32)__ballot(fl0 >= thr) & mine;
        *(volatile LAS u32*)(size_t)scha = lane < 4 ? ktab + 1 : 0;
        if (lane < 16) ((volatile LAS u32*)(smem + LDS_CTL))[lane] = 0;
        *(volatile LAS u32x2*)(smem + LDS_CTL + C_CALL * 4) = u32x2{call, (call % 65535u + 1u) << 16};
        if (lane < 32 && ((nw0 >> lane) & 1))
            *(volatile LAS u32*)(smem + LDS_SCHED + (4 + __popc(nw0 & ((1u << lane) - 1))) * 4) = ent;
        WAITL(0);
        u32 fl, tmp, c, r, nw, t = KBU - 1, pend = 0, lim = 4 + __popc(nw0), done = ~mine | nw0, spins = 0, resume = 1;
#pragma unroll 1
        for (int e = 0; e < KBU; ++e) BAR();
        u64 t0 = 0;
        while ((int)lim < ns) {
            asm volatile(
                "s_cmp_lg_u32 %[res], 0\n\t"
                "s_cbranch_scc1 7f\n"
                "1:\n\t"
                "s_barrier\n"
                "7:\n\t"
                "s_mov_b32 %[res], 64\n\t"
                "s_mov_b32 %[r], %[nspin]\n"
                "6:\n\t"
                "s_cmp_eq_u32 %[pend], 0\n\t"
                "s_cbranch_scc1 2f\n\t"
                "s_getreg_b32 %[c], hwreg(HW_REG_IB_STS, 0, 4)\n\t"
                "s_cmp_lg_u32 %[c], 0\n\t"
                "s_cbranch_scc1 4f\n"
                "5:\n\t"
                "s_waitcnt vmcnt(0)\n\t"
                "v_cmp_le_u32 vcc, %[thr], %[fl]\n\t"
                "s_mov_b32 %[pend], 0\n\t"
                "s_andn2_b32 %[nw], vcc_lo, %[done]\n\t"
                "s_cbranch_scc0 2f\n\t"
                "v_mbcnt_lo_u32_b32 %[tmp], %[nw], 0\n\t"
                "s_or_b32 %[done], %[done], %[nw]\n\t"
                "v_add_u32 %[tmp], %[lim], %[tmp]\n\t"
                "s_bcnt1_i32_b32 %[c], %[nw]\n\t"
                "v_lshl_add_u32 %[tmp], %[tmp], 2, %[sch0]\n\t"
                "s_add_u32 %[lim], %[lim], %[c]\n\t"
                "s_mov_b32 exec_lo, %[nw]\n\t"
                "s_mov_b32 exec_hi, 0\n\t"
                "ds_write_b32 %[tmp], %[ent]\n\t"
                "s_mov_b64 exec, -1\n\t"
                "s_sub_u32 %[r], %[r], 1\n\t"
                "s_cmp_ge_u32 %[lim], %[ns]\n\t"
                "s_cbranch_scc1 9f\n"
                "2:\n\t"
                "s_add_u32 %[c], %[t], %[reach]\n\t"
                "s_cmp_gt_u32 %[lim], %[c]\n\t"
                "s_cbranch_scc1 3f\n\t"
                "s_mov_b32 exec_hi, 0\n\t"
                "global_load_dword %[fl], %[vf], %[flagp] sc0 sc1\n\t"
                "s_mov_b32 exec_hi, -1\n\t"
                "s_mov_b32 %[pend], 1\n"
                "4:\n\t"
                "s_sub_u32 %[r], %[r], 1\n\t"
                "s_cmp_gt_i32 %[r], 0\n\t"
                "s_cbranch_scc1 6b\n\t"
                "s_add_u32 %[c], %[t], 1\n\t"
                "s_cmp_gt_u32 %[lim], %[c]\n\t"
                "s_cbranch_scc1 3f\n\t"
                "s_sub_u32 %[res], %[res], 1\n\t"
                "s_cmp_gt_u32 %[res], 1\n\t"
                "s_cbranch_scc1 5b\n\t"
                "s_branch 9f\n"
                "3:\n\t"
                "s_add_u32 %[t], %[t], 1\n\t"
                "s_branch 1b\n"
                "9:\n\t"
                "s_waitcnt vmcnt(0) lgkmcnt(0)\n\t"
                "s_mov_b32 %[pend], 0"
                : [fl] "=&v"(fl), [tmp] "=&v"(tmp), [c] "=&s"(c), [r] "=&s"(r), [nw] "=&s"(nw), [t] "+s"(t), [pend] "+s"(pend),
                  [lim] "+s"(lim), [done] "+s"(done), [res] "+s"(resume)
                : [thr] "v"(thr), [ent] "v"(ent), [vf] "v"(vf), [flagp] "s"(myws + WS_FLAG), [sch0] "s"(lds0 + LDS_SCHED),
                  [reach] "n"(2 + PMARGIN), [ns] "s"(ns), [nspin] "n"(PSPIN)
                : "memory", "vcc", "scc");
            if ((int)lim >= ns) break;
            if (t0 == 0) t0 = rt();
            spins = 63;
            if (expired(tmo, call, t0, spins)) {
                if (lane >= (int)lim && lane < ns) *(volatile LAS u32*)(size_t)scha = rank * 8 + 1;
                WAITL(0);
                break;
            }
        }
        return;
    }

    if (wid == 9) {
        __builtin_amdgcn_s_setprio(2);
        u32 va = (row0 + (lane >> 3)) * 128 + ((lane & 7) ^ (lane >> 4)) * 16, vb = (row0 + (lane >> 3)) * 128 + ((lane & 7) ^ (4 + (lane >> 4))) * 16;
        u32 ltab = lds0 + LDS_A + (lane & (RING - 1)) * ASLOT;
        asm volatile("" : "+v"(va), "+v"(vb), "+v"(ltab));
#pragma unroll 1
        for (int u = 1; u < 4; ++u) {
            const char* base = px + (size_t)(rank * 512 + row0) * 1024 + (2 * ((u + rot) & 3) + kh) * 128;
            asm volatile(OWN_ISSUE ::[v0] "v"(lv[0]), [v1] "v"(lv[1]), [v2] "v"(lv[2]), [v3] "v"(lv[3]),
                         [b0] "s"((u32)(size_t)base), [b1] "s"((u32)((size_t)base >> 32)), [l] "s"(lds0 + LDS_A + u * ASLOT),
                         [gs] "s"(32768), [isw] "s"(0u) : "memory", "scc", "s60", "s61");
            WAITV(22);
            BAR();
        }
        FENCE();
        const u32 call = rfl(*(volatile LAS u32*)(smem + LDS_CTL + C_CALL * 4));
        const char* stg = myws + WS_STAGE + (size_t)(call % NBUF) * STAGE_BYTES - 65536;
        u32 x = 0, cap, c, so, r;
        int t = 2, u = 4, lim = 4;
#define A_G4                                                                                                                          \
    "global_load_lds_dwordx4 %[va], s[60:61] offset:0 sc0 sc1\n\tglobal_load_lds_dwordx4 %[vb], s[60:61] offset:1024 sc0 sc1\n\t"     \
    "global_load_lds_dwordx4 %[va], s[60:61] offset:2048 sc0 sc1\n\tglobal_load_lds_dwordx4 %[vb], s[60:61] offset:3072 sc0 sc1\n\t"  \
    "s_add_u32 m0, m0, 0x1000\n\ts_add_u32 s60, s60, 0x1000\n\ts_addc_u32 s61, s61, 0\n\t"
#define A_ISSUE A_G4 A_G4 A_G4 A_G4 A_G4                                                                                              \
    "global_load_lds_dwordx4 %[va], s[60:61] offset:0 sc0 sc1\n\tglobal_load_lds_dwordx4 %[vb], s[60:61] offset:1024 sc0 sc1\n\t"
        asm volatile(LOADER_LOOP(A_ISSUE, "s_waitcnt vmcnt(22)\n")
                     : [x] "+v"(x), [cap] "=&s"(cap), [c] "=&s"(c), [so] "=&s"(so), [r] "=&s"(r), [t] "+s"(t), [u] "+s"(u), [lim] "+s"(lim)
                     : [ltab] "v"(ltab), [scha] "v"(scha), [b0] "s"((u32)(size_t)stg), [b1] "s"((u32)((size_t)stg >> 32)),
                       [va] "v"(va), [vb] "v"(vb), [sh] "n"(16), [ns] "s"(ns), [nspin] "n"(ASPIN), [tend] "s"(ns - 1)
                     : "memory", "scc", "vcc", "s60", "s61");
        return;
    }

}

extern "C" void kernel_launch(int rank, int nranks, ncclComm_t comm,
                              hipStream_t stream,
                              void* const* d_in, const int* in_sizes, int n_in,
                              void* d_out, int out_size,
                              const int* in_dtypes, int out_dtype,
                              void* const* d_out_all,
                              void* const* d_ws_all, size_t ws_bytes) {
    if (nranks != NR || (unsigned)rank >= (unsigned)NR || ws_bytes < WS_NEED || n_in < 2 ||
        in_sizes[0] != 4096 * 512 || in_sizes[1] != 4096 * 4096 || out_size != 512 * 4096) {
        fprintf(stderr, "kernel_launch: unexpected problem shape\n");
        return;
    }
    HIP_CHECK(hipFuncSetAttribute((const void*)fused_a2a_gemm, hipFuncAttributeMaxDynamicSharedMemorySize, LDS_BYTES));
    Params P;
    P.x = (const char*)d_in[0];
    P.w = (const char*)d_in[1];
    P.out = (char*)d_out;
    for (int p = 0; p < NR; ++p) P.ws[p] = (char*)d_ws_all[p];
    P.myws = P.ws[rank];
    for (int i = 0; i < NR - 1; ++i) {
        const int p = (rank + 1 + i) % NR;
        P.psrc[i] = P.x + (size_t)p * 512 * 1024;
        P.pws[i] = P.ws[p];
    }
    P.rank = rank;
    P.pad = 0;
    hipLaunchKernelGGL(fused_a2a_gemm, dim3(NWG), dim3(NTHREADS), LDS_BYTES, stream, P);
    HIP_CHECK(hipGetLastError());
}
```
